```python
import numpy as np

kernel_name = "gpu_gemm_a2a_m4096_k4096_n4096_f32_q8"


ABSMAX_RTOL = 0.02

COMPUTE_DTYPE = "float32"
EPILOGUE = "quant_int8"
IN_QUANT = None
OUT_QUANT = "int8"
IN_DTYPES = {"x": "float32", "w_mat": "float32"}
WORLD_SIZE = 8

def setup_inputs():

    x = np.random.randn(4096, 4096).astype(np.float32)
    w_mat = (np.random.randn(4096, 4096) / np.sqrt(4096)).astype(np.float32)
    return {"x": x, "w_mat": w_mat}

def _quant_dequant(y, amax):
    scale = amax / 127.0
    q = np.clip(np.round(y / scale), -127.0, 127.0)
    return (q * scale).astype(np.float32)
def _epilogue(y):
    return _quant_dequant(y, np.abs(y).max())

def reference(x, w_mat):
    y = (x @ w_mat).astype(np.float32)
    return _epilogue(y)
```

```cpp
#include <hip/hip_runtime.h>
#include <rccl/rccl.h>
#include <cstdio>

#define HIP_CHECK(expr) do { hipError_t _e = (expr); if (_e != hipSuccess) { \
    fprintf(stderr, "HIP error %s at %s:%d\n", hipGetErrorString(_e), __FILE__, __LINE__); \
    return; } } while (0)

typedef unsigned u32x4 __attribute__((ext_vector_type(4)));
typedef unsigned u32x2 __attribute__((ext_vector_type(2)));
typedef float f32x4 __attribute__((ext_vector_type(4)));
typedef float f32x2 __attribute__((ext_vector_type(2)));
typedef __bf16 bf16x8 __attribute__((ext_vector_type(8)));
typedef __bf16 bf16x2 __attribute__((ext_vector_type(2)));
typedef unsigned long long u64;
#define GAS __attribute__((address_space(1)))

constexpr int NR = 8;
constexpr int M = 512, N = 4096, K = 4096;
constexpr int NWG = 256;
constexpr int DEPTH = 4;
constexpr int WS_TMO = 0;
constexpr int WS_CNT = 4096;
constexpr int WS_AFLAG = 8192;
constexpr int WS_LOC = 12288;
constexpr int WS_MBOX = 16384;
constexpr int WS_DONE = 20480;
constexpr int WS_AT = 1 << 20;
constexpr size_t WS_NEED = (size_t)WS_AT + (size_t)M * K * 2;
constexpr u64 DEADLINE = 40000000ull;
constexpr int LDS_BYTES = 128 * 1024;

struct Params {
  const float* x;
  const float* w;
  unsigned char* out[NR];
  unsigned char* ws[NR];
  int rank;
  int pad;
};

__device__ __forceinline__ unsigned pk(float a, float b) {
  f32x2 v = {a, b};
  bf16x2 r = __builtin_convertvector(v, bf16x2);
  return __builtin_bit_cast(unsigned, r);
}
__device__ __forceinline__ u64 now() { return __builtin_amdgcn_s_memrealtime(); }
#define RLX_AGENT __ATOMIC_RELAXED, __HIP_MEMORY_SCOPE_AGENT
#define RLX_SYS __ATOMIC_RELAXED, __HIP_MEMORY_SCOPE_SYSTEM

__device__ __forceinline__ bool give_up(GAS unsigned* tmo, unsigned call, u64 t0) {
  bool bad = (__hip_atomic_load(tmo, RLX_AGENT) == call) || (now() - t0 > DEADLINE);
  if (bad) __hip_atomic_store(tmo, call, RLX_AGENT);
  return bad;
}

__global__ void __launch_bounds__(256, 1) fused_gemm_a2a(Params p) {
  extern __shared__ __attribute__((aligned(16))) unsigned char smem[];
  const int tid = threadIdx.x, lane = tid & 63;
  const int w = __builtin_amdgcn_readfirstlane(tid >> 6);
  const int b = blockIdx.x;
  const int xcd = b & 7, j = b >> 3;
  const int mb = j & 3;
  const int nb = (j >> 2) * 8 + xcd;
  const int rank = p.rank;
  unsigned char* ws = p.ws[rank];
  GAS unsigned* tmo = (GAS unsigned*)(ws + WS_TMO);

  const __amdgpu_buffer_rsrc_t rW = __builtin_amdgcn_make_buffer_rsrc((void*)p.w, 0, K * N * 4, 0x00020000);
  const __amdgpu_buffer_rsrc_t rX = __builtin_amdgcn_make_buffer_rsrc((void*)p.x, 0, M * K * 4, 0x00020000);
  const __amdgpu_buffer_rsrc_t rA = __builtin_amdgcn_make_buffer_rsrc((void*)(ws + WS_AT), 0, M * K * 2, 0x00020000);

  const unsigned call = __builtin_amdgcn_readfirstlane(
      __hip_atomic_load((GAS unsigned*)(ws + WS_CNT) + b, RLX_AGENT)) + 1u;
  const unsigned par = call & 1u;

  f32x4 xa[4][2];
  {
    const int t = 2 * nb + (w >> 1);
    const int vo = (lane & 15) * (K * 4) + (lane >> 4) * 32;
#pragma unroll
    for (int f = 0; f < 4; f++) {
      const int i = (w & 1) * 4 + f;
      const int so = (mb * 128 + i * 16) * (K * 4) + t * 128;
      xa[f][0] = __builtin_bit_cast(f32x4, __builtin_amdgcn_raw_buffer_load_b128(rX, vo, so, 0));
      xa[f][1] = __builtin_bit_cast(f32x4, __builtin_amdgcn_raw_buffer_load_b128(rX, vo + 16, so, 0));
    }
  }
  const int vW = (lane >> 4) * 8 * (N * 4) + (lane & 15) * 16;
  const int vA = lane * 16;
  constexpr int NS = 32;
  auto kstep = [&](int s) { return s * 4 + w; };
  f32x4 wb[DEPTH][8];
  u32x4 ab[DEPTH][8];
  auto ldW = [&](int slot, int s) {
    const int so = kstep(s) * 32 * (N * 4) + nb * 256;
#pragma unroll
    for (int jj = 0; jj < 8; jj++)
      wb[slot][jj] = __builtin_bit_cast(f32x4, __builtin_amdgcn_raw_buffer_load_b128(rW, vW, so + jj * (N * 4), 0));
  };
  auto ldA1 = [&](int slot, int s, int i) {
    const int so = (mb * 128 + kstep(s)) * 8192 + i * 1024;
    ab[slot][i] = __builtin_amdgcn_raw_buffer_load_b128(rA, vA, so, 16);
  };
  __builtin_amdgcn_sched_barrier(0);
  ldW(0, 0);
  __builtin_amdgcn_sched_barrier(0);
  {
    const int t = 2 * nb + (w >> 1);
#pragma unroll
    for (int f = 0; f < 4; f++) {
      const int i = (w & 1) * 4 + f;
      u32x4 o;
      o.x = pk(xa[f][0][0], xa[f][0][1]); o.y = pk(xa[f][0][2], xa[f][0][3]);
      o.z = pk(xa[f][1][0], xa[f][1][1]); o.w = pk(xa[f][1][2], xa[f][1][3]);
      __builtin_amdgcn_raw_buffer_store_b128(o, rA, vA, (mb * 128 + t) * 8192 + i * 1024, 16);
    }
  }
  asm volatile("s_waitcnt vmcnt(0)" ::: "memory");
  __builtin_amdgcn_s_barrier();
  if (tid == 0) {
    __hip_atomic_store((GAS unsigned*)(ws + WS_AFLAG) + mb * 64 + nb, call, RLX_AGENT);
    __hip_atomic_store((GAS unsigned*)(ws + WS_CNT) + b, call, RLX_AGENT);
  }
  __builtin_amdgcn_sched_barrier(0);
#pragma unroll
  for (int d = 1; d < DEPTH; d++) ldW(d, d);
  __builtin_amdgcn_sched_barrier(0);
  {
    const u64 t0 = now();
    GAS unsigned* fl = (GAS unsigned*)(ws + WS_AFLAG) + mb * 64 + lane;
    while (true) {
      const unsigned v = __hip_atomic_load(fl, RLX_AGENT);
      if (__all(v >= call)) break;
      if (give_up(tmo, call, t0)) break;
      __builtin_amdgcn_s_sleep(1);
    }
    __builtin_amdgcn_fence(__ATOMIC_ACQUIRE, "wavefront");
  }
  __builtin_amdgcn_sched_barrier(0);
#pragma unroll
  for (int d = 0; d < DEPTH; d++) {
#pragma unroll
    for (int i = 0; i < 8; i++) ldA1(d, d, i);
  }
  __builtin_amdgcn_sched_barrier(0);

  f32x4 acc[8][4];
#pragma unroll
  for (int i = 0; i < 8; i++)
#pragma unroll
    for (int c = 0; c < 4; c++) acc[i][c] = f32x4{0, 0, 0, 0};

  auto step = [&](int slot, int s, bool more) {
    u32x4 bf[4];
#pragma unroll
    for (int c = 0; c < 4; c++) {
      bf[c].x = pk(wb[slot][0][c], wb[slot][1][c]);
      bf[c].y = pk(wb[slot][2][c], wb[slot][3][c]);
      bf[c].z = pk(wb[slot][4][c], wb[slot][5][c]);
      bf[c].w = pk(wb[slot][6][c], wb[slot][7][c]);
    }
    __builtin_amdgcn_sched_barrier(0);
    if (more) ldW(slot, s + DEPTH);
    __builtin_amdgcn_sched_barrier(0);
#pragma unroll
    for (int i = 0; i < 8; i++) {
#pragma unroll
      for (int c = 0; c < 4; c++)
        acc[i][c] = __builtin_amdgcn_mfma_f32_16x16x32_bf16(__builtin_bit_cast(bf16x8, bf[c]),
                                                            __builtin_bit_cast(bf16x8, ab[slot][i]), acc[i][c], 0, 0, 0);
      if (more) ldA1(slot, s + DEPTH, i);
      __builtin_amdgcn_sched_barrier(0);
    }
  };
  for (int s = 0; s < NS - DEPTH; s += DEPTH) {
#pragma unroll
    for (int d = 0; d < DEPTH; d++) step(d, s + d, true);
  }
#pragma unroll
  for (int d = 0; d < DEPTH; d++) step(d, NS - DEPTH + d, false);

  f32x4* red = (f32x4*)smem;
#pragma unroll
  for (int i = 0; i < 8; i++) {
    if ((i >> 1) != w) {
#pragma unroll
      for (int c = 0; c < 4; c++) red[((w * 8 + i) * 4 + c) * 64 + lane] = acc[i][c];
    }
  }
  __syncthreads();
  f32x4 fin[2][4];
  float am = 0.f;
#pragma unroll
  for (int ii = 0; ii < 2; ii++)
#pragma unroll
    for (int c = 0; c < 4; c++) {
      f32x4 v = (w == 0) ? acc[0 + ii][c] : (w == 1) ? acc[2 + ii][c] : (w == 2) ? acc[4 + ii][c] : acc[6 + ii][c];
#pragma unroll
      for (int o = 1; o < 4; o++) {
        const int sw = (w + o) & 3;
        v += red[((sw * 8 + 2 * w + ii) * 4 + c) * 64 + lane];
      }
      fin[ii][c] = v;
      am = fmaxf(am, fmaxf(fmaxf(fabsf(v[0]), fabsf(v[1])), fmaxf(fabsf(v[2]), fabsf(v[3]))));
    }
#pragma unroll
  for (int o = 32; o > 0; o >>= 1) am = fmaxf(am, __shfl_xor(am, o));
  float* wmax = (float*)(smem + ((0 * 8 + 0) * 4) * 64 * 16);
  if (lane == 0) wmax[w] = am;
  __syncthreads();
  if (w == 0) {
    const float m4 = fmaxf(fmaxf(wmax[0], wmax[1]), fmaxf(wmax[2], wmax[3]));
    if (lane == 0)
      __hip_atomic_store((GAS u64*)(ws + WS_LOC) + par * 256 + b,
                         ((u64)call << 32) | (u64)__builtin_bit_cast(unsigned, m4), RLX_AGENT);
  }
  if (b == 0 && w == 1) {
    const u64 t0 = now();
    GAS u64* g = (GAS u64*)(ws + WS_LOC) + par * 256 + lane;
    unsigned mx = 0;
    while (true) {
      bool ok = true;
      mx = 0;
#pragma unroll
      for (int q = 0; q < 4; q++) {
        const u64 v = __hip_atomic_load(g + q * 64, RLX_AGENT);
        ok &= ((unsigned)(v >> 32) == call);
        mx = max(mx, (unsigned)v);
      }
      if (__all(ok)) break;
      if (give_up(tmo, call, t0)) break;
    }
#pragma unroll
    for (int o = 32; o > 0; o >>= 1) mx = max(mx, (unsigned)__shfl_xor((int)mx, o));
    if (lane < NR)
      __hip_atomic_store((GAS u64*)(p.ws[lane] + WS_MBOX + par * 128) + rank, ((u64)call << 32) | (u64)mx, RLX_SYS);
  }
  unsigned gmax = 0;
  {
    const u64 t0 = now();
    GAS u64* g = (GAS u64*)(ws + WS_MBOX + par * 128) + (lane & 7);
    while (true) {
      const u64 v = __hip_atomic_load(g, RLX_SYS);
      gmax = (unsigned)v;
      if (__all((unsigned)(v >> 32) == call)) break;
      if (give_up(tmo, call, t0)) break;
      __builtin_amdgcn_s_sleep(1);
    }
#pragma unroll
    for (int o = 4; o > 0; o >>= 1) gmax = max(gmax, (unsigned)__shfl_xor((int)gmax, o));
  }
  const bool failed = (__hip_atomic_load(tmo, RLX_AGENT) == call);
  const float amax = __builtin_bit_cast(float, gmax);
  const float rs = amax > 0.f ? 127.0f / amax : 0.f;
  const unsigned dst = (unsigned)(nb >> 3);
  if (dst < (unsigned)NR) {
    unsigned char* ob = p.out[dst];
#pragma unroll
    for (int ii = 0; ii < 2; ii++) {
      u32x4 o;
#pragma unroll
      for (int rg = 0; rg < 4; rg++) {
        const unsigned t0 = __builtin_bit_cast(unsigned, __builtin_fmaf(fin[ii][0][rg], rs, 12582912.0f));
        const unsigned t1 = __builtin_bit_cast(unsigned, __builtin_fmaf(fin[ii][1][rg], rs, 12582912.0f));
        const unsigned t2 = __builtin_bit_cast(unsigned, __builtin_fmaf(fin[ii][2][rg], rs, 12582912.0f));
        const unsigned t3 = __builtin_bit_cast(unsigned, __builtin_fmaf(fin[ii][3][rg], rs, 12582912.0f));
        const unsigned lo = __builtin_amdgcn_perm(t1, t0, 0x0c0c0400u);
        const unsigned hi = __builtin_amdgcn_perm(t3, t2, 0x04000c0cu);
        o[rg] = failed ? 0x80808080u : (lo | hi);
      }
      const int row = rank * M + mb * 128 + 16 * (2 * w + ii) + (lane & 15);
      unsigned char* ptr = ob + (size_t)row * 512 + (nb & 7) * 64 + 16 * (lane >> 4);
      asm volatile("global_store_dwordx4 %0, %1, off sc0 sc1\n\ts_nop 1" ::"v"(ptr), "v"(o) : "memory");
    }
    asm volatile("s_waitcnt vmcnt(0)" ::: "memory");
    __builtin_amdgcn_s_barrier();
    if (tid == 0 && dst != (unsigned)rank)
      __hip_atomic_store((GAS unsigned*)(p.ws[dst] + WS_DONE) + rank * 32 + (nb & 7) * 4 + mb, call, RLX_SYS);
  }
  if ((int)(nb >> 3) == rank && (nb & 7) == 0 && mb == 0 && w == 0) {
    const u64 t0 = now();
    GAS unsigned* d = (GAS unsigned*)(ws + WS_DONE) + lane;
    bool bad = false;
    while (true) {
      bool ok = true;
#pragma unroll
      for (int q = 0; q < 4; q++) {
        const int idx = q * 64 + lane;
        const unsigned v = __hip_atomic_load(d + q * 64, RLX_SYS);
        ok &= (v >= call) || ((idx >> 5) == rank);
      }
      if (__all(ok)) break;
      if (give_up(tmo, call, t0)) { bad = true; break; }
    }
    if (bad && lane == 0) p.out[rank][(size_t)rank * M * 512] = 0x80;
  }
}

extern "C" void kernel_launch(int rank, int nranks, ncclComm_t comm, hipStream_t stream,
                              void* const* d_in, const int* in_sizes, int n_in,
                              void* d_out, int out_size, const int* in_dtypes, int out_dtype,
                              void* const* d_out_all, void* const* d_ws_all, size_t ws_bytes) {
  if (nranks != NR || n_in < 2 || ws_bytes < WS_NEED || in_sizes[0] != M * K || in_sizes[1] != K * N) {
    fprintf(stderr, "unexpected problem shape\n");
    return;
  }
  static bool init = false;
  if (!init) {
    HIP_CHECK(hipFuncSetAttribute((const void*)fused_gemm_a2a, hipFuncAttributeMaxDynamicSharedMemorySize, LDS_BYTES));
    init = true;
  }
  Params p;
  p.x = (const float*)d_in[0];
  p.w = (const float*)d_in[1];
  for (int i = 0; i < NR; i++) { p.out[i] = (unsigned char*)d_out_all[i]; p.ws[i] = (unsigned char*)d_ws_all[i]; }
  p.rank = rank;
  p.pad = 0;
  hipLaunchKernelGGL(fused_gemm_a2a, dim3(NWG), dim3(256), LDS_BYTES, stream, p);
  HIP_CHECK(hipGetLastError());
}
```

```cpp
#include <hip/hip_runtime.h>
#include <rccl/rccl.h>
#include <cstdio>
#include <utility>

#define HIP_CHECK(expr) do { hipError_t _e = (expr); if (_e != hipSuccess) { \
    fprintf(stderr, "HIP error %s at %s:%d\n", hipGetErrorString(_e), __FILE__, __LINE__); \
    return; } } while (0)

typedef unsigned u32x4 __attribute__((ext_vector_type(4)));
typedef unsigned u32x2 __attribute__((ext_vector_type(2)));
typedef float f32x4 __attribute__((ext_vector_type(4)));
typedef float f32x2 __attribute__((ext_vector_type(2)));
typedef __bf16 bf16x8 __attribute__((ext_vector_type(8)));
typedef __bf16 bf16x2 __attribute__((ext_vector_type(2)));
typedef unsigned long long u64;
#define GAS __attribute__((address_space(1)))
#define LAS __attribute__((address_space(3)))
#define DI __device__ __forceinline__
#pragma clang diagnostic ignored "-Wint-to-pointer-cast"

constexpr int NR = 8;
constexpr int M = 512, N = 4096, K = 4096;
constexpr int NWG = 256;
constexpr int NSTEP = 32;
constexpr int WS_TMO = 0;
constexpr int WS_CNT = 4096;
constexpr int WS_LOC = 8192;
constexpr int WS_AFLAG = 128 * 1024;
constexpr int WS_DONE = 256 * 1024;
constexpr int WS_MBOX = 384 * 1024;
constexpr int WS_XFLAG = 576 * 1024;
constexpr int WS_AT = 1 << 20;
constexpr int WS_X = 5 << 20;
constexpr int X_BYTES = NWG * 3 * 4 * 4096;
constexpr size_t WS_NEED = (size_t)WS_X + X_BYTES;
constexpr u64 DEADLINE = 40000000ull;
#define NWS 19
constexpr int L_W = 0;
constexpr int L_F = L_W + NWS * 8192;
constexpr int L_SC = L_F - 16384;
constexpr int LDS_BYTES = L_F + 256;
static_assert(LDS_BYTES <= 160 * 1024, "LDS");
constexpr int F_FULLW = 4, F_DONE = 8, F_MAX = 16, F_CNT = 17, F_CALL = 18, F_GMAX = 24, F_GOK = 25, F_ST = 26, F_DR = 27, F_RDY = 28, F_RV = 29;
constexpr int AUX_SYS = 17;
#define CPRIO 3
#define APRIO 0

struct Params {
  const float* x;
  const float* w;
  unsigned char* self;
  int rank;
  int pad;
  unsigned char* out[NR];
  unsigned char* ws[NR];
};

DI unsigned pk(float a, float b) {
  f32x2 v = {a, b};
  bf16x2 r = __builtin_convertvector(v, bf16x2);
  return __builtin_bit_cast(unsigned, r);
}
DI u64 now() { return __builtin_amdgcn_s_memrealtime(); }
#define RLX_AGENT __ATOMIC_RELAXED, __HIP_MEMORY_SCOPE_AGENT
#define RLX_SYS __ATOMIC_RELAXED, __HIP_MEMORY_SCOPE_SYSTEM
#define CFENCE() asm volatile("" ::: "memory")
#define SB() __builtin_amdgcn_sched_barrier(0)

DI bool give_up(GAS unsigned* tmo, unsigned call, u64 t0) {
  bool bad = (__hip_atomic_load(tmo, RLX_AGENT) == call) || (now() - t0 > DEADLINE);
  if (bad) __hip_atomic_store(tmo, call, RLX_AGENT);
  return bad;
}
DI void lds_put(unsigned addr, unsigned v) {
  asm volatile("ds_write_b32 %0, %1" ::"v"(addr), "v"(v) : "memory");
}
DI void lds_put2(unsigned addr, unsigned v) {
  u32x2 t = {v, v};
  asm volatile("ds_write_b64 %0, %1" ::"v"(addr), "v"(t) : "memory");
}
DI unsigned lds_get(unsigned addr) {
  unsigned v;
  asm volatile("ds_read_b32 %0, %1\n\ts_waitcnt lgkmcnt(0)" : "=v"(v) : "v"(addr) : "memory");
  return __builtin_amdgcn_readfirstlane(v);
}
DI unsigned lds_min2(unsigned addr) {
  u32x2 v;
  asm volatile("ds_read_b64 %0, %1\n\ts_waitcnt lgkmcnt(0)" : "=v"(v) : "v"(addr) : "memory");
  return __builtin_amdgcn_readfirstlane(min(v.x, v.y));
}
DI unsigned lds_min4(unsigned addr) {
  u32x4 v;
  asm volatile("ds_read_b128 %0, %1\n\ts_waitcnt lgkmcnt(0)" : "=v"(v) : "v"(addr) : "memory");
  return __builtin_amdgcn_readfirstlane(min(min(v.x, v.y), min(v.z, v.w)));
}
DI void x_to_lds(const f32x4 (&xa)[8], unsigned tb, unsigned kq) {
#pragma unroll
  for (int i = 0; i < 8; i++) {
    u32x2 o = {pk(xa[i][0], xa[i][1]), pk(xa[i][2], xa[i][3])};
    *(LAS u32x2*)(tb + (((unsigned)(i >> 1)) ^ kq) * 1024u + (i & 1) * 128) = o;
  }
}

enum : int { O_, P_, CW_, G_, WS_, WA_, CO_, T_, R_, END_ };
struct Op { int k, a; };
#define SCH 9
#define NGRP 1
#define TPER 148
#define TGRP 4
#define TTOL 16
constexpr int MAXOPS = 64;
struct Sched { Op op[MAXOPS]; int n; };
constexpr Op PLAN[2][MAXOPS] = {
#define TG(r) {G_,4*(r)-7},{O_,r},{CO_,r},{G_,4*(r)-5},{P_,r},{CW_,r}
  {{O_,0},{WS_,0},{P_,0},{O_,1},{P_,1},{CW_,0},{CW_,1},{WA_,0},TG(2),TG(3),TG(4),TG(5),TG(6),TG(7),{END_,0}},
  {{O_,0},{WS_,0},{P_,0},{O_,1},{P_,1},{CW_,0},{CW_,1},{WA_,0},TG(2),TG(3),TG(4),TG(5),TG(6),TG(7),{END_,0}},
};
constexpr Sched make_sched(int g) {
  Sched s{};
  while (PLAN[g][s.n].k != END_) { s.op[s.n] = PLAN[g][s.n]; s.n++; }
  return s;
}
constexpr int NSET = 6;
constexpr bool is_ld(int k) { return k == O_ || k == P_; }
template <int G> struct SC {
  static constexpr Sched S = make_sched(G);
  static constexpr int nvm(int i) { return is_ld(S.op[i].k) ? 8 : 0; }
  static constexpr int find_op(int k, int a) { for (int i = 0; i < S.n; i++) if (S.op[i].k == k && S.op[i].a == a) return i; return -1; }
  static constexpr int younger(int t, int i) { int n = 0; for (int j = t + 1; j < i; j++) n += nvm(j); return n > 63 ? 63 : n; }
  static constexpr int set_of(int op) {
    int owner[NSET] = {-1, -1, -1, -1, -1, -1};
    for (int i = 0; i < S.n; i++) {
      if (is_ld(S.op[i].k)) {
        int s = -1;
        for (int q = 0; q < NSET; q++) if (owner[q] < 0) { s = q; break; }
        if (s < 0) return -1;
        owner[s] = S.op[i].a;
        if (i == op) return s;
      } else if (S.op[i].k == CW_) {
        for (int q = 0; q < NSET; q++) if (owner[q] == S.op[i].a) owner[q] = -1;
      }
    }
    return -1;
  }
  static constexpr bool can_reach(int n, int i) { return n <= 0 || find_op(CW_, n / 4) < i; }
  static constexpr bool ok() {
    for (int r = 0; r < 8; r++) {
      const int o = find_op(O_, r), p = find_op(P_, r), c = find_op(CW_, r);
      if (o < 0 || p < 0 || c < o || c < p || set_of(o) < 0 || set_of(p) < 0) return false;
      if (r > 0 && c < find_op(CW_, r - 1)) return false;
      if (!can_reach(4 * r + 3 - NWS + 1, c)) return false;
      if (4 * r + 3 >= NWS - 2 && c < find_op(WS_, 0)) return false;
    }
    for (int i = 0; i < S.n; i++) if (S.op[i].k == G_ && !can_reach(S.op[i].a, i)) return false;
    return true;
  }
};
static_assert(SC<0>::ok() && SC<1>::ok(), "loader schedule");

struct Loader {
  f32x4 wb[NSET][8];
  __amdgpu_buffer_rsrc_t rW;
  int vW[8];
  unsigned sW, own, ldsW, l, doneAll, tofs;
  unsigned fl, pDone, pDoneOwn, pFullW;
  unsigned tr0;
};
template <int G, int I> DI void run_op(Loader& L) {
  using C = SC<G>;
  constexpr int k = C::S.op[I].k, a = C::S.op[I].a;
  if constexpr (k == WS_) {
    while (lds_get(L.fl + F_ST * 4) < 4u) __builtin_amdgcn_s_sleep(1);
  } else if constexpr (k == WA_) {
    while (lds_get(L.fl + F_RDY * 4) == 0u) {}
  } else if constexpr (is_ld(k)) {
    constexpr int s = C::set_of(I);
    const unsigned so = (unsigned)a * (128u * N * 4u) + L.sW + ((k == O_) ? L.own : (256u - L.own));
    asm volatile(
        "buffer_load_dwordx4 %0, %8, %16, %17 offen\n\t"
        "buffer_load_dwordx4 %1, %9, %16, %17 offen\n\t"
        "buffer_load_dwordx4 %2, %10, %16, %17 offen\n\t"
        "buffer_load_dwordx4 %3, %11, %16, %17 offen\n\t"
        "buffer_load_dwordx4 %4, %12, %16, %17 offen\n\t"
        "buffer_load_dwordx4 %5, %13, %16, %17 offen\n\t"
        "buffer_load_dwordx4 %6, %14, %16, %17 offen\n\t"
        "buffer_load_dwordx4 %7, %15, %16, %17 offen"
        : "=&v"(L.wb[s][0]), "=&v"(L.wb[s][1]), "=&v"(L.wb[s][2]), "=&v"(L.wb[s][3]),
          "=&v"(L.wb[s][4]), "=&v"(L.wb[s][5]), "=&v"(L.wb[s][6]), "=&v"(L.wb[s][7])
        : "v"(L.vW[0]), "v"(L.vW[1]), "v"(L.vW[2]), "v"(L.vW[3]), "v"(L.vW[4]), "v"(L.vW[5]), "v"(L.vW[6]), "v"(L.vW[7]),
          "s"(L.rW), "s"(so)
        : "memory");
  } else if constexpr (k == R_) {
    asm volatile("s_mov_b64 exec, 1\n\tds_add_u32 %0, %1\n\ts_mov_b64 exec, -1" ::"v"(L.fl + F_RV * 4), "v"(1u) : "memory");
    while (lds_get(L.fl + F_RV * 4) < 4u * a) {}
  } else if constexpr (k == T_) {
    const unsigned t = (unsigned)now(), ph = (t - L.tofs) % (unsigned)TPER;
    if (ph >= (unsigned)TTOL) {
      const unsigned tgt = t + (TPER - ph);
      while ((int)((unsigned)now() - tgt) < 0) __builtin_amdgcn_s_sleep(1);
    }
  } else if constexpr (k == CW_ || k == CO_) {
    constexpr bool first = k == CO_ || C::find_op(CO_, a) < 0;
    constexpr int io = C::find_op(O_, a), ip = C::find_op(P_, a);
    constexpr int so = C::set_of(io), sp = C::set_of(ip);
    const unsigned t = 4u * a + L.l;
    if constexpr (first && 4 * a + 3 >= NWS) {
      const int need = (int)t - NWS + 1;
      while ((int)L.doneAll < need) { L.doneAll = lds_min4(L.pDone); if ((int)L.doneAll < need) __builtin_amdgcn_s_sleep(1); }
    }
    unsigned slot = (4u * a) % NWS + L.l;
    if (slot >= (unsigned)NWS) slot -= NWS;
    const unsigned base = L.ldsW + slot * 8192u;
    auto conv = [&](f32x4(&w)[8], unsigned dst) {
#pragma unroll
      for (int c = 0; c < 4; c++) {
        u32x4 bf;
        bf.x = pk(w[0][c], w[1][c]); bf.y = pk(w[2][c], w[3][c]);
        bf.z = pk(w[4][c], w[5][c]); bf.w = pk(w[6][c], w[7][c]);
        *(LAS u32x4*)(dst + c * 1024) = bf;
      }
    };
    if constexpr (first) {
    asm volatile("s_waitcnt vmcnt(%8)"
                 : "+v"(L.wb[so][0]), "+v"(L.wb[so][1]), "+v"(L.wb[so][2]), "+v"(L.wb[so][3]),
                   "+v"(L.wb[so][4]), "+v"(L.wb[so][5]), "+v"(L.wb[so][6]), "+v"(L.wb[so][7])
                 : "i"(C::younger(io, I)) : "memory");
    conv(L.wb[so], base + L.own * 16u);
    }
    if constexpr (k == CW_) {
    asm volatile("s_waitcnt vmcnt(%8)"
                 : "+v"(L.wb[sp][0]), "+v"(L.wb[sp][1]), "+v"(L.wb[sp][2]), "+v"(L.wb[sp][3]),
                   "+v"(L.wb[sp][4]), "+v"(L.wb[sp][5]), "+v"(L.wb[sp][6]), "+v"(L.wb[sp][7])
                 : "i"(C::younger(ip, I)) : "memory");
    conv(L.wb[sp], base + (4096u - L.own * 16u));
    lds_put(L.pFullW, t + 1u);
    }
  } else {
    while (lds_get(L.pDoneOwn) < (unsigned)a) __builtin_amdgcn_s_sleep(1);
  }
}
template <int G, int... I> DI void run_all(Loader& L, std::integer_sequence<int, I...>) { (run_op<G, I>(L), ...); }
template <int G> DI void run_grp(Loader& L) { run_all<G>(L, std::make_integer_sequence<int, SC<G>::S.n>{}); }

#define STAMP(i) do {} while (0)

__global__ void __launch_bounds__(512, 2) fused_gemm_a2a(Params p) {
  extern __shared__ __attribute__((aligned(16))) unsigned char smem[];
  const int tid = threadIdx.x, lane = tid & 63;
  const int w = __builtin_amdgcn_readfirstlane(tid >> 6);
  const int b = blockIdx.x;
  STAMP(0);
  const int xcd = b & 7, j = b >> 3;
  const int kq = xcd >> 1, mb2 = j & 1;
  const int nb = (xcd & 1) * 16 + (j >> 1);
  const int rank = p.rank;
  unsigned char* ws = p.self;
  GAS unsigned* tmo = (GAS unsigned*)(ws + WS_TMO);
  const unsigned lds0 = (unsigned)(size_t)(LAS unsigned char*)smem;
  const unsigned fl = lds0 + L_F;
  const int c = w & 3;
  const unsigned dst = (unsigned)(nb >> 2);
  const int idx = ((nb & 3) * 2 + mb2) * 4 + kq;
  const __amdgpu_buffer_rsrc_t rX = __builtin_amdgcn_make_buffer_rsrc((void*)p.x, 0, M * K * 4, 0x00020000);
  const int vX = (lane >> 3) * (K * 4) + (lane & 7) * 16;
  const unsigned vT = ((lane & 7) >> 1) * 256 + (lane >> 3) * 16 + (lane & 1) * 8;
  auto ldx = [&](f32x4(&xa)[8], int cc, int step) {
    const int so = (mb2 * 256 + cc * 64) * (K * 4) + (kq * 1024 + step * 32) * 4;
#pragma unroll
    for (int i = 0; i < 8; i++)
      xa[i] = __builtin_bit_cast(f32x4, __builtin_amdgcn_raw_buffer_load_b128(rX, vX, so + i * 8 * (K * 4), 0));
  };

  GAS unsigned* afl = (GAS unsigned*)(ws + WS_AFLAG) + (mb2 * 4 + kq) * NSTEP * 32;
  if (w < 4) {
    unsigned call;
    {
      f32x4 xa[8];
      ldx(xa, c, nb);
      GAS unsigned* cnt = (GAS unsigned*)(ws + WS_CNT) + b * 4 + c;
      const unsigned cv = __hip_atomic_load(cnt, RLX_AGENT);
      unsigned char* pub = ws + WS_AT + (((mb2 * 4 + kq) * NSTEP + nb) * 4 + c) * 4096 + lane * 16;
      const unsigned tw0 = __hip_atomic_load((GAS unsigned*)(pub - lane * 16), RLX_AGENT);
      const unsigned tw1 = __hip_atomic_load(afl + nb * 32, RLX_AGENT);
      if (tid < 32) *(LAS unsigned*)(fl + tid * 4) = 0u;
      asm volatile("s_waitcnt lgkmcnt(0)" ::: "memory");
      SB();
      STAMP(12);
      __builtin_amdgcn_s_barrier();
      CFENCE();
      STAMP(13);
      const unsigned sc = lds0 + L_SC + c * 4096;
      x_to_lds(xa, sc + vT, kq);
      call = __builtin_amdgcn_readfirstlane(cv) + 1u;
      asm volatile("" ::"v"(tw0), "v"(tw1));
      STAMP(15);
      u32x4 fr[4];
#pragma unroll
      for (int f = 0; f < 4; f++) fr[f] = *(LAS u32x4*)(sc + f * 1024 + lane * 16);
#pragma unroll
      for (int f = 0; f < 4; f++)
        asm volatile("global_store_dwordx4 %0, %1, off offset:%2 sc0 sc1\n\ts_nop 1" ::"v"(pub), "v"(fr[f]), "i"(f * 1024) : "memory");
      if (lane == 0) __hip_atomic_fetch_add((LAS unsigned*)(fl + F_ST * 4), 1u, __ATOMIC_RELAXED, __HIP_MEMORY_SCOPE_WORKGROUP);
      lds_put(fl + F_CALL * 4, call);
      STAMP(2);
      asm volatile("s_waitcnt vmcnt(0)" ::: "memory");
      if (lane == 0) {
        const unsigned old = __hip_atomic_fetch_add((LAS unsigned*)(fl + F_DR * 4), 1u, __ATOMIC_RELAXED, __HIP_MEMORY_SCOPE_WORKGROUP);
        if (old == 3u) __hip_atomic_store(afl + nb * 32, call, RLX_SYS);
        __hip_atomic_store(cnt, call, RLX_AGENT);
      }
    }
    STAMP(1);
    if (c == 0) {
      while (lds_get(fl + F_DR * 4) < 4u) {}
      const u64 t0 = now();
      while (true) {
        const unsigned v = __hip_atomic_load(afl + (lane & 31) * 32, RLX_SYS);
        if (__all(v >= call)) break;
        if (give_up(tmo, call, t0)) break;
      }
      asm volatile("s_waitcnt vmcnt(0)" ::: "memory");
      lds_put(fl + F_RDY * 4, 1u);
    }
    STAMP(14);
    f32x4 acc[4][8];
#pragma unroll
    for (int m = 0; m < 4; m++)
#pragma unroll
      for (int n = 0; n < 8; n++) acc[m][n] = f32x4{0, 0, 0, 0};
#define SPIN_T0
#define SPIN_T1
    const unsigned wBase = lds0 + L_W + lane * 16;
    unsigned pfw = fl + F_FULLW * 4, pdone = fl + (F_DONE + c) * 4, vdone = 0;
    asm volatile("" : "+v"(pfw), "+v"(pdone), "+v"(vdone));
    const __amdgpu_buffer_rsrc_t rA = __builtin_amdgcn_make_buffer_rsrc(
        (void*)(ws + WS_AT + ((mb2 * 4 + kq) * NSTEP * 4 + c) * 4096), 0, (NSTEP - 1) * 16384 + 4096, 0x00020000);
    int vA[4];
#pragma unroll
    for (int m = 0; m < 4; m++) { vA[m] = lane * 16 + m * 1024; asm volatile("" : "+v"(vA[m])); }
    u32x4 A[4][4], B[4];
    unsigned fw;
    if (c != 0) while (lds_get(fl + F_RDY * 4) == 0u) {}
#pragma unroll
    for (int d = 0; d < 3; d++)
#pragma unroll
      for (int m = 0; m < 4; m++) { A[d][m] = __builtin_amdgcn_raw_buffer_load_b128(rA, vA[m], d * 16384, AUX_SYS); SB(); }
    SB();
    while (lds_get(pfw) < 1u) {}
    __builtin_amdgcn_s_setprio(CPRIO);
    STAMP(3);
    B[0] = *(LAS u32x4*)(wBase);
    B[1] = *(LAS u32x4*)(wBase + 1024);
    B[2] = *(LAS u32x4*)(wBase + 2048);
    CFENCE(); fw = *(LAS unsigned*)(pfw + 4); CFENCE();
    unsigned woff = 0, wc = wBase;
    auto mf = [&](int m, int n, u32x4(&Ac)[4]) {
      asm volatile("v_mfma_f32_16x16x32_bf16 %0, %1, %2, %0" : "+v"(acc[m][n]) : "v"(B[n & 3]), "v"(Ac[m]));
    };
    auto step = [&](unsigned s, int q) {
      unsigned wn = 0, sf = 0;
#pragma unroll
      for (int n = 0; n < 8; n++) {
        mf(0, n, A[q]); SB();
        if (n < 5) B[(n + 3) & 3] = *(LAS u32x4*)(wc + (n + 3) * 1024);
        else B[(n + 3) & 3] = *(LAS u32x4*)(wn + (n - 5) * 1024);
        SB(); mf(1, n, A[q]); SB();
        if (n == 1) { woff = (woff == (NWS - 1) * 8192u) ? 0u : woff + 8192u; wn = wBase + woff; asm volatile("" : "+v"(wn)); }
        if (n == 2) sf = __builtin_amdgcn_readfirstlane(fw);
        if (n == 4) lds_put(pdone, vdone);
        if (n == 7) { CFENCE(); fw = *(LAS unsigned*)(pfw + ((q + 2) & 3) * 4); CFENCE(); }
        SB(); mf(2, n, A[q]); SB();
        if ((n & 1) == 0) A[(q + 3) & 3][n >> 1] = __builtin_amdgcn_raw_buffer_load_b128(rA, vA[n >> 1], (s + 3u) * 16384u, AUX_SYS);
        if (n == 1) { vdone += 1; asm volatile("" : "+v"(vdone)); }
        if (n == 3) {
          const unsigned nd = s + 2u;
          if (__builtin_expect(sf < nd && nd <= (unsigned)NSTEP, 0)) {
            SPIN_T0;
            while (lds_get(pfw + ((q + 1) & 3) * 4) < nd) {}
            SPIN_T1;
          }
          CFENCE();
        }
        SB(); mf(3, n, A[q]); SB();
      }
      wc = wn;
    };
    asm volatile("s_nop 7");
    for (unsigned s = 0; s < (unsigned)NSTEP; s += 4) {
      step(s, 0);
      step(s + 1, 1);
      step(s + 2, 2);
      step(s + 3, 3);
    }
    asm volatile("s_nop 15\n\ts_nop 15");
    SB();
    __builtin_amdgcn_s_setprio(0);
    STAMP(4);
    const unsigned par = call & 1u;
    const __amdgpu_buffer_rsrc_t rE = __builtin_amdgcn_make_buffer_rsrc((void*)(ws + WS_X), 0, X_BYTES, 0x00020000);
#pragma unroll
    for (int m = 1; m < 4; m++) {
      const int so = (((b ^ (m << 1)) * 3 + (m - 1)) * 4 + c) * 4096;
#pragma unroll
      for (int t = 0; t < 4; t++) {
        u32x4 o;
        o.x = pk(acc[m][2 * t][0], acc[m][2 * t][1]); o.y = pk(acc[m][2 * t][2], acc[m][2 * t][3]);
        o.z = pk(acc[m][2 * t + 1][0], acc[m][2 * t + 1][1]); o.w = pk(acc[m][2 * t + 1][2], acc[m][2 * t + 1][3]);
        __builtin_amdgcn_raw_buffer_store_b128(o, rE, lane * 16, so + t * 1024, AUX_SYS);
      }
    }
    asm volatile("s_waitcnt vmcnt(0)" ::: "memory");
    GAS unsigned* xfl = (GAS unsigned*)(ws + WS_XFLAG);
    if (lane < 3) __hip_atomic_store(xfl + (((b ^ ((lane + 1) << 1)) * 4 + c) * 3 + lane) * 32, call, RLX_SYS);
    STAMP(5);
    {
      const u64 t0 = now();
      while (true) {
        const unsigned v = __hip_atomic_load(xfl + ((b * 4 + c) * 3 + min(lane, 2)) * 32, RLX_SYS);
        if (__all(v >= call)) break;
        if (give_up(tmo, call, t0)) break;
      }
      __builtin_amdgcn_fence(__ATOMIC_ACQUIRE, "wavefront");
    }
    u32x4 in[3][4];
#pragma unroll
    for (int m = 0; m < 3; m++)
#pragma unroll
      for (int t = 0; t < 4; t++)
        in[m][t] = __builtin_amdgcn_raw_buffer_load_b128(rE, lane * 16, ((b * 3 + m) * 4 + c) * 4096 + t * 1024, AUX_SYS);
    float am = 0.f;
#pragma unroll
    for (int t = 0; t < 4; t++) {
#pragma unroll
      for (int m = 0; m < 3; m++) {
#pragma unroll
        for (int e = 0; e < 4; e++) {
          const unsigned v = in[m][t][e];
          acc[0][2 * t + (e >> 1)][(e & 1) * 2] += __builtin_bit_cast(float, v << 16);
          acc[0][2 * t + (e >> 1)][(e & 1) * 2 + 1] += __builtin_bit_cast(float, v & 0xffff0000u);
        }
      }
#pragma unroll
      for (int e = 0; e < 8; e++) am = fmaxf(am, fabsf(acc[0][2 * t + (e >> 2)][e & 3]));
    }
#pragma unroll
    for (int o = 32; o > 0; o >>= 1) am = fmaxf(am, __shfl_xor(am, o));
    if (lane == 0) {
      __hip_atomic_fetch_max((LAS unsigned*)(fl + F_MAX * 4), __builtin_bit_cast(unsigned, am), __ATOMIC_RELAXED,
                             __HIP_MEMORY_SCOPE_WORKGROUP);
      CFENCE();
      const unsigned old = __hip_atomic_fetch_add((LAS unsigned*)(fl + F_CNT * 4), 1u, __ATOMIC_RELAXED,
                                                  __HIP_MEMORY_SCOPE_WORKGROUP);
      CFENCE();
      if (old == 3u) {
        const unsigned mx = __hip_atomic_load((LAS unsigned*)(fl + F_MAX * 4), __ATOMIC_RELAXED, __HIP_MEMORY_SCOPE_WORKGROUP);
        __hip_atomic_store((GAS u64*)(ws + WS_LOC) + (par * 256 + b) * 16, ((u64)call << 32) | (u64)mx, RLX_SYS);
      }
    }
    STAMP(6);
    while (lds_get(fl + F_GOK * 4) == 0u) __builtin_amdgcn_s_sleep(1);
    const unsigned gmax = lds_get(fl + F_GMAX * 4);
    STAMP(7);
    const bool failed = (__hip_atomic_load(tmo, RLX_AGENT) == call);
    const float amax = __builtin_bit_cast(float, gmax);
    const float rs = amax > 0.f ? 127.0f / amax : 0.f;
    if (dst < (unsigned)NR) {
      unsigned char* ob = p.out[dst];
      const int row = rank * M + mb2 * 256 + c * 64 + kq * 16 + (lane & 15);
#pragma unroll
      for (int hf = 0; hf < 2; hf++) {
        u32x4 o;
#pragma unroll
        for (int rg = 0; rg < 4; rg++) {
          const unsigned t0 = __builtin_bit_cast(unsigned, __builtin_fmaf(acc[0][hf * 4 + 0][rg], rs, 12582912.0f));
          const unsigned t1 = __builtin_bit_cast(unsigned, __builtin_fmaf(acc[0][hf * 4 + 1][rg], rs, 12582912.0f));
          const unsigned t2 = __builtin_bit_cast(unsigned, __builtin_fmaf(acc[0][hf * 4 + 2][rg], rs, 12582912.0f));
          const unsigned t3 = __builtin_bit_cast(unsigned, __builtin_fmaf(acc[0][hf * 4 + 3][rg], rs, 12582912.0f));
          const unsigned lo = __builtin_amdgcn_perm(t1, t0, 0x0c0c0400u);
          const unsigned hi = __builtin_amdgcn_perm(t3, t2, 0x04000c0cu);
          o[rg] = failed ? 0x80808080u : (lo | hi);
        }
        unsigned char* ptr = ob + (size_t)row * 512 + (nb & 3) * 128 + hf * 64 + 16 * (lane >> 4);
        asm volatile("global_store_dwordx4 %0, %1, off sc0 sc1\n\ts_nop 1" ::"v"(ptr), "v"(o) : "memory");
      }
      asm volatile("s_waitcnt vmcnt(0)" ::: "memory");
      STAMP(8);
      if (lane == 0 && dst != (unsigned)rank)
        __hip_atomic_store((GAS unsigned*)(p.ws[dst] + WS_DONE) + ((idx * 4 + c) * 8 + rank) * 32, call, RLX_SYS);
    }
    STAMP(9);
  } else {
    Loader L;
    L.l = c;
        L.rW = __builtin_amdgcn_make_buffer_rsrc((void*)p.w, 0, K * N * 4, 0x00020000);
#pragma unroll
    for (int i = 0; i < 8; i++) L.vW[i] = ((lane >> 4) * 8 + i) * (N * 4) + (lane & 15) * 16;
    L.sW = (unsigned)(kq * 1024 + c * 32) * (N * 4u) + (unsigned)nb * 512u;
    L.own = (unsigned)mb2 * 256u;
    L.ldsW = lds0 + L_W + lane * 16;
    L.fl = fl; L.pDone = fl + F_DONE * 4; L.pDoneOwn = fl + (F_DONE + c) * 4; L.pFullW = fl + (F_FULLW + c) * 4;
    asm volatile("" : "+v"(L.fl), "+v"(L.pDone), "+v"(L.pDoneOwn), "+v"(L.pFullW));
    L.doneAll = 0;
    L.tofs = (unsigned)((j >> 1) % TGRP) * (TPER / TGRP);
    L.tr0 = 0;
    __builtin_amdgcn_s_setprio(APRIO);
    __builtin_amdgcn_s_barrier();
    CFENCE();
    {
#define GSEL ((j >> 1) % NGRP)
      const int g = GSEL;
      if (NGRP == 1 || g == 0) run_grp<0>(L);
      else run_grp<1>(L);
    }
    const unsigned call = lds_get(fl + F_CALL * 4);
    STAMP(10);
    const unsigned par = call & 1u;
    const u64 t0 = now();
    if (c == 0 && j == 0) {
      GAS u64* g = (GAS u64*)(ws + WS_LOC) + (par * 256 + (lane & 31) * 8 + xcd) * 16;
      unsigned mx = 0;
      while (true) {
        const u64 v = __hip_atomic_load(g, RLX_SYS);
        mx = (unsigned)v;
        if (__all((unsigned)(v >> 32) == call)) break;
        if (give_up(tmo, call, t0)) break;
      }
#pragma unroll
      for (int o = 16; o > 0; o >>= 1) mx = max(mx, (unsigned)__shfl_xor((int)mx, o));
      const int srcid = rank;
      __hip_atomic_store((GAS u64*)(p.ws[lane >> 3] + WS_MBOX) + (((par * 8 + (lane & 7)) * 8 + srcid) * 8 + xcd) * 16,
                         ((u64)call << 32) | (u64)mx, RLX_SYS);
    }
    if (c == 1) {
      GAS u64* g = (GAS u64*)(ws + WS_MBOX) + ((par * 8 + xcd) * 64 + lane) * 16;
      unsigned mx = 0;
      while (lds_get(fl + F_CNT * 4) < 4u) __builtin_amdgcn_s_sleep(2);
      while (true) {
        const u64 v = __hip_atomic_load(g, RLX_SYS);
        mx = (unsigned)v;
        if (__all((unsigned)(v >> 32) == call)) break;
        if (give_up(tmo, call, t0)) break;
      }
#pragma unroll
      for (int o = 32; o > 0; o >>= 1) mx = max(mx, (unsigned)__shfl_xor((int)mx, o));
      lds_put(fl + F_GMAX * 4, mx);
      lds_put(fl + F_GOK * 4, 1u);
    }
    if (dst == (unsigned)rank) {
      while (lds_get(fl + F_GOK * 4) == 0u) __builtin_amdgcn_s_sleep(2);
      GAS unsigned* d = (GAS unsigned*)(ws + WS_DONE) + ((idx * 4 + c) * 8 + (lane & 7)) * 32;
      bool bad = false;
      while (true) {
        const unsigned v = __hip_atomic_load(d, RLX_SYS);
        if (__all(v >= call || (lane & 7) == rank)) break;
        if (give_up(tmo, call, t0)) { bad = true; break; }
        __builtin_amdgcn_s_sleep(1);
      }
      if (bad && lane == 0) p.out[rank][(size_t)(rank * M + mb2 * 256 + c * 64 + kq * 16) * 512 + (nb & 3) * 128] = 0x80;
    }
    STAMP(11);
  }
}

extern "C" void kernel_launch(int rank, int nranks, ncclComm_t comm, hipStream_t stream,
                              void* const* d_in, const int* in_sizes, int n_in,
                              void* d_out, int out_size, const int* in_dtypes, int out_dtype,
                              void* const* d_out_all, void* const* d_ws_all, size_t ws_bytes) {
  if (nranks != NR || n_in < 2 || ws_bytes < WS_NEED || in_sizes[0] != M * K || in_sizes[1] != K * N) {
    fprintf(stderr, "unexpected problem shape\n");
    return;
  }
  static bool init = false;
  if (!init) {
    HIP_CHECK(hipFuncSetAttribute((const void*)fused_gemm_a2a, hipFuncAttributeMaxDynamicSharedMemorySize, LDS_BYTES));
    init = true;
  }
  Params p;
  p.x = (const float*)d_in[0];
  p.w = (const float*)d_in[1];
  for (int i = 0; i < NR; i++) { p.out[i] = (unsigned char*)d_out_all[i]; p.ws[i] = (unsigned char*)d_ws_all[i]; }
  p.rank = rank;
  p.self = (unsigned char*)d_ws_all[rank];
  p.pad = 0;
  hipLaunchKernelGGL(fused_gemm_a2a, dim3(NWG), dim3(512), LDS_BYTES, stream, p);
  HIP_CHECK(hipGetLastError());
}
```

```cpp
#include <hip/hip_runtime.h>
#include <rccl/rccl.h>
#include <cstdio>
#include <utility>

#define HIP_CHECK(expr) do { hipError_t _e = (expr); if (_e != hipSuccess) { \
    fprintf(stderr, "HIP error %s at %s:%d\n", hipGetErrorString(_e), __FILE__, __LINE__); \
    return; } } while (0)

typedef unsigned u32x4 __attribute__((ext_vector_type(4)));
typedef unsigned u32x2 __attribute__((ext_vector_type(2)));
typedef float f32x4 __attribute__((ext_vector_type(4)));
typedef float f32x2 __attribute__((ext_vector_type(2)));
typedef __bf16 bf16x8 __attribute__((ext_vector_type(8)));
typedef __bf16 bf16x2 __attribute__((ext_vector_type(2)));
typedef unsigned long long u64;
#define GAS __attribute__((address_space(1)))
#define LAS __attribute__((address_space(3)))
#define DI __device__ __forceinline__
#pragma clang diagnostic ignored "-Wint-to-pointer-cast"

constexpr int NR = 8;
constexpr int M = 512, N = 4096, K = 4096;
constexpr int NWG = 256;
constexpr int NSTEP = 32;
constexpr int WS_TMO = 0;
constexpr int WS_CNT = 4096;
constexpr int WS_LOC = 8192;
constexpr int WS_AFLAG = 128 * 1024;
constexpr int WS_DONE = 256 * 1024;
constexpr int WS_MBOX = 384 * 1024;
constexpr int WS_XFLAG = 576 * 1024;
constexpr int WS_AT = 1 << 20;
constexpr int WS_X = 5 << 20;
constexpr int X_BYTES = NWG * 3 * 4 * 4096;
constexpr int WS_EF = 17 << 20;
constexpr int WS_E = 18 << 20;
constexpr size_t WS_NEED = (size_t)WS_E + 2 * 8 * 128 * 2048;
constexpr u64 DEADLINE = 40000000ull;
#define NWS 19
constexpr int L_W = 0;
constexpr int L_F = L_W + NWS * 8192;
constexpr int L_SC = L_F - 16384;
constexpr int LDS_BYTES = L_F + 256;
static_assert(LDS_BYTES <= 160 * 1024, "LDS");
constexpr int F_FULLW = 4, F_DONE = 8, F_MAX = 16, F_CNT = 17, F_CALL = 18, F_GMAX = 24, F_GOK = 25, F_ST = 26, F_DR = 27, F_RV = 29, F_FS = 30, F_RDY = 12, F_GS = 31, F_LMAX = 19, F_LCNT = 20;
constexpr int AUX_SYS = 17;
#define PSLEEP 2
#define GSLEEP 1
#define NDEF 0
#define XFLAG_AT 11
static_assert(NDEF % 4 == 0 && NDEF <= NWS - 3, "deferred steps stay in the ring");
#define EGAP 5u
#define ECAP 128
#define ESLEEP 12
#define CPRIO 3
#define APRIO 0

struct Params {
  const float* x;
  const float* w;
  unsigned char* self;
  int rank;
  int pad;
  unsigned char* out[NR];
  unsigned char* ws[NR];
};

DI unsigned pk(float a, float b) {
  f32x2 v = {a, b};
  bf16x2 r = __builtin_convertvector(v, bf16x2);
  return __builtin_bit_cast(unsigned, r);
}
DI u64 now() { return __builtin_amdgcn_s_memrealtime(); }
#define RLX_AGENT __ATOMIC_RELAXED, __HIP_MEMORY_SCOPE_AGENT
#define RLX_SYS __ATOMIC_RELAXED, __HIP_MEMORY_SCOPE_SYSTEM
#define CFENCE() asm volatile("" ::: "memory")
#define SB() __builtin_amdgcn_sched_barrier(0)

DI bool give_up(GAS unsigned* tmo, unsigned call, u64 t0) {
  bool bad = (__hip_atomic_load(tmo, RLX_AGENT) == call) || (now() - t0 > DEADLINE);
  if (bad) __hip_atomic_store(tmo, call, RLX_AGENT);
  return bad;
}
DI void lds_put(unsigned addr, unsigned v) {
  asm volatile("ds_write_b32 %0, %1" ::"v"(addr), "v"(v) : "memory");
}
DI void lds_put2(unsigned addr, unsigned v) {
  u32x2 t = {v, v};
  asm volatile("ds_write_b64 %0, %1" ::"v"(addr), "v"(t) : "memory");
}
DI unsigned lds_get(unsigned addr) {
  unsigned v;
  asm volatile("ds_read_b32 %0, %1\n\ts_waitcnt lgkmcnt(0)" : "=v"(v) : "v"(addr) : "memory");
  return __builtin_amdgcn_readfirstlane(v);
}
DI unsigned lds_min2(unsigned addr) {
  u32x2 v;
  asm volatile("ds_read_b64 %0, %1\n\ts_waitcnt lgkmcnt(0)" : "=v"(v) : "v"(addr) : "memory");
  return __builtin_amdgcn_readfirstlane(min(v.x, v.y));
}
DI unsigned lds_min4(unsigned addr) {
  u32x4 v;
  asm volatile("ds_read_b128 %0, %1\n\ts_waitcnt lgkmcnt(0)" : "=v"(v) : "v"(addr) : "memory");
  return __builtin_amdgcn_readfirstlane(min(min(v.x, v.y), min(v.z, v.w)));
}
DI void wait8(const GAS unsigned* f8, unsigned call, GAS unsigned* tmo) {
  const u64 t0 = now();
  while (true) {
    unsigned a0, a1, a2, a3, a4, a5, a6, a7;
    asm volatile(
        "s_load_dword %0, %8, 0x0 glc\n\ts_load_dword %1, %8, 0x80 glc\n\ts_load_dword %2, %8, 0x100 glc\n\t"
        "s_load_dword %3, %8, 0x180 glc\n\ts_load_dword %4, %8, 0x200 glc\n\ts_load_dword %5, %8, 0x280 glc\n\t"
        "s_load_dword %6, %8, 0x300 glc\n\ts_load_dword %7, %8, 0x380 glc\n\ts_waitcnt lgkmcnt(0)"
        : "=&s"(a0), "=&s"(a1), "=&s"(a2), "=&s"(a3), "=&s"(a4), "=&s"(a5), "=&s"(a6), "=&s"(a7)
        : "s"(f8) : "memory");
    if (min(min(min(a0, a1), min(a2, a3)), min(min(a4, a5), min(a6, a7))) >= call) break;
    if (now() - t0 > DEADLINE) {
      asm volatile("global_store_dword %0, %1, off sc1\n\ts_waitcnt vmcnt(0)" ::"v"(tmo), "v"(call) : "memory");
      break;
    }
    __builtin_amdgcn_s_sleep(PSLEEP);
  }
}
DI void x_to_lds(const f32x4 (&xa)[8], unsigned tb, unsigned kq) {
#pragma unroll
  for (int i = 0; i < 8; i++) {
    u32x2 o = {pk(xa[i][0], xa[i][1]), pk(xa[i][2], xa[i][3])};
    *(LAS u32x2*)(tb + (((unsigned)(i >> 1)) ^ kq) * 1024u + (i & 1) * 128) = o;
  }
}

enum : int { O_, P_, CW_, G_, WS_, WA_, CO_, T_, R_, WD_, Q_, END_ };
struct Op { int k, a; };
#define SCH 20
#define NGRP 1
#define WAUX ""
#define TPER 148
#define TGRP 4
#define TTOL 16
constexpr int MAXOPS = 64;
struct Sched { Op op[MAXOPS]; int n; };
#define TG(r) {G_,4*(r)-7},{O_,r},{CO_,r},{G_,4*(r)-5},{P_,r},{CW_,r}
#define TP(r) {T_,0},{O_,(r) + 1},{CW_,r},{CO_,(r) + 1},{P_,(r) + 1}
constexpr Op PLAN[1][MAXOPS] = {
  {{O_,0},{WS_,0},{P_,0},{WD_,0},{O_,1},{P_,1},{CW_,0},{CW_,1},{Q_,0},TG(2),TG(3),TG(4),TG(5),TG(6),TG(7),{END_,0}},
};
constexpr Sched make_sched(int g) {
  Sched s{};
  while (PLAN[g][s.n].k != END_) { s.op[s.n] = PLAN[g][s.n]; s.n++; }
  return s;
}
constexpr int NSET = 6;
constexpr bool is_ld(int k) { return k == O_ || k == P_; }
template <int G> struct SC {
  static constexpr Sched S = make_sched(G);
  static constexpr int nvm(int i) { return is_ld(S.op[i].k) ? 8 : 0; }
  static constexpr int find_op(int k, int a) { for (int i = 0; i < S.n; i++) if (S.op[i].k == k && S.op[i].a == a) return i; return -1; }
  static constexpr int younger(int t, int i) { int n = 0; for (int j = t + 1; j < i; j++) n += nvm(j); return n > 63 ? 63 : n; }
  static constexpr int set_of(int op) {
    int owner[NSET] = {-1, -1, -1, -1, -1, -1};
    for (int i = 0; i < S.n; i++) {
      if (is_ld(S.op[i].k)) {
        int s = -1;
        for (int q = 0; q < NSET; q++) if (owner[q] < 0) { s = q; break; }
        if (s < 0) return -1;
        owner[s] = S.op[i].a;
        if (i == op) return s;
      } else if (S.op[i].k == CW_) {
        for (int q = 0; q < NSET; q++) if (owner[q] == S.op[i].a) owner[q] = -1;
      }
    }
    return -1;
  }
  static constexpr bool can_reach(int n, int i) { return n <= 0 || find_op(CW_, n / 4) < i; }
  static constexpr bool ok() {
    for (int r = 0; r < 8; r++) {
      const int o = find_op(O_, r), p = find_op(P_, r), c = find_op(CW_, r);
      if (o < 0 || p < 0 || c < o || c < p || set_of(o) < 0 || set_of(p) < 0) return false;
      if (r > 0 && c < find_op(CW_, r - 1)) return false;
      if (!can_reach(4 * r + 3 - NWS + 1, c)) return false;
      if (4 * r + 3 >= NWS - 2 && c < find_op(WS_, 0)) return false;
    }
    for (int i = 0; i < S.n; i++) if (S.op[i].k == G_ && !can_reach(S.op[i].a, i)) return false;
    return true;
  }
};
static_assert(SC<0>::ok() && SC<0>::find_op(Q_, 0) >= 0, "loader schedule");

struct Loader {
  f32x4 wb[NSET][8];
  __amdgpu_buffer_rsrc_t rW;
  int vW[8];
  unsigned sW, own, ldsW, l, doneAll, tofs;
  unsigned fl, pDone, pDoneOwn, pFullW;
  const GAS unsigned* afl;
  GAS unsigned* tmo;
  unsigned tr0;
};
template <int G, int I> DI void run_op(Loader& L) {
  using C = SC<G>;
  constexpr int k = C::S.op[I].k, a = C::S.op[I].a;
  if constexpr (k == WS_) {
    while (lds_get(L.fl + F_ST * 4) < 4u) __builtin_amdgcn_s_sleep(1);
  } else if constexpr (k == WD_) {
    while (lds_get(L.fl + F_FS * 4) < 4u) __builtin_amdgcn_s_sleep(1);
  } else if constexpr (k == WA_) {
    while (lds_get(L.pDoneOwn) == 0u) __builtin_amdgcn_s_sleep(1);
  } else if constexpr (k == Q_) {
    if (L.l != 0u) {
      unsigned call;
      while ((call = lds_get(L.fl + F_CALL * 4)) == 0u) {}
      wait8(L.afl + L.l * 8u * 32u, call, L.tmo);
      lds_put(L.fl + (F_RDY + L.l) * 4, 1u);
    }
  } else if constexpr (is_ld(k)) {
    constexpr int s = C::set_of(I);
    const unsigned so = (unsigned)a * (128u * N * 4u) + L.sW + ((k == O_) ? L.own : (256u - L.own));
    asm volatile(
        "buffer_load_dwordx4 %0, %8, %16, %17 offen" WAUX "\n\t"
        "buffer_load_dwordx4 %1, %9, %16, %17 offen" WAUX "\n\t"
        "buffer_load_dwordx4 %2, %10, %16, %17 offen" WAUX "\n\t"
        "buffer_load_dwordx4 %3, %11, %16, %17 offen" WAUX "\n\t"
        "buffer_load_dwordx4 %4, %12, %16, %17 offen" WAUX "\n\t"
        "buffer_load_dwordx4 %5, %13, %16, %17 offen" WAUX "\n\t"
        "buffer_load_dwordx4 %6, %14, %16, %17 offen" WAUX "\n\t"
        "buffer_load_dwordx4 %7, %15, %16, %17 offen" WAUX
        : "=&v"(L.wb[s][0]), "=&v"(L.wb[s][1]), "=&v"(L.wb[s][2]), "=&v"(L.wb[s][3]),
          "=&v"(L.wb[s][4]), "=&v"(L.wb[s][5]), "=&v"(L.wb[s][6]), "=&v"(L.wb[s][7])
        : "v"(L.vW[0]), "v"(L.vW[1]), "v"(L.vW[2]), "v"(L.vW[3]), "v"(L.vW[4]), "v"(L.vW[5]), "v"(L.vW[6]), "v"(L.vW[7]),
          "s"(L.rW), "s"(so)
        : "memory");
  } else if constexpr (k == R_) {
    asm volatile("s_mov_b64 exec, 1\n\tds_add_u32 %0, %1\n\ts_mov_b64 exec, -1" ::"v"(L.fl + F_RV * 4), "v"(1u) : "memory");
    while (lds_get(L.fl + F_RV * 4) < 4u * a) {}
  } else if constexpr (k == T_) {
    const unsigned t = (unsigned)now(), ph = (t - L.tofs) % (unsigned)TPER;
    if (ph >= (unsigned)TTOL) {
      const unsigned tgt = t + (TPER - ph);
      while ((int)((unsigned)now() - tgt) < 0) __builtin_amdgcn_s_sleep(GSLEEP);
    }
  } else if constexpr (k == CW_ || k == CO_) {
    constexpr bool first = k == CO_ || C::find_op(CO_, a) < 0;
    constexpr int io = C::find_op(O_, a), ip = C::find_op(P_, a);
    constexpr int so = C::set_of(io), sp = C::set_of(ip);
    const unsigned t = 4u * a + L.l;
    if constexpr (first && 4 * a + 3 >= NWS) {
      const int need = (int)t - NWS + 1;
      while ((int)L.doneAll < need) { L.doneAll = lds_min4(L.pDone); if ((int)L.doneAll < need) __builtin_amdgcn_s_sleep(1); }
    }
    unsigned slot = (4u * a) % NWS + L.l;
    if (slot >= (unsigned)NWS) slot -= NWS;
    const unsigned base = L.ldsW + slot * 8192u;
    auto conv = [&](f32x4(&w)[8], unsigned dst) {
#pragma unroll
      for (int c = 0; c < 4; c++) {
        u32x4 bf;
        bf.x = pk(w[0][c], w[1][c]); bf.y = pk(w[2][c], w[3][c]);
        bf.z = pk(w[4][c], w[5][c]); bf.w = pk(w[6][c], w[7][c]);
        *(LAS u32x4*)(dst + c * 1024) = bf;
      }
    };
    if constexpr (first) {
    asm volatile("s_waitcnt vmcnt(%8)"
                 : "+v"(L.wb[so][0]), "+v"(L.wb[so][1]), "+v"(L.wb[so][2]), "+v"(L.wb[so][3]),
                   "+v"(L.wb[so][4]), "+v"(L.wb[so][5]), "+v"(L.wb[so][6]), "+v"(L.wb[so][7])
                 : "i"(C::younger(io, I)) : "memory");
    conv(L.wb[so], base + L.own * 16u);
    }
    if constexpr (k == CW_) {
    asm volatile("s_waitcnt vmcnt(%8)"
                 : "+v"(L.wb[sp][0]), "+v"(L.wb[sp][1]), "+v"(L.wb[sp][2]), "+v"(L.wb[sp][3]),
                   "+v"(L.wb[sp][4]), "+v"(L.wb[sp][5]), "+v"(L.wb[sp][6]), "+v"(L.wb[sp][7])
                 : "i"(C::younger(ip, I)) : "memory");
    conv(L.wb[sp], base + (4096u - L.own * 16u));
    lds_put(L.pFullW, t + 1u);
    }
  } else {
    while (lds_get(L.pDoneOwn) < (unsigned)a) __builtin_amdgcn_s_sleep(GSLEEP);
  }
}
template <int G, int... I> DI void run_all(Loader& L, std::integer_sequence<int, I...>) { (run_op<G, I>(L), ...); }
template <int G> DI void run_grp(Loader& L) { run_all<G>(L, std::make_integer_sequence<int, SC<G>::S.n>{}); }

#define STAMP(i) do {} while (0)

__global__ void __launch_bounds__(512, 2) fused_gemm_a2a(Params p) {
  extern __shared__ __attribute__((aligned(16))) unsigned char smem[];
  const int tid = threadIdx.x, lane = tid & 63;
  const int w = __builtin_amdgcn_readfirstlane(tid >> 6);
  const int b = blockIdx.x;
  STAMP(0);
  const int xcd = b & 7, j = b >> 3;
  const int kq = xcd >> 1, mb2 = j & 1;
  const int nb = (xcd & 1) * 16 + (j >> 1);
  const int rank = p.rank;
  unsigned char* ws = p.self;
  GAS unsigned* tmo = (GAS unsigned*)(ws + WS_TMO);
  const unsigned lds0 = (unsigned)(size_t)(LAS unsigned char*)smem;
  const unsigned fl = lds0 + L_F;
  const int c = w & 3;
  const unsigned dst = (unsigned)(nb >> 2);
  const int idx = ((nb & 3) * 2 + mb2) * 4 + kq;
  const __amdgpu_buffer_rsrc_t rX = __builtin_amdgcn_make_buffer_rsrc((void*)p.x, 0, M * K * 4, 0x00020000);
  const int vX = (lane >> 3) * (K * 4) + (lane & 7) * 16;
  const unsigned vT = ((lane & 7) >> 1) * 256 + (lane >> 3) * 16 + (lane & 1) * 8;
  auto ldx = [&](f32x4(&xa)[8], int cc, int step) {
    const int so = (mb2 * 256 + cc * 64) * (K * 4) + (kq * 1024 + step * 32) * 4;
#pragma unroll
    for (int i = 0; i < 8; i++)
      xa[i] = __builtin_bit_cast(f32x4, __builtin_amdgcn_raw_buffer_load_b128(rX, vX, so + i * 8 * (K * 4), 0));
  };

  const bool remote = dst != (unsigned)rank && dst < (unsigned)NR;
  const int slot = idx * 4 + c;
  const unsigned me = (unsigned)rank;
  const int rowin = mb2 * 256 + c * 64 + kq * 16 + (lane & 15), colb = (nb & 3) * 128 + 16 * (lane >> 4);
  auto q4 = [](float a, float bb, float cc, float d, float rs) {
    const unsigned t0 = __builtin_bit_cast(unsigned, __builtin_fmaf(a, rs, 12582912.0f));
    const unsigned t1 = __builtin_bit_cast(unsigned, __builtin_fmaf(bb, rs, 12582912.0f));
    const unsigned t2 = __builtin_bit_cast(unsigned, __builtin_fmaf(cc, rs, 12582912.0f));
    const unsigned t3 = __builtin_bit_cast(unsigned, __builtin_fmaf(d, rs, 12582912.0f));
    return __builtin_amdgcn_perm(t1, t0, 0x0c0c0400u) | __builtin_amdgcn_perm(t3, t2, 0x04000c0cu);
  };
  auto scale_of = [](unsigned gmax) {
    const float amax = __builtin_bit_cast(float, gmax);
    return amax > 0.f ? 127.0f / amax : 0.f;
  };
  GAS unsigned* afl = (GAS unsigned*)(ws + WS_AFLAG) + (mb2 * 4 + kq) * NSTEP * 32;
  if (w < 4) {
    unsigned call;
    {
      f32x4 xa[8];
      ldx(xa, c, nb);
      GAS unsigned* cnt = (GAS unsigned*)(ws + WS_CNT) + b * 4 + c;
      const unsigned cv = __hip_atomic_load(cnt, RLX_AGENT);
      unsigned char* pub = ws + WS_AT + (((mb2 * 4 + kq) * NSTEP + nb) * 4 + c) * 4096 + lane * 16;
      const unsigned tw0 = __hip_atomic_load((GAS unsigned*)(pub - lane * 16), RLX_AGENT);
      const unsigned tw1 = __hip_atomic_load(afl + nb * 32, RLX_AGENT);
      if (tid < 32) *(LAS unsigned*)(fl + tid * 4) = 0u;
      asm volatile("s_waitcnt lgkmcnt(0)" ::: "memory");
      SB();
      STAMP(12);
      __builtin_amdgcn_s_barrier();
      CFENCE();
      STAMP(13);
      const unsigned sc = lds0 + L_SC + c * 4096;
      x_to_lds(xa, sc + vT, kq);
      call = __builtin_amdgcn_readfirstlane(cv) + 1u;
      asm volatile("" ::"v"(tw0), "v"(tw1));
      u32x4 fr[4];
#pragma unroll
      for (int f = 0; f < 4; f++) fr[f] = *(LAS u32x4*)(sc + f * 1024 + lane * 16);
#pragma unroll
      for (int f = 0; f < 4; f++)
        asm volatile("global_store_dwordx4 %0, %1, off offset:%2 sc0 sc1\n\ts_nop 1" ::"v"(pub), "v"(fr[f]), "i"(f * 1024) : "memory");
      if (lane == 0) __hip_atomic_fetch_add((LAS unsigned*)(fl + F_ST * 4), 1u, __ATOMIC_RELAXED, __HIP_MEMORY_SCOPE_WORKGROUP);
      lds_put(fl + F_CALL * 4, call);
      STAMP(2);
      asm volatile("s_waitcnt vmcnt(0)" ::: "memory");
      if (lane == 0) {
        const unsigned old = __hip_atomic_fetch_add((LAS unsigned*)(fl + F_DR * 4), 1u, __ATOMIC_RELAXED, __HIP_MEMORY_SCOPE_WORKGROUP);
        if (old == 3u) { __hip_atomic_store(afl + nb * 32, call, RLX_SYS); lds_put(fl + F_FS * 4, 4u); }
        __hip_atomic_store(cnt, call, RLX_AGENT);
      }
    }
    STAMP(1);
    if (c == 0) {
      while (lds_get(fl + F_FS * 4) == 0u) {}
      wait8(afl, call, tmo);
      lds_put(fl + F_RDY * 4, 1u);
    }
    f32x4 acc[4][8];
#pragma unroll
    for (int m = 0; m < 4; m++)
#pragma unroll
      for (int n = 0; n < 8; n++) acc[m][n] = f32x4{0, 0, 0, 0};
#define SPIN_T0
#define SPIN_T1
    const unsigned wBase = lds0 + L_W + lane * 16;
    unsigned pfw = fl + F_FULLW * 4, pdone = fl + (F_DONE + c) * 4, vdone = 0;
    asm volatile("" : "+v"(pfw), "+v"(pdone), "+v"(vdone));
    const __amdgpu_buffer_rsrc_t rA = __builtin_amdgcn_make_buffer_rsrc(
        (void*)(ws + WS_AT + ((mb2 * 4 + kq) * NSTEP * 4 + c) * 4096), 0, (NSTEP - 1) * 16384 + 4096, 0x00020000);
    int vA[4];
#pragma unroll
    for (int m = 0; m < 4; m++) { vA[m] = lane * 16 + m * 1024; asm volatile("" : "+v"(vA[m])); }
    u32x4 A[4][4], B[4];
    unsigned fw;
    while (lds_get(fl + F_RDY * 4) == 0u) __builtin_amdgcn_s_sleep(1);
    STAMP(14);
#pragma unroll
    for (int d = 0; d < 3; d++)
#pragma unroll
      for (int m = 0; m < 4; m++) { A[d][m] = __builtin_amdgcn_raw_buffer_load_b128(rA, vA[m], d * 16384, AUX_SYS); SB(); }
    SB();
    while (lds_get(pfw) < 1u) {}
    __builtin_amdgcn_s_setprio(CPRIO);
    STAMP(3);
    B[0] = *(LAS u32x4*)(wBase);
    B[1] = *(LAS u32x4*)(wBase + 1024);
    B[2] = *(LAS u32x4*)(wBase + 2048);
    CFENCE(); fw = *(LAS unsigned*)(pfw + 4); CFENCE();
    unsigned woff = 0, wc = wBase;
    auto mf = [&](int m, int n, u32x4(&Ac)[4]) {
      asm volatile("v_mfma_f32_16x16x32_bf16 %0, %1, %2, %0" : "+v"(acc[m][n]) : "v"(B[n & 3]), "v"(Ac[m]));
    };
    auto step = [&](unsigned s, int q, bool full) {
      unsigned wn = 0, sf = 0;
#pragma unroll
      for (int n = 0; n < 8; n++) {
        mf(full ? 0 : 1, n, A[q]); SB();
        if (n < 5) B[(n + 3) & 3] = *(LAS u32x4*)(wc + (n + 3) * 1024);
        else B[(n + 3) & 3] = *(LAS u32x4*)(wn + (n - 5) * 1024);
        SB(); mf(full ? 1 : 2, n, A[q]); SB();
        if (n == 1) { woff = (woff == (NWS - 1) * 8192u) ? 0u : woff + 8192u; wn = wBase + woff; asm volatile("" : "+v"(wn)); }
        if (n == 2) sf = __builtin_amdgcn_readfirstlane(fw);
        if (n == 4) lds_put(pdone, vdone);
        if (n == 7) { CFENCE(); fw = *(LAS unsigned*)(pfw + ((q + 2) & 3) * 4); CFENCE(); }
        if (full) { SB(); mf(2, n, A[q]); SB(); }
        if (n == 0 && q == 1 && (s & 4u) && s < 24u) { while (lds_get(fl + F_RDY * 4 + ((s + 3u) >> 3) * 4) == 0u) {} CFENCE(); }
        if (full ? (n & 1) == 0 : (n == 0 || n == 3 || n == 5) ) {
          const int m = full ? n >> 1 : (n == 0 ? 1 : n == 3 ? 2 : 3);
          A[(q + 3) & 3][m] = __builtin_amdgcn_raw_buffer_load_b128(rA, vA[m], (s + 3u) * 16384u, AUX_SYS);
        }
        if (n == (full ? 1 : 3)) { vdone += 1; asm volatile("" : "+v"(vdone)); }
        SB(); mf(3, n, A[q]); SB();
        if (n == 3) {
          const unsigned nd = s + 2u;
          if (__builtin_expect(sf < nd && nd <= (unsigned)NSTEP, 0)) {
            SPIN_T0;
            while (lds_get(pfw + ((q + 1) & 3) * 4) < nd) {}
            SPIN_T1;
          }
          CFENCE();
          SB();
        }
      }
      wc = wn;
    };
    asm volatile("s_nop 7");
    for (unsigned s = 0; s < (unsigned)(NSTEP - NDEF); s += 4) {
      step(s, 0, true);
      step(s + 1, 1, true);
      step(s + 2, 2, true);
      step(s + 3, 3, true);
    }
    for (unsigned s = NSTEP - NDEF; s < (unsigned)NSTEP; s += 4) {
      step(s, 0, false);
      step(s + 1, 1, false);
      step(s + 2, 2, false);
      step(s + 3, 3, false);
    }
    u32x4 A0[NDEF ? NDEF : 1];
#pragma unroll
    for (int i = 0; i < NDEF; i++) { A0[i] = __builtin_amdgcn_raw_buffer_load_b128(rA, vA[0], (NSTEP - NDEF + i) * 16384, AUX_SYS); SB(); }
    asm volatile("s_nop 15\n\ts_nop 15");
    SB();
    STAMP(4);
    const unsigned par = call & 1u;
    const __amdgpu_buffer_rsrc_t rE = __builtin_amdgcn_make_buffer_rsrc((void*)(ws + WS_X), 0, X_BYTES, 0x00020000);
#pragma unroll
    for (int m = 1; m < 4; m++) {
      const int so = (((b ^ (m << 1)) * 3 + (m - 1)) * 4 + c) * 4096;
#pragma unroll
      for (int t = 0; t < 4; t++) {
        u32x4 o;
        o.x = pk(acc[m][2 * t][0], acc[m][2 * t][1]); o.y = pk(acc[m][2 * t][2], acc[m][2 * t][3]);
        o.z = pk(acc[m][2 * t + 1][0], acc[m][2 * t + 1][1]); o.w = pk(acc[m][2 * t + 1][2], acc[m][2 * t + 1][3]);
        __builtin_amdgcn_raw_buffer_store_b128(o, rE, lane * 16, so + t * 1024, AUX_SYS);
      }
    }
    SB();
    GAS unsigned* xfl = (GAS unsigned*)(ws + WS_XFLAG);
    auto xflag = [&]() {
      asm volatile("s_waitcnt vmcnt(0)" ::: "memory");
      if (lane < 3) __hip_atomic_store(xfl + (((b ^ ((lane + 1) << 1)) * 4 + c) * 3 + lane) * 32, call, RLX_SYS);
      STAMP(5);
    };
    if (NDEF) {
      u32x4 Bd[8];
      auto wslot = [&](int i) { return wBase + (unsigned)((NSTEP - NDEF + i) % NWS) * 8192u; };
#pragma unroll
      for (int n = 0; n < 8; n++) Bd[n] = *(LAS u32x4*)(wslot(0) + n * 1024);
#pragma unroll
      for (int i = 0; i < NDEF; i++) {
        if (i == XFLAG_AT) xflag();
#pragma unroll
        for (int n = 0; n < 8; n++) {
          SB();
          asm volatile("v_mfma_f32_16x16x32_bf16 %0, %1, %2, %0" : "+v"(acc[0][n]) : "v"(Bd[n]), "v"(A0[i]));
          SB();
          if (i + 1 < NDEF) Bd[n] = *(LAS u32x4*)(wslot(i + 1) + n * 1024);
        }
      }
      asm volatile("s_nop 15\n\ts_nop 15");
      SB();
    }
    if (XFLAG_AT >= NDEF) xflag();
    __builtin_amdgcn_s_setprio(0);
    {
      const u64 t0 = now();
      while (true) {
        const unsigned v = __hip_atomic_load(xfl + ((b * 4 + c) * 3 + min(lane, 2)) * 32, RLX_SYS);
        if (__all(v >= call)) break;
        if (give_up(tmo, call, t0)) break;
      }
      __builtin_amdgcn_fence(__ATOMIC_ACQUIRE, "wavefront");
    }
    u32x4 in[3][4];
#pragma unroll
    for (int m = 0; m < 3; m++)
#pragma unroll
      for (int t = 0; t < 4; t++)
        in[m][t] = __builtin_amdgcn_raw_buffer_load_b128(rE, lane * 16, ((b * 3 + m) * 4 + c) * 4096 + t * 1024, AUX_SYS);
    float am = 0.f;
#pragma unroll
    for (int t = 0; t < 4; t++) {
#pragma unroll
      for (int m = 0; m < 3; m++) {
#pragma unroll
        for (int e = 0; e < 4; e++) {
          const unsigned v = in[m][t][e];
          acc[0][2 * t + (e >> 1)][(e & 1) * 2] += __builtin_bit_cast(float, v << 16);
          acc[0][2 * t + (e >> 1)][(e & 1) * 2 + 1] += __builtin_bit_cast(float, v & 0xffff0000u);
        }
      }
#pragma unroll
      for (int e = 0; e < 8; e++) am = fmaxf(am, fabsf(acc[0][2 * t + (e >> 2)][e & 3]));
    }
#pragma unroll
    for (int o = 32; o > 0; o >>= 1) am = fmaxf(am, __shfl_xor(am, o));
    if (lane == 0) {
      __hip_atomic_fetch_max((LAS unsigned*)(fl + F_MAX * 4), __builtin_bit_cast(unsigned, am), __ATOMIC_RELAXED,
                             __HIP_MEMORY_SCOPE_WORKGROUP);
      CFENCE();
      const unsigned old = __hip_atomic_fetch_add((LAS unsigned*)(fl + F_CNT * 4), 1u, __ATOMIC_RELAXED,
                                                  __HIP_MEMORY_SCOPE_WORKGROUP);
      CFENCE();
      if (old == 3u) {
        const unsigned mx = __hip_atomic_load((LAS unsigned*)(fl + F_MAX * 4), __ATOMIC_RELAXED, __HIP_MEMORY_SCOPE_WORKGROUP);
        __hip_atomic_store((GAS u64*)(ws + WS_LOC) + (par * 256 + b) * 16, ((u64)call << 32) | (u64)mx, RLX_SYS);
        lds_put(fl + F_GS * 4, 1u);
      }
    }
    STAMP(6);
    bool sent = false;
    {
      const unsigned tgo = (unsigned)now() + (unsigned)slot * EGAP;
      while (lds_get(fl + F_GOK * 4) == 0u) {
        if (remote && !sent && slot < ECAP && (int)((unsigned)now() - tgo) >= 0 && lds_get(fl + F_GS * 4) != 0u) {
          unsigned char* ep = p.ws[dst] + WS_E + (size_t)((par * 8 + me) * 128 + slot) * 2048 + lane * 16;
#pragma unroll
          for (int t = 0; t < 2; t++) {
            u32x4 o;
            o.x = pk(acc[0][0][2 * t], acc[0][1][2 * t]); o.y = pk(acc[0][2][2 * t], acc[0][3][2 * t]);
            o.z = pk(acc[0][0][2 * t + 1], acc[0][1][2 * t + 1]); o.w = pk(acc[0][2][2 * t + 1], acc[0][3][2 * t + 1]);
            asm volatile("global_store_dwordx4 %0, %1, off offset:%2 sc0 sc1\n\ts_nop 1" ::"v"(ep), "v"(o), "i"(t * 1024) : "memory");
          }
          asm volatile("s_waitcnt vmcnt(0)" ::: "memory");
          if (lane == 0) __hip_atomic_store((GAS unsigned*)(p.ws[dst] + WS_EF) + (slot * 8 + me) * 32, 2u * call + 1u, RLX_SYS);
          sent = true;
          STAMP(15);
        }
      }
    }
    const float rs = scale_of(lds_get(fl + F_GMAX * 4));
    STAMP(7);
    const bool failed = lds_get(fl + F_GOK * 4) != 1u;
    if (dst < (unsigned)NR) {
      unsigned char* ptr = p.out[dst] + (size_t)(rank * M + rowin) * 512 + colb;
#pragma unroll
      for (int hf = 1; hf >= 0; hf--) {
        if (hf == 0 && sent) break;
        u32x4 o;
#pragma unroll
        for (int rg = 0; rg < 4; rg++)
          o[rg] = failed ? 0x80808080u
                         : q4(acc[0][hf * 4 + 0][rg], acc[0][hf * 4 + 1][rg], acc[0][hf * 4 + 2][rg], acc[0][hf * 4 + 3][rg], rs);
        asm volatile("global_store_dwordx4 %0, %1, off offset:%2 sc0 sc1\n\ts_nop 1" ::"v"(ptr), "v"(o), "i"(hf * 64) : "memory");
      }
    }
    if (remote) {
      if (!sent && lane == 0) __hip_atomic_store((GAS unsigned*)(p.ws[dst] + WS_EF) + (slot * 8 + me) * 32, 2u * call, RLX_SYS);
      asm volatile("s_waitcnt vmcnt(0)" ::: "memory");
      STAMP(8);
      if (lane == 0) __hip_atomic_store((GAS unsigned*)(p.ws[dst] + WS_DONE) + (slot * 8 + me) * 32, call, RLX_SYS);
    } else {
      STAMP(8);
    }
    STAMP(9);
  } else {
    Loader L;
    L.l = c;
        L.rW = __builtin_amdgcn_make_buffer_rsrc((void*)p.w, 0, K * N * 4, 0x00020000);
#pragma unroll
    for (int i = 0; i < 8; i++) L.vW[i] = ((lane >> 4) * 8 + i) * (N * 4) + (lane & 15) * 16;
    L.sW = (unsigned)(kq * 1024 + c * 32) * (N * 4u) + (unsigned)nb * 512u;
    L.own = (unsigned)mb2 * 256u;
    L.ldsW = lds0 + L_W + lane * 16;
    L.fl = fl; L.pDone = fl + F_DONE * 4; L.pDoneOwn = fl + (F_DONE + c) * 4; L.pFullW = fl + (F_FULLW + c) * 4;
    asm volatile("" : "+v"(L.fl), "+v"(L.pDone), "+v"(L.pDoneOwn), "+v"(L.pFullW));
    L.doneAll = 0;
    L.afl = afl; L.tmo = tmo;
    L.tofs = (unsigned)((j >> 1) % TGRP) * (TPER / TGRP);
    L.tr0 = 0;
    __builtin_amdgcn_s_setprio(APRIO);
    __builtin_amdgcn_s_barrier();
    CFENCE();
    run_grp<0>(L);
    const unsigned call = lds_get(fl + F_CALL * 4);
    STAMP(10);
    const unsigned par = call & 1u;
    const u64 t0 = now();
    if (dst == (unsigned)rank && idx == 0) {
      GAS u64* g = (GAS u64*)(ws + WS_LOC) + (par * 256 + c * 64 + lane) * 16;
      unsigned mx = 0;
      while (true) {
        const u64 v = __hip_atomic_load(g, RLX_SYS);
        mx = (unsigned)v;
        if (__all((unsigned)(v >> 32) == call)) break;
        if (give_up(tmo, call, t0)) break;
      }
#pragma unroll
      for (int o = 32; o > 0; o >>= 1) mx = max(mx, (unsigned)__shfl_xor((int)mx, o));
      unsigned old = 0;
      if (lane == 0) {
        __hip_atomic_fetch_max((LAS unsigned*)(fl + F_LMAX * 4), mx, __ATOMIC_RELAXED, __HIP_MEMORY_SCOPE_WORKGROUP);
        CFENCE();
        old = __hip_atomic_fetch_add((LAS unsigned*)(fl + F_LCNT * 4), 1u, __ATOMIC_RELAXED, __HIP_MEMORY_SCOPE_WORKGROUP);
      }
      if (__builtin_amdgcn_readfirstlane(old) == 3u) {
        mx = lds_get(fl + F_LMAX * 4);
        const int srcid = rank;
        __hip_atomic_store((GAS u64*)(p.ws[lane >> 3] + WS_MBOX) + ((par * 8 + (lane & 7)) * 8 + srcid) * 16,
                           ((u64)call << 32) | (u64)mx, RLX_SYS);
      }
    }
    if (c == 1) {
      while (lds_get(fl + F_CNT * 4) < 4u) __builtin_amdgcn_s_sleep(2);
      unsigned mx = 0;
      bool bad;
      GAS u64* g = (GAS u64*)(ws + WS_MBOX) + ((par * 8 + xcd) * 8 + (lane & 7)) * 16;
      while (true) {
        const u64 v = __hip_atomic_load(g, RLX_SYS);
        bad = __hip_atomic_load(tmo, RLX_AGENT) == call;
        mx = (unsigned)v;
        if (bad || __all((unsigned)(v >> 32) == call)) break;
        if (now() - t0 > DEADLINE) { __hip_atomic_store(tmo, call, RLX_AGENT); bad = true; break; }
      }
#pragma unroll
      for (int o = 4; o > 0; o >>= 1) mx = max(mx, (unsigned)__shfl_xor((int)mx, o));
      lds_put(fl + F_GMAX * 4, mx);
      lds_put(fl + F_GOK * 4, bad ? 2u : 1u);
    }
    if (remote) {
      GAS unsigned* ef = (GAS unsigned*)(ws + WS_EF) + (slot * 8 + dst) * 32;
      while (lds_get(fl + F_GS * 4) == 0u) __builtin_amdgcn_s_sleep(4);
      unsigned e;
      bool bad = false;
      while ((e = __builtin_amdgcn_readfirstlane(__hip_atomic_load(ef, RLX_SYS))) < 2u * call) {
        const unsigned ok = lds_get(fl + F_GOK * 4);
        if (ok == 2u || now() - t0 > DEADLINE) { __hip_atomic_store(tmo, call, RLX_AGENT); bad = true; break; }
        if (ok == 0u) __builtin_amdgcn_s_sleep(ESLEEP);
      }
      if (bad || (e & 1u)) {
        __builtin_amdgcn_fence(__ATOMIC_ACQUIRE, "wavefront");
        const __amdgpu_buffer_rsrc_t rI = __builtin_amdgcn_make_buffer_rsrc(
            (void*)(ws + WS_E + (size_t)((par * 8 + dst) * 128 + slot) * 2048), 0, 2048, 0x00020000);
        const u32x4 in0 = __builtin_amdgcn_raw_buffer_load_b128(rI, lane * 16, 0, AUX_SYS);
        const u32x4 in1 = __builtin_amdgcn_raw_buffer_load_b128(rI, lane * 16, 1024, AUX_SYS);
        unsigned ok;
        while ((ok = lds_get(fl + F_GOK * 4)) == 0u) __builtin_amdgcn_s_sleep(1);
        const float rs = scale_of(lds_get(fl + F_GMAX * 4));
        auto lo = [](unsigned v) { return __builtin_bit_cast(float, v << 16); };
        auto hi = [](unsigned v) { return __builtin_bit_cast(float, v & 0xffff0000u); };
        u32x4 o;
        o.x = q4(lo(in0.x), hi(in0.x), lo(in0.y), hi(in0.y), rs);
        o.y = q4(lo(in0.z), hi(in0.z), lo(in0.w), hi(in0.w), rs);
        o.z = q4(lo(in1.x), hi(in1.x), lo(in1.y), hi(in1.y), rs);
        o.w = q4(lo(in1.z), hi(in1.z), lo(in1.w), hi(in1.w), rs);
        if (bad || ok != 1u) o = u32x4{0x80808080u, 0x80808080u, 0x80808080u, 0x80808080u};
        *(GAS u32x4*)(p.out[rank] + (size_t)(dst * M + rowin) * 512 + colb) = o;
      }
    }
    if (dst == (unsigned)rank) {
      while (lds_get(fl + F_GOK * 4) == 0u) __builtin_amdgcn_s_sleep(2);
      GAS unsigned* d = (GAS unsigned*)(ws + WS_DONE) + ((idx * 4 + c) * 8 + (lane & 7)) * 32;
      bool bad = false;
      while (true) {
        const unsigned v = __hip_atomic_load(d, RLX_SYS);
        if (__all(v >= call || (lane & 7) == rank)) break;
        if (give_up(tmo, call, t0)) { bad = true; break; }
        __builtin_amdgcn_s_sleep(1);
      }
      if (bad && lane == 0) p.out[rank][(size_t)(rank * M + mb2 * 256 + c * 64 + kq * 16) * 512 + (nb & 3) * 128] = 0x80;
    }
    STAMP(11);
  }
}

extern "C" void kernel_launch(int rank, int nranks, ncclComm_t comm, hipStream_t stream,
                              void* const* d_in, const int* in_sizes, int n_in,
                              void* d_out, int out_size, const int* in_dtypes, int out_dtype,
                              void* const* d_out_all, void* const* d_ws_all, size_t ws_bytes) {
  if (nranks != NR || n_in < 2 || ws_bytes < WS_NEED || in_sizes[0] != M * K || in_sizes[1] != K * N) {
    fprintf(stderr, "unexpected problem shape\n");
    return;
  }
  static bool init = false;
  if (!init) {
    HIP_CHECK(hipFuncSetAttribute((const void*)fused_gemm_a2a, hipFuncAttributeMaxDynamicSharedMemorySize, LDS_BYTES));
    init = true;
  }
  Params p;
  p.x = (const float*)d_in[0];
  p.w = (const float*)d_in[1];
  for (int i = 0; i < NR; i++) { p.out[i] = (unsigned char*)d_out_all[i]; p.ws[i] = (unsigned char*)d_ws_all[i]; }
  p.rank = rank;
  p.self = (unsigned char*)d_ws_all[rank];
  p.pad = 0;
  hipLaunchKernelGGL(fused_gemm_a2a, dim3(NWG), dim3(512), LDS_BYTES, stream, p);
  HIP_CHECK(hipGetLastError());
}
```

```cpp
#include <hip/hip_runtime.h>
#include <rccl/rccl.h>
#include <cstdio>
#include <utility>

#define HIP_CHECK(expr) do { hipError_t _e = (expr); if (_e != hipSuccess) { \
    fprintf(stderr, "HIP error %s at %s:%d\n", hipGetErrorString(_e), __FILE__, __LINE__); \
    return; } } while (0)

typedef unsigned u32x4 __attribute__((ext_vector_type(4)));
typedef unsigned u32x2 __attribute__((ext_vector_type(2)));
typedef float f32x4 __attribute__((ext_vector_type(4)));
typedef float f32x2 __attribute__((ext_vector_type(2)));
typedef __bf16 bf16x8 __attribute__((ext_vector_type(8)));
typedef __bf16 bf16x2 __attribute__((ext_vector_type(2)));
typedef unsigned long long u64;
#define GAS __attribute__((address_space(1)))
#define LAS __attribute__((address_space(3)))
#define DI __device__ __forceinline__
#pragma clang diagnostic ignored "-Wint-to-pointer-cast"

constexpr int NR = 8;
constexpr int M = 512, N = 4096, K = 4096;
constexpr int NWG = 256;
constexpr int NSTEP = 32;
constexpr int WS_TMO = 0;
constexpr int WS_CNT = 4096;
constexpr int WS_LOC = 8192;
constexpr int WS_AFLAG = 128 * 1024;
constexpr int WS_DONE = 256 * 1024;
constexpr int WS_MBOX = 384 * 1024;
constexpr int WS_XFLAG = 576 * 1024;
constexpr int WS_AT = 1 << 20;
constexpr int WS_X = 5 << 20;
constexpr int X_BYTES = NWG * 3 * 4 * 4096;
constexpr int WS_EF = 17 << 20;
constexpr int WS_E = 18 << 20;
constexpr size_t WS_NEED = (size_t)WS_E + 2 * 8 * 128 * 2048;
constexpr u64 DEADLINE = 40000000ull;
#define NWS 19
constexpr int L_W = 0;
constexpr int L_F = L_W + NWS * 8192;
constexpr int L_SC = L_F - 16384;
constexpr int LDS_BYTES = L_F + 256;
static_assert(LDS_BYTES <= 160 * 1024, "LDS");
constexpr int F_FULLW = 4, F_DONE = 8, F_MAX = 32, F_CNT = 17, F_CALL = 18, F_GOK = 24, F_GRS = 25, F_ST = 26, F_DR = 27, F_RV = 29, F_FS = 30, F_RDY = 12, F_GS = 31, F_LMAX = 36, F_LCNT = 20;
constexpr int AUX_SYS = 17;
#define PSLEEP 2
#define GSLEEP 1
#define NDEF 0
#define XFLAG_AT 11
static_assert(NDEF % 4 == 0 && NDEF <= NWS - 3, "deferred steps stay in the ring");
#define EGAP 5u
#define ECAP 128
#define ESLEEP 12
#define CPRIO 3
#define APRIO 0

struct Params {
  const float* x;
  const float* w;
  unsigned char* self;
  int rank;
  int pad;
  unsigned char* out[NR];
  unsigned char* ws[NR];
};

DI unsigned pk(float a, float b) {
  f32x2 v = {a, b};
  bf16x2 r = __builtin_convertvector(v, bf16x2);
  return __builtin_bit_cast(unsigned, r);
}
DI u64 now() { return __builtin_amdgcn_s_memrealtime(); }
#define RLX_AGENT __ATOMIC_RELAXED, __HIP_MEMORY_SCOPE_AGENT
#define RLX_SYS __ATOMIC_RELAXED, __HIP_MEMORY_SCOPE_SYSTEM
#define CFENCE() asm volatile("" ::: "memory")
#define SB() __builtin_amdgcn_sched_barrier(0)

DI bool give_up(GAS unsigned* tmo, unsigned call, u64 t0) {
  bool bad = (__hip_atomic_load(tmo, RLX_AGENT) == call) || (now() - t0 > DEADLINE);
  if (bad) __hip_atomic_store(tmo, call, RLX_AGENT);
  return bad;
}
DI unsigned wave_max(unsigned v) {
  v = max(v, (unsigned)__builtin_amdgcn_update_dpp(0, (int)v, 0x111, 0xf, 0xf, true));
  v = max(v, (unsigned)__builtin_amdgcn_update_dpp(0, (int)v, 0x112, 0xf, 0xf, true));
  v = max(v, (unsigned)__builtin_amdgcn_update_dpp(0, (int)v, 0x114, 0xf, 0xf, true));
  v = max(v, (unsigned)__builtin_amdgcn_update_dpp(0, (int)v, 0x118, 0xf, 0xf, true));
  const unsigned a = __builtin_amdgcn_readlane(v, 15), b = __builtin_amdgcn_readlane(v, 31);
  const unsigned c = __builtin_amdgcn_readlane(v, 47), d = __builtin_amdgcn_readlane(v, 63);
  return max(max(a, b), max(c, d));
}
DI bool expired(GAS unsigned* tmo, unsigned tv, unsigned call, u64 t0) {
  if (tv == call) return true;
  if (now() - t0 > DEADLINE) { __hip_atomic_store(tmo, call, RLX_AGENT); return true; }
  return false;
}
#define TMO_LOAD() __builtin_amdgcn_readfirstlane(__hip_atomic_load(tmo, RLX_AGENT))
DI void lds_put(unsigned addr, unsigned v) {
  asm volatile("ds_write_b32 %0, %1" ::"v"(addr), "v"(v) : "memory");
}
DI void lds_put2(unsigned addr, unsigned v) {
  u32x2 t = {v, v};
  asm volatile("ds_write_b64 %0, %1" ::"v"(addr), "v"(t) : "memory");
}
DI unsigned lds_get(unsigned addr) {
  unsigned v;
  asm volatile("ds_read_b32 %0, %1\n\ts_waitcnt lgkmcnt(0)" : "=v"(v) : "v"(addr) : "memory");
  return __builtin_amdgcn_readfirstlane(v);
}
DI u32x2 lds_get2(unsigned addr) {
  u32x2 v;
  asm volatile("ds_read_b64 %0, %1\n\ts_waitcnt lgkmcnt(0)" : "=v"(v) : "v"(addr) : "memory");
  return v;
}
DI unsigned lds_min2(unsigned addr) {
  u32x2 v;
  asm volatile("ds_read_b64 %0, %1\n\ts_waitcnt lgkmcnt(0)" : "=v"(v) : "v"(addr) : "memory");
  return __builtin_amdgcn_readfirstlane(min(v.x, v.y));
}
DI unsigned lds_min4(unsigned addr) {
  u32x4 v;
  asm volatile("ds_read_b128 %0, %1\n\ts_waitcnt lgkmcnt(0)" : "=v"(v) : "v"(addr) : "memory");
  return __builtin_amdgcn_readfirstlane(min(min(v.x, v.y), min(v.z, v.w)));
}
DI void wait8(const GAS unsigned* f8, unsigned call, GAS unsigned* tmo) {
  const u64 t0 = now();
  while (true) {
    unsigned a0, a1, a2, a3, a4, a5, a6, a7;
    asm volatile(
        "s_load_dword %0, %8, 0x0 glc\n\ts_load_dword %1, %8, 0x80 glc\n\ts_load_dword %2, %8, 0x100 glc\n\t"
        "s_load_dword %3, %8, 0x180 glc\n\ts_load_dword %4, %8, 0x200 glc\n\ts_load_dword %5, %8, 0x280 glc\n\t"
        "s_load_dword %6, %8, 0x300 glc\n\ts_load_dword %7, %8, 0x380 glc\n\ts_waitcnt lgkmcnt(0)"
        : "=&s"(a0), "=&s"(a1), "=&s"(a2), "=&s"(a3), "=&s"(a4), "=&s"(a5), "=&s"(a6), "=&s"(a7)
        : "s"(f8) : "memory");
    if (min(min(min(a0, a1), min(a2, a3)), min(min(a4, a5), min(a6, a7))) >= call) break;
    if (now() - t0 > DEADLINE) {
      asm volatile("global_store_dword %0, %1, off sc1\n\ts_waitcnt vmcnt(0)" ::"v"(tmo), "v"(call) : "memory");
      break;
    }
    __builtin_amdgcn_s_sleep(PSLEEP);
  }
}
DI void x_to_lds(const f32x4 (&xa)[8], unsigned tb, unsigned kq) {
#pragma unroll
  for (int i = 0; i < 8; i++) {
    u32x2 o = {pk(xa[i][0], xa[i][1]), pk(xa[i][2], xa[i][3])};
    *(LAS u32x2*)(tb + (((unsigned)(i >> 1)) ^ kq) * 1024u + (i & 1) * 128) = o;
  }
}

enum : int { O_, P_, CW_, G_, WS_, WA_, CO_, T_, R_, WD_, Q_, END_ };
struct Op { int k, a; };
#define SCH 70
#define NGRP 1
#define WAUX ""
#define TPER 148
#define TGRP 4
#define TTOL 16
constexpr int MAXOPS = 64;
struct Sched { Op op[MAXOPS]; int n; };
#define TG(r) {G_,4*(r)-7},{O_,r},{CO_,r},{G_,4*(r)-5},{P_,r},{CW_,r}
#define TP(r) {T_,0},{O_,(r) + 1},{CW_,r},{CO_,(r) + 1},{P_,(r) + 1}
constexpr Op PLAN[1][MAXOPS] = {
#define TB(r) {G_,4*(r)-6},{P_,r},{O_,(r)+1},{CW_,r}
  {{O_,0},{WS_,0},{P_,0},{WD_,0},{O_,1},{P_,1},{O_,2},{CW_,0},{CW_,1},{Q_,0},TB(2),TB(3),TB(4),TB(5),TB(6),{G_,22},{P_,7},{CW_,7},{END_,0}},
};
constexpr Sched make_sched(int g) {
  Sched s{};
  while (PLAN[g][s.n].k != END_) { s.op[s.n] = PLAN[g][s.n]; s.n++; }
  return s;
}
constexpr int NSET = 6;
constexpr bool is_ld(int k) { return k == O_ || k == P_; }
template <int G> struct SC {
  static constexpr Sched S = make_sched(G);
  static constexpr int nvm(int i) { return is_ld(S.op[i].k) ? 8 : 0; }
  static constexpr int find_op(int k, int a) { for (int i = 0; i < S.n; i++) if (S.op[i].k == k && S.op[i].a == a) return i; return -1; }
  static constexpr int younger(int t, int i) { int n = 0; for (int j = t + 1; j < i; j++) n += nvm(j); return n > 63 ? 63 : n; }
  static constexpr int set_of(int op) {
    int owner[NSET] = {-1, -1, -1, -1, -1, -1};
    for (int i = 0; i < S.n; i++) {
      if (is_ld(S.op[i].k)) {
        int s = -1;
        for (int q = 0; q < NSET; q++) if (owner[q] < 0) { s = q; break; }
        if (s < 0) return -1;
        owner[s] = S.op[i].a;
        if (i == op) return s;
      } else if (S.op[i].k == CW_) {
        for (int q = 0; q < NSET; q++) if (owner[q] == S.op[i].a) owner[q] = -1;
      }
    }
    return -1;
  }
  static constexpr bool can_reach(int n, int i) { return n <= 0 || find_op(CW_, n / 4) < i; }
  static constexpr bool ok() {
    for (int r = 0; r < 8; r++) {
      const int o = find_op(O_, r), p = find_op(P_, r), c = find_op(CW_, r);
      if (o < 0 || p < 0 || c < o || c < p || set_of(o) < 0 || set_of(p) < 0) return false;
      if (r > 0 && c < find_op(CW_, r - 1)) return false;
      if (!can_reach(4 * r + 3 - NWS + 1, c)) return false;
      if (4 * r + 3 >= NWS - 2 && c < find_op(WS_, 0)) return false;
    }
    for (int i = 0; i < S.n; i++) if (S.op[i].k == G_ && !can_reach(S.op[i].a, i)) return false;
    return true;
  }
};
static_assert(SC<0>::ok() && SC<0>::find_op(Q_, 0) >= 0, "loader schedule");

struct Loader {
  f32x4 wb[NSET][8];
  __amdgpu_buffer_rsrc_t rW;
  int vW[8];
  unsigned sW, own, ldsW, l, doneAll, tofs;
  unsigned fl, pDone, pDoneOwn, pFullW;
  const GAS unsigned* afl;
  GAS unsigned* tmo;
  unsigned tr0;
};
template <int G, int I> DI void run_op(Loader& L) {
  using C = SC<G>;
  constexpr int k = C::S.op[I].k, a = C::S.op[I].a;
  if constexpr (k == WS_) {
    while (lds_get(L.fl + F_ST * 4) < 4u) __builtin_amdgcn_s_sleep(1);
  } else if constexpr (k == WD_) {
    while (lds_get(L.fl + F_FS * 4) < 4u) __builtin_amdgcn_s_sleep(1);
  } else if constexpr (k == WA_) {
    while (lds_get(L.pDoneOwn) == 0u) __builtin_amdgcn_s_sleep(1);
  } else if constexpr (k == Q_) {
    if (L.l != 0u) {
      unsigned call;
      while ((call = lds_get(L.fl + F_CALL * 4)) == 0u) {}
      wait8(L.afl + L.l * 8u * 32u, call, L.tmo);
      lds_put(L.fl + (F_RDY + L.l) * 4, 1u);
    }
  } else if constexpr (is_ld(k)) {
    constexpr int s = C::set_of(I);
    const unsigned so = (unsigned)a * (128u * N * 4u) + L.sW + ((k == O_) ? L.own : (256u - L.own));
    asm volatile(
        "buffer_load_dwordx4 %0, %8, %16, %17 offen" WAUX "\n\t"
        "buffer_load_dwordx4 %1, %9, %16, %17 offen" WAUX "\n\t"
        "buffer_load_dwordx4 %2, %10, %16, %17 offen" WAUX "\n\t"
        "buffer_load_dwordx4 %3, %11, %16, %17 offen" WAUX "\n\t"
        "buffer_load_dwordx4 %4, %12, %16, %17 offen" WAUX "\n\t"
        "buffer_load_dwordx4 %5, %13, %16, %17 offen" WAUX "\n\t"
        "buffer_load_dwordx4 %6, %14, %16, %17 offen" WAUX "\n\t"
        "buffer_load_dwordx4 %7, %15, %16, %17 offen" WAUX
        : "=&v"(L.wb[s][0]), "=&v"(L.wb[s][1]), "=&v"(L.wb[s][2]), "=&v"(L.wb[s][3]),
          "=&v"(L.wb[s][4]), "=&v"(L.wb[s][5]), "=&v"(L.wb[s][6]), "=&v"(L.wb[s][7])
        : "v"(L.vW[0]), "v"(L.vW[1]), "v"(L.vW[2]), "v"(L.vW[3]), "v"(L.vW[4]), "v"(L.vW[5]), "v"(L.vW[6]), "v"(L.vW[7]),
          "s"(L.rW), "s"(so)
        : "memory");
  } else if constexpr (k == R_) {
    asm volatile("s_mov_b64 exec, 1\n\tds_add_u32 %0, %1\n\ts_mov_b64 exec, -1" ::"v"(L.fl + F_RV * 4), "v"(1u) : "memory");
    while (lds_get(L.fl + F_RV * 4) < 4u * a) {}
  } else if constexpr (k == T_) {
    const unsigned t = (unsigned)now(), ph = (t - L.tofs) % (unsigned)TPER;
    if (ph >= (unsigned)TTOL) {
      const unsigned tgt = t + (TPER - ph);
      while ((int)((unsigned)now() - tgt) < 0) __builtin_amdgcn_s_sleep(GSLEEP);
    }
  } else if constexpr (k == CW_ || k == CO_) {
    constexpr bool first = k == CO_ || C::find_op(CO_, a) < 0;
    constexpr int io = C::find_op(O_, a), ip = C::find_op(P_, a);
    constexpr int so = C::set_of(io), sp = C::set_of(ip);
    const unsigned t = 4u * a + L.l;
    if constexpr (first && 4 * a + 3 >= NWS) {
      const int need = (int)t - NWS + 1;
      while ((int)L.doneAll < need) { L.doneAll = lds_min4(L.pDone); if ((int)L.doneAll < need) __builtin_amdgcn_s_sleep(1); }
    }
    unsigned slot = (4u * a) % NWS + L.l;
    if (slot >= (unsigned)NWS) slot -= NWS;
    const unsigned base = L.ldsW + slot * 8192u;
    auto conv = [&](f32x4(&w)[8], unsigned dst) {
#pragma unroll
      for (int c = 0; c < 4; c++) {
        u32x4 bf;
        bf.x = pk(w[0][c], w[1][c]); bf.y = pk(w[2][c], w[3][c]);
        bf.z = pk(w[4][c], w[5][c]); bf.w = pk(w[6][c], w[7][c]);
        *(LAS u32x4*)(dst + c * 1024) = bf;
      }
    };
    if constexpr (first) {
    asm volatile("s_waitcnt vmcnt(%8)"
                 : "+v"(L.wb[so][0]), "+v"(L.wb[so][1]), "+v"(L.wb[so][2]), "+v"(L.wb[so][3]),
                   "+v"(L.wb[so][4]), "+v"(L.wb[so][5]), "+v"(L.wb[so][6]), "+v"(L.wb[so][7])
                 : "i"(C::younger(io, I)) : "memory");
    conv(L.wb[so], base + L.own * 16u);
    }
    if constexpr (k == CW_) {
    asm volatile("s_waitcnt vmcnt(%8)"
                 : "+v"(L.wb[sp][0]), "+v"(L.wb[sp][1]), "+v"(L.wb[sp][2]), "+v"(L.wb[sp][3]),
                   "+v"(L.wb[sp][4]), "+v"(L.wb[sp][5]), "+v"(L.wb[sp][6]), "+v"(L.wb[sp][7])
                 : "i"(C::younger(ip, I)) : "memory");
    conv(L.wb[sp], base + (4096u - L.own * 16u));
    lds_put(L.pFullW, t + 1u);
    }
  } else {
    while (lds_get(L.pDoneOwn) < (unsigned)a) __builtin_amdgcn_s_sleep(GSLEEP);
  }
}
template <int G, int... I> DI void run_all(Loader& L, std::integer_sequence<int, I...>) { (run_op<G, I>(L), ...); }
template <int G> DI void run_grp(Loader& L) { run_all<G>(L, std::make_integer_sequence<int, SC<G>::S.n>{}); }

#define STAMP(i) do {} while (0)

__global__ void __launch_bounds__(512, 2) fused_gemm_a2a(Params p) {
  extern __shared__ __attribute__((aligned(16))) unsigned char smem[];
  const int tid = threadIdx.x, lane = tid & 63;
  const int w = __builtin_amdgcn_readfirstlane(tid >> 6);
  const int b = blockIdx.x;
  STAMP(0);
  const int xcd = b & 7, j = b >> 3;
  const int kq = xcd >> 1, mb2 = j & 1;
  const int nb = (xcd & 1) * 16 + (j >> 1);
  const __amdgpu_buffer_rsrc_t rX = __builtin_amdgcn_make_buffer_rsrc((void*)p.x, 0, w < 4 ? M * K * 4 : 0, 0x00020000);
  f32x4 xa[8];
  unsigned cv = 0, tw0 = 0, tw1 = 0;
  if (w < 4) {
    const int vX = (lane >> 3) * (K * 4) + (lane & 7) * 16;
    const int so = (mb2 * 256 + w * 64) * (K * 4) + (kq * 1024 + nb * 32) * 4;
#pragma unroll
    for (int i = 0; i < 8; i++)
      xa[i] = __builtin_bit_cast(f32x4, __builtin_amdgcn_raw_buffer_load_b128(rX, vX, so + i * 8 * (K * 4), 0));
    SB();
    cv = __hip_atomic_load((GAS unsigned*)(p.self + WS_CNT) + b * 4 + w, RLX_AGENT);
    tw0 = __hip_atomic_load((GAS unsigned*)(p.self + WS_AT + (((mb2 * 4 + kq) * NSTEP + nb) * 4 + w) * 4096), RLX_AGENT);
    tw1 = __hip_atomic_load((GAS unsigned*)(p.self + WS_AFLAG) + ((mb2 * 4 + kq) * NSTEP + nb) * 32, RLX_AGENT);
    if (tid < 64) *(LAS unsigned*)((unsigned)(size_t)(LAS unsigned char*)smem + L_F + tid * 4) = 0u;
    asm volatile("s_waitcnt lgkmcnt(0)" ::: "memory");
    SB();
  }
  __builtin_amdgcn_s_barrier();
  asm volatile("" ::: "memory");
  SB();
  const int rank = p.rank;
  unsigned char* ws = p.self;
  GAS unsigned* tmo = (GAS unsigned*)(ws + WS_TMO);
  const unsigned lds0 = (unsigned)(size_t)(LAS unsigned char*)smem;
  const unsigned fl = lds0 + L_F;
  const int c = w & 3;
  const unsigned dst = (unsigned)(nb >> 2);
  const int idx = ((nb & 3) * 2 + mb2) * 4 + kq;
  const unsigned vT = ((lane & 7) >> 1) * 256 + (lane >> 3) * 16 + (lane & 1) * 8;

  const bool remote = dst != (unsigned)rank && dst < (unsigned)NR;
  const int slot = idx * 4 + c;
  const unsigned me = (unsigned)rank;
  const int rowin = mb2 * 256 + c * 64 + kq * 16 + (lane & 15), colb = (nb & 3) * 128 + 16 * (lane >> 4);
  auto q4 = [](float a, float bb, float cc, float d, float rs) {
    const unsigned t0 = __builtin_bit_cast(unsigned, __builtin_fmaf(a, rs, 12582912.0f));
    const unsigned t1 = __builtin_bit_cast(unsigned, __builtin_fmaf(bb, rs, 12582912.0f));
    const unsigned t2 = __builtin_bit_cast(unsigned, __builtin_fmaf(cc, rs, 12582912.0f));
    const unsigned t3 = __builtin_bit_cast(unsigned, __builtin_fmaf(d, rs, 12582912.0f));
    return __builtin_amdgcn_perm(t1, t0, 0x0c0c0400u) | __builtin_amdgcn_perm(t3, t2, 0x04000c0cu);
  };
  auto scale_of = [](unsigned gmax) {
    const float amax = __builtin_bit_cast(float, gmax);
    return amax > 0.f ? 127.0f / amax : 0.f;
  };
  GAS unsigned* afl = (GAS unsigned*)(ws + WS_AFLAG) + (mb2 * 4 + kq) * NSTEP * 32;
  if (__builtin_expect(w < 4, 1)) {
    unsigned call;
    {
      GAS unsigned* cnt = (GAS unsigned*)(ws + WS_CNT) + b * 4 + c;
      unsigned char* pub = ws + WS_AT + (((mb2 * 4 + kq) * NSTEP + nb) * 4 + c) * 4096 + lane * 16;
      STAMP(12);
      CFENCE();
      STAMP(13);
      const unsigned sc = lds0 + L_SC + c * 4096;
      x_to_lds(xa, sc + vT, kq);
      call = __builtin_amdgcn_readfirstlane(cv) + 1u;
      asm volatile("" ::"v"(tw0), "v"(tw1));
      u32x4 fr[4];
#pragma unroll
      for (int f = 0; f < 4; f++) fr[f] = *(LAS u32x4*)(sc + f * 1024 + lane * 16);
#pragma unroll
      for (int f = 0; f < 4; f++)
        asm volatile("global_store_dwordx4 %0, %1, off offset:%2 sc0 sc1\n\ts_nop 1" ::"v"(pub), "v"(fr[f]), "i"(f * 1024) : "memory");
      if (lane == 0) __hip_atomic_fetch_add((LAS unsigned*)(fl + F_ST * 4), 1u, __ATOMIC_RELAXED, __HIP_MEMORY_SCOPE_WORKGROUP);
      lds_put(fl + F_CALL * 4, call);
      STAMP(2);
      asm volatile("s_waitcnt vmcnt(0)" ::: "memory");
      if (lane == 0) {
        const unsigned old = __hip_atomic_fetch_add((LAS unsigned*)(fl + F_DR * 4), 1u, __ATOMIC_RELAXED, __HIP_MEMORY_SCOPE_WORKGROUP);
        if (old == 3u) { __hip_atomic_store(afl + nb * 32, call, RLX_SYS); lds_put(fl + F_FS * 4, 4u); }
        __hip_atomic_store(cnt, call, RLX_AGENT);
      }
    }
    STAMP(1);
    if (c == 0) {
      while (lds_get(fl + F_FS * 4) == 0u) {}
      wait8(afl, call, tmo);
      lds_put(fl + F_RDY * 4, 1u);
    }
    f32x4 acc[4][8];
#pragma unroll
    for (int m = 0; m < 4; m++)
#pragma unroll
      for (int n = 0; n < 8; n++) acc[m][n] = f32x4{0, 0, 0, 0};
#define SPIN_T0
#define SPIN_T1
    const unsigned wBase = lds0 + L_W + lane * 16;
    unsigned pfw = fl + F_FULLW * 4, pdone = fl + (F_DONE + c) * 4, vdone = 0;
    asm volatile("" : "+v"(pfw), "+v"(pdone), "+v"(vdone));
    const __amdgpu_buffer_rsrc_t rA = __builtin_amdgcn_make_buffer_rsrc(
        (void*)(ws + WS_AT + ((mb2 * 4 + kq) * NSTEP * 4 + c) * 4096), 0, (NSTEP - 1) * 16384 + 4096, 0x00020000);
    int vA[4];
#pragma unroll
    for (int m = 0; m < 4; m++) { vA[m] = lane * 16 + m * 1024; asm volatile("" : "+v"(vA[m])); }
    u32x4 A[4][4], B[4];
    unsigned fw;
    while (lds_get(fl + F_RDY * 4) == 0u) __builtin_amdgcn_s_sleep(1);
    STAMP(14);
#pragma unroll
    for (int d = 0; d < 3; d++)
#pragma unroll
      for (int m = 0; m < 4; m++) { A[d][m] = __builtin_amdgcn_raw_buffer_load_b128(rA, vA[m], d * 16384, AUX_SYS); SB(); }
    SB();
    while (lds_get(pfw) < 1u) {}
    __builtin_amdgcn_s_setprio(CPRIO);
    STAMP(3);
    B[0] = *(LAS u32x4*)(wBase);
    B[1] = *(LAS u32x4*)(wBase + 1024);
    B[2] = *(LAS u32x4*)(wBase + 2048);
    CFENCE(); fw = *(LAS unsigned*)(pfw + 4); CFENCE();
    unsigned woff = 0, wc = wBase;
    auto mf = [&](int m, int n, u32x4(&Ac)[4]) {
      asm volatile("v_mfma_f32_16x16x32_bf16 %0, %1, %2, %0" : "+v"(acc[m][n]) : "v"(B[n & 3]), "v"(Ac[m]));
    };
    auto step = [&](unsigned s, int q, bool full) {
      unsigned wn = 0, sf = 0;
#pragma unroll
      for (int n = 0; n < 8; n++) {
        mf(full ? 0 : 1, n, A[q]); SB();
        if (n < 5) B[(n + 3) & 3] = *(LAS u32x4*)(wc + (n + 3) * 1024);
        else B[(n + 3) & 3] = *(LAS u32x4*)(wn + (n - 5) * 1024);
        SB(); mf(full ? 1 : 2, n, A[q]); SB();
        if (n == 1) { woff = (woff == (NWS - 1) * 8192u) ? 0u : woff + 8192u; wn = wBase + woff; asm volatile("" : "+v"(wn)); }
        if (n == 2) sf = __builtin_amdgcn_readfirstlane(fw);
        if (n == 4) lds_put(pdone, vdone);
        if (n == 7) { CFENCE(); fw = *(LAS unsigned*)(pfw + ((q + 2) & 3) * 4); CFENCE(); }
        if (full) { SB(); mf(2, n, A[q]); SB(); }
        if (n == 0 && q == 1 && (s & 4u) && s < 24u) { while (lds_get(fl + F_RDY * 4 + ((s + 3u) >> 3) * 4) == 0u) {} CFENCE(); }
        if (full ? (n & 1) == 0 : (n == 0 || n == 3 || n == 5) ) {
          const int m = full ? n >> 1 : (n == 0 ? 1 : n == 3 ? 2 : 3);
          A[(q + 3) & 3][m] = __builtin_amdgcn_raw_buffer_load_b128(rA, vA[m], (s + 3u) * 16384u, AUX_SYS);
        }
        if (n == (full ? 1 : 3)) { vdone += 1; asm volatile("" : "+v"(vdone)); }
        SB(); mf(3, n, A[q]); SB();
        if (n == 3) {
          const unsigned nd = s + 2u;
          if (__builtin_expect(sf < nd && nd <= (unsigned)NSTEP, 0)) {
            SPIN_T0;
            while (lds_get(pfw + ((q + 1) & 3) * 4) < nd) {}
            SPIN_T1;
          }
          CFENCE();
          SB();
        }
      }
      wc = wn;
    };
    asm volatile("s_nop 7");
    for (unsigned s = 0; s < (unsigned)(NSTEP - NDEF); s += 4) {
      step(s, 0, true);
      step(s + 1, 1, true);
      step(s + 2, 2, true);
      step(s + 3, 3, true);
    }
    for (unsigned s = NSTEP - NDEF; s < (unsigned)NSTEP; s += 4) {
      step(s, 0, false);
      step(s + 1, 1, false);
      step(s + 2, 2, false);
      step(s + 3, 3, false);
    }
    u32x4 A0[NDEF ? NDEF : 1];
#pragma unroll
    for (int i = 0; i < NDEF; i++) { A0[i] = __builtin_amdgcn_raw_buffer_load_b128(rA, vA[0], (NSTEP - NDEF + i) * 16384, AUX_SYS); SB(); }
    asm volatile("s_nop 15\n\ts_nop 15");
    SB();
    STAMP(4);
    const unsigned par = call & 1u;
    const __amdgpu_buffer_rsrc_t rE = __builtin_amdgcn_make_buffer_rsrc((void*)(ws + WS_X), 0, X_BYTES, 0x00020000);
#pragma unroll
    for (int m = 1; m < 4; m++) {
      const int so = (((b ^ (m << 1)) * 3 + (m - 1)) * 4 + c) * 4096;
#pragma unroll
      for (int t = 0; t < 4; t++) {
        u32x4 o;
        o.x = pk(acc[m][2 * t][0], acc[m][2 * t][1]); o.y = pk(acc[m][2 * t][2], acc[m][2 * t][3]);
        o.z = pk(acc[m][2 * t + 1][0], acc[m][2 * t + 1][1]); o.w = pk(acc[m][2 * t + 1][2], acc[m][2 * t + 1][3]);
        __builtin_amdgcn_raw_buffer_store_b128(o, rE, lane * 16, so + t * 1024, AUX_SYS);
      }
    }
    SB();
    GAS unsigned* xfl = (GAS unsigned*)(ws + WS_XFLAG);
    auto xflag = [&]() {
      asm volatile("s_waitcnt vmcnt(0)" ::: "memory");
      if (lane < 3) __hip_atomic_store(xfl + (((b ^ ((lane + 1) << 1)) * 4 + c) * 3 + lane) * 32, call, RLX_SYS);
      STAMP(5);
    };
    if (NDEF) {
      u32x4 Bd[8];
      auto wslot = [&](int i) { return wBase + (unsigned)((NSTEP - NDEF + i) % NWS) * 8192u; };
#pragma unroll
      for (int n = 0; n < 8; n++) Bd[n] = *(LAS u32x4*)(wslot(0) + n * 1024);
#pragma unroll
      for (int i = 0; i < NDEF; i++) {
        if (i == XFLAG_AT) xflag();
#pragma unroll
        for (int n = 0; n < 8; n++) {
          SB();
          asm volatile("v_mfma_f32_16x16x32_bf16 %0, %1, %2, %0" : "+v"(acc[0][n]) : "v"(Bd[n]), "v"(A0[i]));
          SB();
          if (i + 1 < NDEF) Bd[n] = *(LAS u32x4*)(wslot(i + 1) + n * 1024);
        }
      }
      asm volatile("s_nop 15\n\ts_nop 15");
      SB();
    }
    if (XFLAG_AT >= NDEF) xflag();
    __builtin_amdgcn_s_setprio(0);
    {
      const u64 t0 = now();
      while (true) {
        const unsigned v = __hip_atomic_load(xfl + ((b * 4 + c) * 3 + min(lane, 2)) * 32, RLX_SYS);
        const unsigned tv = __hip_atomic_load(tmo, RLX_AGENT);
        if (__all(v >= call)) break;
        if (expired(tmo, __builtin_amdgcn_readfirstlane(tv), call, t0)) break;
      }
      __builtin_amdgcn_fence(__ATOMIC_ACQUIRE, "wavefront");
    }
    u32x4 in[3][4];
#pragma unroll
    for (int m = 0; m < 3; m++)
#pragma unroll
      for (int t = 0; t < 4; t++)
        in[m][t] = __builtin_amdgcn_raw_buffer_load_b128(rE, lane * 16, ((b * 3 + m) * 4 + c) * 4096 + t * 1024, AUX_SYS);
    float am = 0.f;
#pragma unroll
    for (int t = 0; t < 4; t++) {
#pragma unroll
      for (int m = 0; m < 3; m++) {
#pragma unroll
        for (int e = 0; e < 4; e++) {
          const unsigned v = in[m][t][e];
          acc[0][2 * t + (e >> 1)][(e & 1) * 2] += __builtin_bit_cast(float, v << 16);
          acc[0][2 * t + (e >> 1)][(e & 1) * 2 + 1] += __builtin_bit_cast(float, v & 0xffff0000u);
        }
      }
#pragma unroll
      for (int e = 0; e < 8; e++) am = fmaxf(am, fabsf(acc[0][2 * t + (e >> 2)][e & 3]));
    }
    {
      const unsigned wm = wave_max(__builtin_bit_cast(unsigned, am));
      unsigned old;
      asm volatile("s_mov_b64 exec, 1\n\tds_write_b32 %1, %2\n\tds_add_rtn_u32 %0, %3, %4\n\ts_mov_b64 exec, -1\n\ts_waitcnt lgkmcnt(0)"
                   : "=&v"(old) : "v"(fl + (F_MAX + c) * 4), "v"(wm), "v"(fl + F_CNT * 4), "v"(1u) : "memory");
      if (__builtin_amdgcn_readfirstlane(old) == 3u) {
        u32x4 m4;
        asm volatile("ds_read_b128 %0, %1\n\ts_waitcnt lgkmcnt(0)" : "=v"(m4) : "v"(fl + F_MAX * 4) : "memory");
        const unsigned mx = max(max(m4.x, m4.y), max(m4.z, m4.w));
        if (lane == 0) __hip_atomic_store((GAS u64*)(ws + WS_LOC) + (par * 256 + b) * 16, ((u64)call << 32) | (u64)mx, RLX_SYS);
        lds_put(fl + F_GS * 4, 1u);
      }
    }
    STAMP(6);
    bool sent = false;
    u32x2 gv;
    unsigned char* optr = p.out[dst & 7] + (size_t)(rank * M + rowin) * 512 + colb;
    GAS unsigned* dnp = (GAS unsigned*)(p.ws[dst & 7] + WS_DONE) + (slot * 8 + me) * 32;
    asm volatile("" : "+v"(optr), "+v"(dnp));
    {
      const unsigned tgo = (unsigned)now() + (unsigned)slot * EGAP;
      while (gv = lds_get2(fl + F_GOK * 4), __builtin_amdgcn_readfirstlane(gv.x) == 0u) {
        if (remote && !sent && slot < ECAP && (int)((unsigned)now() - tgo) >= 0 && lds_get(fl + F_GS * 4) != 0u) {
          unsigned char* ep = p.ws[dst] + WS_E + (size_t)((par * 8 + me) * 128 + slot) * 2048 + lane * 16;
#pragma unroll
          for (int t = 0; t < 2; t++) {
            u32x4 o;
            o.x = pk(acc[0][0][2 * t], acc[0][1][2 * t]); o.y = pk(acc[0][2][2 * t], acc[0][3][2 * t]);
            o.z = pk(acc[0][0][2 * t + 1], acc[0][1][2 * t + 1]); o.w = pk(acc[0][2][2 * t + 1], acc[0][3][2 * t + 1]);
            asm volatile("global_store_dwordx4 %0, %1, off offset:%2 sc0 sc1\n\ts_nop 1" ::"v"(ep), "v"(o), "i"(t * 1024) : "memory");
          }
          asm volatile("s_waitcnt vmcnt(0)" ::: "memory");
          if (lane == 0) __hip_atomic_store((GAS unsigned*)(p.ws[dst] + WS_EF) + (slot * 8 + me) * 32, 2u * call + 1u, RLX_SYS);
          sent = true;
          STAMP(15);
        }
      }
    }
    const unsigned rsb = gv.y;
    const float rs = __builtin_bit_cast(float, rsb);
    STAMP(7);
    const bool failed = gv.x != 1u;
    if (dst < (unsigned)NR) {
      unsigned char* ptr = optr;
#pragma unroll
      for (int hf = 1; hf >= 0; hf--) {
        if (hf == 0 && sent) break;
        u32x4 o;
#pragma unroll
        for (int rg = 0; rg < 4; rg++)
        {
          float v0 = acc[0][hf * 4 + 0][rg], v1 = acc[0][hf * 4 + 1][rg], v2 = acc[0][hf * 4 + 2][rg], v3 = acc[0][hf * 4 + 3][rg];
          if (hf == 0 && remote) {
            const unsigned p01 = pk(v0, v1), p23 = pk(v2, v3);
            v0 = __builtin_bit_cast(float, p01 << 16); v1 = __builtin_bit_cast(float, p01 & 0xffff0000u);
            v2 = __builtin_bit_cast(float, p23 << 16); v3 = __builtin_bit_cast(float, p23 & 0xffff0000u);
          }
          o[rg] = failed ? 0x80808080u : q4(v0, v1, v2, v3, rs);
        }
        asm volatile("global_store_dwordx4 %0, %1, off offset:%2 sc0 sc1\n\ts_nop 1" ::"v"(ptr), "v"(o), "i"(hf * 64) : "memory");
      }
    }
    if (remote) {
      if (!sent && lane == 0) __hip_atomic_store((GAS unsigned*)(p.ws[dst] + WS_EF) + (slot * 8 + me) * 32, 2u * call, RLX_SYS);
      asm volatile("s_waitcnt vmcnt(0)" ::: "memory");
      STAMP(8);
      if (lane == 0) __hip_atomic_store(dnp, call, RLX_SYS);
    } else {
      STAMP(8);
    }
    STAMP(9);
  } else {
    Loader L;
    L.l = c;
        L.rW = __builtin_amdgcn_make_buffer_rsrc((void*)p.w, 0, K * N * 4, 0x00020000);
#pragma unroll
    for (int i = 0; i < 8; i++) L.vW[i] = ((lane >> 4) * 8 + i) * (N * 4) + (lane & 15) * 16;
    L.sW = (unsigned)(kq * 1024 + c * 32) * (N * 4u) + (unsigned)nb * 512u;
    L.own = (unsigned)mb2 * 256u;
    L.ldsW = lds0 + L_W + lane * 16;
    L.fl = fl; L.pDone = fl + F_DONE * 4; L.pDoneOwn = fl + (F_DONE + c) * 4; L.pFullW = fl + (F_FULLW + c) * 4;
    asm volatile("" : "+v"(L.fl), "+v"(L.pDone), "+v"(L.pDoneOwn), "+v"(L.pFullW));
    L.doneAll = 0;
    L.afl = afl; L.tmo = tmo;
    L.tofs = (unsigned)((j >> 1) % TGRP) * (TPER / TGRP);
    L.tr0 = 0;
    __builtin_amdgcn_s_setprio(APRIO);
    CFENCE();
    run_grp<0>(L);
    const unsigned call = lds_get(fl + F_CALL * 4);
    STAMP(10);
    const unsigned par = call & 1u;
    const u64 t0 = now();
    if (dst == (unsigned)rank && idx == 0) {
      GAS u64* g = (GAS u64*)(ws + WS_LOC) + (par * 256 + c * 64 + lane) * 16;
      const int srcid = rank;
      GAS u64* mb = (GAS u64*)(p.ws[lane >> 3] + WS_MBOX) + ((par * 8 + (lane & 7)) * 8 + srcid) * 16;
      asm volatile("" : "+v"(mb));
      unsigned mx = 0;
      while (true) {
        const u64 v = __hip_atomic_load(g, RLX_SYS);
        const unsigned tv = __hip_atomic_load(tmo, RLX_AGENT);
        mx = (unsigned)v;
        if (__all((unsigned)(v >> 32) == call)) break;
        if (expired(tmo, __builtin_amdgcn_readfirstlane(tv), call, t0)) break;
      }
      mx = wave_max(mx);
      unsigned old;
      asm volatile("s_mov_b64 exec, 1\n\tds_write_b32 %1, %2\n\tds_add_rtn_u32 %0, %3, %4\n\ts_mov_b64 exec, -1\n\ts_waitcnt lgkmcnt(0)"
                   : "=&v"(old) : "v"(fl + (F_LMAX + c) * 4), "v"(mx), "v"(fl + F_LCNT * 4), "v"(1u) : "memory");
      if (__builtin_amdgcn_readfirstlane(old) == 3u) {
        u32x4 m4;
        asm volatile("ds_read_b128 %0, %1\n\ts_waitcnt lgkmcnt(0)" : "=v"(m4) : "v"(fl + F_LMAX * 4) : "memory");
        mx = max(max(m4.x, m4.y), max(m4.z, m4.w));
        __hip_atomic_store(mb, ((u64)call << 32) | (u64)mx, RLX_SYS);
      }
    }
    if (c == 1) {
      while (lds_get(fl + F_CNT * 4) < 4u) __builtin_amdgcn_s_sleep(2);
      unsigned mx = 0;
      bool bad;
      GAS u64* g = (GAS u64*)(ws + WS_MBOX) + ((par * 8 + xcd) * 8 + (lane & 7)) * 16;
      while (true) {
        const u64 v = __hip_atomic_load(g, RLX_SYS);
        bad = __hip_atomic_load(tmo, RLX_AGENT) == call;
        mx = (unsigned)v;
        if (bad || __all((unsigned)(v >> 32) == call)) break;
        if (now() - t0 > DEADLINE) { __hip_atomic_store(tmo, call, RLX_AGENT); bad = true; break; }
      }
      mx = wave_max(mx);
      u32x2 gw = {bad ? 2u : 1u, __builtin_bit_cast(unsigned, scale_of(mx))};
      asm volatile("ds_write_b64 %0, %1" ::"v"(fl + F_GOK * 4), "v"(gw) : "memory");
    }
    if (remote) {
      GAS unsigned* ef = (GAS unsigned*)(ws + WS_EF) + (slot * 8 + dst) * 32;
      while (lds_get(fl + F_GS * 4) == 0u) __builtin_amdgcn_s_sleep(4);
      unsigned e;
      bool bad = false;
      while ((e = __builtin_amdgcn_readfirstlane(__hip_atomic_load(ef, RLX_SYS))) < 2u * call) {
        const unsigned ok = lds_get(fl + F_GOK * 4);
        if (ok == 2u || now() - t0 > DEADLINE) { __hip_atomic_store(tmo, call, RLX_AGENT); bad = true; break; }
        if (ok == 0u) __builtin_amdgcn_s_sleep(ESLEEP);
      }
      if (bad || (e & 1u)) {
        __builtin_amdgcn_fence(__ATOMIC_ACQUIRE, "wavefront");
        const __amdgpu_buffer_rsrc_t rI = __builtin_amdgcn_make_buffer_rsrc(
            (void*)(ws + WS_E + (size_t)((par * 8 + dst) * 128 + slot) * 2048), 0, 2048, 0x00020000);
        const u32x4 in0 = __builtin_amdgcn_raw_buffer_load_b128(rI, lane * 16, 0, AUX_SYS);
        const u32x4 in1 = __builtin_amdgcn_raw_buffer_load_b128(rI, lane * 16, 1024, AUX_SYS);
        unsigned ok;
        while ((ok = lds_get(fl + F_GOK * 4)) == 0u) __builtin_amdgcn_s_sleep(1);
        const float rs = __builtin_bit_cast(float, lds_get(fl + F_GRS * 4));
        auto lo = [](unsigned v) { return __builtin_bit_cast(float, v << 16); };
        auto hi = [](unsigned v) { return __builtin_bit_cast(float, v & 0xffff0000u); };
        u32x4 o;
        o.x = q4(lo(in0.x), hi(in0.x), lo(in0.y), hi(in0.y), rs);
        o.y = q4(lo(in0.z), hi(in0.z), lo(in0.w), hi(in0.w), rs);
        o.z = q4(lo(in1.x), hi(in1.x), lo(in1.y), hi(in1.y), rs);
        o.w = q4(lo(in1.z), hi(in1.z), lo(in1.w), hi(in1.w), rs);
        if (bad || ok != 1u) o = u32x4{0x80808080u, 0x80808080u, 0x80808080u, 0x80808080u};
        *(GAS u32x4*)(p.out[rank] + (size_t)(dst * M + rowin) * 512 + colb) = o;
      }
    }
    if (dst == (unsigned)rank) {
      while (lds_get(fl + F_GOK * 4) == 0u) __builtin_amdgcn_s_sleep(2);
      GAS unsigned* d = (GAS unsigned*)(ws + WS_DONE) + ((idx * 4 + c) * 8 + (lane & 7)) * 32;
      bool bad = false;
      while (true) {
        const unsigned v = __hip_atomic_load(d, RLX_SYS);
        const unsigned tv = __hip_atomic_load(tmo, RLX_AGENT);
        if (__all(v >= call || (lane & 7) == rank)) break;
        if (expired(tmo, __builtin_amdgcn_readfirstlane(tv), call, t0)) { bad = true; break; }
      }
      if (bad && lane == 0) p.out[rank][(size_t)(rank * M + mb2 * 256 + c * 64 + kq * 16) * 512 + (nb & 3) * 128] = 0x80;
    }
    STAMP(11);
  }
}

extern "C" void kernel_launch(int rank, int nranks, ncclComm_t comm, hipStream_t stream,
                              void* const* d_in, const int* in_sizes, int n_in,
                              void* d_out, int out_size, const int* in_dtypes, int out_dtype,
                              void* const* d_out_all, void* const* d_ws_all, size_t ws_bytes) {
  if (nranks != NR || n_in < 2 || ws_bytes < WS_NEED || in_sizes[0] != M * K || in_sizes[1] != K * N) {
    fprintf(stderr, "unexpected problem shape\n");
    return;
  }
  static bool init = false;
  if (!init) {
    HIP_CHECK(hipFuncSetAttribute((const void*)fused_gemm_a2a, hipFuncAttributeMaxDynamicSharedMemorySize, LDS_BYTES));
    init = true;
  }
  Params p;
  p.x = (const float*)d_in[0];
  p.w = (const float*)d_in[1];
  for (int i = 0; i < NR; i++) { p.out[i] = (unsigned char*)d_out_all[i]; p.ws[i] = (unsigned char*)d_ws_all[i]; }
  p.rank = rank;
  p.self = (unsigned char*)d_ws_all[rank];
  p.pad = 0;
  hipLaunchKernelGGL(fused_gemm_a2a, dim3(NWG), dim3(512), LDS_BYTES, stream, p);
  HIP_CHECK(hipGetLastError());
}
```

```cpp
#include <hip/hip_runtime.h>
#include <rccl/rccl.h>
#include <cstdio>
#include <utility>

#define HIP_CHECK(expr) do { hipError_t _e = (expr); if (_e != hipSuccess) { \
    fprintf(stderr, "HIP error %s at %s:%d\n", hipGetErrorString(_e), __FILE__, __LINE__); \
    return; } } while (0)

typedef unsigned u32x4 __attribute__((ext_vector_type(4)));
typedef unsigned u32x2 __attribute__((ext_vector_type(2)));
typedef float f32x4 __attribute__((ext_vector_type(4)));
typedef float f32x2 __attribute__((ext_vector_type(2)));
typedef __bf16 bf16x2 __attribute__((ext_vector_type(2)));
typedef unsigned long long u64;
#define GAS __attribute__((address_space(1)))
#define LAS __attribute__((address_space(3)))
#define DI __device__ __forceinline__
#pragma clang diagnostic ignored "-Wint-to-pointer-cast"

constexpr int NR = 8;
constexpr int M = 512, N = 4096, K = 4096;
constexpr int NWG = 256;
constexpr int NSTEP = 32;
constexpr int WS_TMO = 0;
constexpr int WS_CNT = 4096;
constexpr int WS_LOC = 8192;
constexpr int WS_AFLAG = 64 * 1024;
constexpr int WS_MBOX = 96 * 1024;
constexpr int WS_BOOK = 120 * 1024;
constexpr int WS_DONE = 128 * 1024;
constexpr int WS_EF = 256 * 1024;
constexpr int WS_XFLAG = 576 * 1024;
constexpr int WS_AT = 1 << 20;
constexpr int WS_X = 5 << 20;
constexpr int X_BYTES = NWG * 3 * 4 * 4096;
constexpr int WS_E = 18 << 20;
constexpr size_t WS_NEED = (size_t)WS_E + 2 * 8 * 128 * 2048;
constexpr u64 DEADLINE = 40000000ull;
constexpr int NWS = 19;
constexpr int L_F = NWS * 8192;
constexpr int L_SC = L_F - 16384;
constexpr int LDS_BYTES = L_F + 256;
static_assert(LDS_BYTES <= 160 * 1024, "LDS");
enum : int {
  F_FULLW = 0,
  F_DONE = 4,
  F_RDY = 8,
  F_MAX = 12,
  F_LMAX = 16,
  F_GOK = 20,
  F_CALL = 22, F_ST, F_DR, F_FS, F_CNT, F_GS, F_LCNT, F_TS
};
constexpr int AUX_SYS = 17;
constexpr unsigned EGAP = 5;

struct Params {
  const float* x;
  const float* w;
  unsigned char* self;
  int rank;
  int pad;
  unsigned char* out[NR];
  unsigned char* ws[NR];
};

#define RLX_AGENT __ATOMIC_RELAXED, __HIP_MEMORY_SCOPE_AGENT
#define RLX_SYS __ATOMIC_RELAXED, __HIP_MEMORY_SCOPE_SYSTEM
#define CFENCE() asm volatile("" ::: "memory")
#define SB() __builtin_amdgcn_sched_barrier(0)
#define UNI(x) __builtin_amdgcn_readfirstlane(x)

DI unsigned pk(float a, float b) {
  f32x2 v = {a, b};
  bf16x2 r = __builtin_convertvector(v, bf16x2);
  return __builtin_bit_cast(unsigned, r);
}
DI float bf_lo(unsigned v) { return __builtin_bit_cast(float, v << 16); }
DI float bf_hi(unsigned v) { return __builtin_bit_cast(float, v & 0xffff0000u); }
DI unsigned q4(float a, float b, float c, float d, float rs) {
  const unsigned t0 = __builtin_bit_cast(unsigned, __builtin_fmaf(a, rs, 12582912.0f));
  const unsigned t1 = __builtin_bit_cast(unsigned, __builtin_fmaf(b, rs, 12582912.0f));
  const unsigned t2 = __builtin_bit_cast(unsigned, __builtin_fmaf(c, rs, 12582912.0f));
  const unsigned t3 = __builtin_bit_cast(unsigned, __builtin_fmaf(d, rs, 12582912.0f));
  return __builtin_amdgcn_perm(t1, t0, 0x0c0c0400u) | __builtin_amdgcn_perm(t3, t2, 0x04000c0cu);
}
DI u64 now() { return __builtin_amdgcn_s_memrealtime(); }
DI unsigned wave_max(unsigned v) {
  v = max(v, (unsigned)__builtin_amdgcn_update_dpp(0, (int)v, 0x111, 0xf, 0xf, true));
  v = max(v, (unsigned)__builtin_amdgcn_update_dpp(0, (int)v, 0x112, 0xf, 0xf, true));
  v = max(v, (unsigned)__builtin_amdgcn_update_dpp(0, (int)v, 0x114, 0xf, 0xf, true));
  v = max(v, (unsigned)__builtin_amdgcn_update_dpp(0, (int)v, 0x118, 0xf, 0xf, true));
  const unsigned a = __builtin_amdgcn_readlane(v, 15), b = __builtin_amdgcn_readlane(v, 31);
  const unsigned c = __builtin_amdgcn_readlane(v, 47), d = __builtin_amdgcn_readlane(v, 63);
  return max(max(a, b), max(c, d));
}
DI bool expired(GAS unsigned* tmo, unsigned tv, unsigned call, u64 t0) {
  if (UNI(tv) == call) return true;
  if (now() - t0 > DEADLINE) { __hip_atomic_store(tmo, call, RLX_AGENT); return true; }
  return false;
}
DI void store_sys(const void* p, u32x4 v) {
  asm volatile("global_store_dwordx4 %0, %1, off sc0 sc1\n\ts_nop 1" ::"v"(p), "v"(v) : "memory");
}
DI void lds_put(unsigned addr, unsigned v) { asm volatile("ds_write_b32 %0, %1" ::"v"(addr), "v"(v) : "memory"); }
DI unsigned lds_get(unsigned addr) {
  unsigned v;
  asm volatile("ds_read_b32 %0, %1\n\ts_waitcnt lgkmcnt(0)" : "=v"(v) : "v"(addr) : "memory");
  return UNI(v);
}
DI u32x2 lds_get2(unsigned addr) {
  u32x2 v;
  asm volatile("ds_read_b64 %0, %1\n\ts_waitcnt lgkmcnt(0)" : "=v"(v) : "v"(addr) : "memory");
  return v;
}
DI u32x4 lds_get4(unsigned addr) {
  u32x4 v;
  asm volatile("ds_read_b128 %0, %1\n\ts_waitcnt lgkmcnt(0)" : "=v"(v) : "v"(addr) : "memory");
  return v;
}
DI unsigned lds_count(unsigned cnt) {
  unsigned old;
  asm volatile("s_mov_b64 exec, 1\n\tds_add_rtn_u32 %0, %1, %2\n\ts_mov_b64 exec, -1\n\ts_waitcnt lgkmcnt(0)"
               : "=&v"(old) : "v"(cnt), "v"(1u) : "memory");
  return UNI(old);
}
DI void wait8(const GAS unsigned* f8, unsigned call, GAS unsigned* tmo) {
  const u64 t0 = now();
  for (unsigned it = 1;; it++) {
    unsigned a0, a1, a2, a3, a4, a5, a6, a7;
    asm volatile(
        "s_load_dword %0, %8, 0x0 glc\n\ts_load_dword %1, %8, 0x80 glc\n\ts_load_dword %2, %8, 0x100 glc\n\t"
        "s_load_dword %3, %8, 0x180 glc\n\ts_load_dword %4, %8, 0x200 glc\n\ts_load_dword %5, %8, 0x280 glc\n\t"
        "s_load_dword %6, %8, 0x300 glc\n\ts_load_dword %7, %8, 0x380 glc\n\ts_waitcnt lgkmcnt(0)"
        : "=&s"(a0), "=&s"(a1), "=&s"(a2), "=&s"(a3), "=&s"(a4), "=&s"(a5), "=&s"(a6), "=&s"(a7)
        : "s"(f8) : "memory");
    if (min(min(min(a0, a1), min(a2, a3)), min(min(a4, a5), min(a6, a7))) >= call) break;
    if ((it & 255u) == 0u) {
      unsigned tv;
      asm volatile("global_load_dword %0, %1, off sc1\n\ts_waitcnt vmcnt(0)" : "=&v"(tv) : "v"(tmo) : "memory");
      if (UNI(tv) == call) break;
      if (now() - t0 > DEADLINE) {
        asm volatile("global_store_dword %0, %1, off sc1\n\ts_waitcnt vmcnt(0)" ::"v"(tmo), "v"(call) : "memory");
        break;
      }
    }
    __builtin_amdgcn_s_sleep(2);
  }
}

enum : int { O_, P_, CW_, G_, WS_, WF_, Q_ };
struct Op { int k, a; };
#define RND(r) {G_, 4 * (r) - 6}, {P_, r}, {O_, (r) + 1}, {CW_, r}
constexpr Op SCHED[] = {
  {O_, 0}, {WS_, 0}, {P_, 0}, {WF_, 0}, {O_, 1}, {P_, 1}, {O_, 2}, {CW_, 0}, {CW_, 1}, {Q_, 0},
  RND(2), RND(3), RND(4), RND(5), RND(6), {G_, 22}, {P_, 7}, {CW_, 7},
};
constexpr int NOPS = sizeof(SCHED) / sizeof(Op);
constexpr int NSET = 5;
constexpr bool is_ld(int k) { return k == O_ || k == P_; }
constexpr int find_op(int k, int a) { for (int i = 0; i < NOPS; i++) if (SCHED[i].k == k && SCHED[i].a == a) return i; return -1; }
constexpr int younger(int t, int i) { int n = 0; for (int j = t + 1; j < i; j++) n += is_ld(SCHED[j].k) ? 8 : 0; return n; }
constexpr int set_of(int op) {
  int owner[NSET] = {-1, -1, -1, -1, -1};
  for (int i = 0; i < NOPS; i++) {
    if (is_ld(SCHED[i].k)) {
      int s = -1;
      for (int q = 0; q < NSET; q++) if (owner[q] < 0) { s = q; break; }
      if (s < 0) return -1;
      owner[s] = SCHED[i].a;
      if (i == op) return s;
    } else if (SCHED[i].k == CW_) {
      for (int q = 0; q < NSET; q++) if (owner[q] == SCHED[i].a) owner[q] = -1;
    }
  }
  return -1;
}
constexpr bool can_reach(int n, int i) { return n <= 0 || find_op(CW_, n / 4) < i; }
constexpr bool sched_ok() {
  for (int r = 0; r < 8; r++) {
    const int o = find_op(O_, r), p = find_op(P_, r), c = find_op(CW_, r);
    if (o < 0 || p < o || c < p || set_of(o) < 0 || set_of(p) < 0 || younger(o, c) > 63) return false;
    if (r > 0 && c < find_op(CW_, r - 1)) return false;
    if (!can_reach(4 * r + 3 - NWS + 1, c)) return false;
    if (4 * r + 3 >= NWS - 2 && c < find_op(WS_, 0)) return false;
  }
  for (int i = 0; i < NOPS; i++) if (SCHED[i].k == G_ && (!can_reach(SCHED[i].a, i) || i < find_op(Q_, 0))) return false;
  return true;
}
static_assert(sched_ok(), "loader schedule");

struct Loader {
  f32x4 wb[NSET][8];
  __amdgpu_buffer_rsrc_t rW;
  int vW[8];
  unsigned sW, own, ldsW, l, fl;
  const GAS unsigned* afl;
  GAS unsigned* tmo;
};
#define WAIT_SET(S, N)                                                                               \
  asm volatile("s_waitcnt vmcnt(%8)"                                                                 \
               : "+v"(L.wb[S][0]), "+v"(L.wb[S][1]), "+v"(L.wb[S][2]), "+v"(L.wb[S][3]),             \
                 "+v"(L.wb[S][4]), "+v"(L.wb[S][5]), "+v"(L.wb[S][6]), "+v"(L.wb[S][7])              \
               : "i"(N) : "memory")
template <int I> DI void run_op(Loader& L) {
  constexpr int k = SCHED[I].k, a = SCHED[I].a;
  if constexpr (is_ld(k)) {
    constexpr int s = set_of(I);
    const unsigned so = (unsigned)a * (128u * N * 4u) + L.sW + ((k == O_) ? L.own : (256u - L.own));
    asm volatile(
        "buffer_load_dwordx4 %0, %8, %16, %17 offen\n\t"
        "buffer_load_dwordx4 %1, %9, %16, %17 offen\n\t"
        "buffer_load_dwordx4 %2, %10, %16, %17 offen\n\t"
        "buffer_load_dwordx4 %3, %11, %16, %17 offen\n\t"
        "buffer_load_dwordx4 %4, %12, %16, %17 offen\n\t"
        "buffer_load_dwordx4 %5, %13, %16, %17 offen\n\t"
        "buffer_load_dwordx4 %6, %14, %16, %17 offen\n\t"
        "buffer_load_dwordx4 %7, %15, %16, %17 offen"
        : "=&v"(L.wb[s][0]), "=&v"(L.wb[s][1]), "=&v"(L.wb[s][2]), "=&v"(L.wb[s][3]),
          "=&v"(L.wb[s][4]), "=&v"(L.wb[s][5]), "=&v"(L.wb[s][6]), "=&v"(L.wb[s][7])
        : "v"(L.vW[0]), "v"(L.vW[1]), "v"(L.vW[2]), "v"(L.vW[3]), "v"(L.vW[4]), "v"(L.vW[5]), "v"(L.vW[6]), "v"(L.vW[7]),
          "s"(L.rW), "s"(so)
        : "memory");
  } else if constexpr (k == CW_) {
    constexpr int io = find_op(O_, a), ip = find_op(P_, a), so = set_of(io), sp = set_of(ip);
    const unsigned t = 4u * a + L.l;
    if constexpr (4 * a + 3 >= NWS) {
      const int need = (int)t - NWS + 1;
      while (true) {
        const u32x4 d = lds_get4(L.fl + F_DONE * 4);
        if ((int)UNI(min(min(d.x, d.y), min(d.z, d.w))) >= need) break;
        __builtin_amdgcn_s_sleep(1);
      }
    }
    unsigned slot = (4u * a) % NWS + L.l;
    if (slot >= (unsigned)NWS) slot -= NWS;
    const unsigned base = L.ldsW + slot * 8192u;
    auto conv = [&](f32x4(&w)[8], unsigned dst) {
#pragma unroll
      for (int c = 0; c < 4; c++) {
        u32x4 bf;
        bf.x = pk(w[0][c], w[1][c]); bf.y = pk(w[2][c], w[3][c]);
        bf.z = pk(w[4][c], w[5][c]); bf.w = pk(w[6][c], w[7][c]);
        *(LAS u32x4*)(dst + c * 1024) = bf;
      }
    };
    WAIT_SET(so, younger(io, I));
    conv(L.wb[so], base + L.own * 16u);
    WAIT_SET(sp, younger(ip, I));
    conv(L.wb[sp], base + (4096u - L.own * 16u));
    lds_put(L.fl + (F_FULLW + L.l) * 4, t + 1u);
  } else if constexpr (k == G_) {
    while (lds_get(L.fl + (F_DONE + L.l) * 4) < (unsigned)a) __builtin_amdgcn_s_sleep(1);
  } else if constexpr (k == WS_) {
    while (lds_get(L.fl + F_ST * 4) < 4u) __builtin_amdgcn_s_sleep(1);
  } else if constexpr (k == WF_) {
    while (lds_get(L.fl + F_FS * 4) == 0u) __builtin_amdgcn_s_sleep(1);
  } else if (L.l != 0u) {
    wait8(L.afl + L.l * 8u * 32u, lds_get(L.fl + F_CALL * 4), L.tmo);
    lds_put(L.fl + (F_RDY + L.l) * 4, 1u);
  }
}
template <int... I> DI void run_all(Loader& L, std::integer_sequence<int, I...>) { (run_op<I>(L), ...); }


__global__ void __launch_bounds__(512, 2) fused_gemm_a2a(Params p) {
  extern __shared__ __attribute__((aligned(16))) unsigned char smem[];
  const int tid = threadIdx.x, lane = tid & 63;
  const int w = UNI(tid >> 6);
  const int b = blockIdx.x;
  const int xcd = b & 7, j = b >> 3;
  const int kq = xcd >> 1, mb2 = j & 1;
  const int nb = (xcd & 1) * 16 + (j >> 1);
  const int grp = mb2 * 4 + kq;
  const unsigned lds0 = (unsigned)(size_t)(LAS unsigned char*)smem;
  const unsigned fl = lds0 + L_F;
  f32x4 xa[8];
  unsigned cv = 0, tw0 = 0, tw1 = 0;
  if (w < 4) {
    const __amdgpu_buffer_rsrc_t rX = __builtin_amdgcn_make_buffer_rsrc((void*)p.x, 0, M * K * 4, 0x00020000);
    const int vX = (lane >> 3) * (K * 4) + (lane & 7) * 16;
    const int so = (mb2 * 256 + w * 64) * (K * 4) + (kq * 1024 + nb * 32) * 4;
#pragma unroll
    for (int i = 0; i < 8; i++)
      xa[i] = __builtin_bit_cast(f32x4, __builtin_amdgcn_raw_buffer_load_b128(rX, vX, so + i * 8 * (K * 4), 0));
    SB();
    cv = __hip_atomic_load((GAS unsigned*)(p.self + WS_CNT) + b * 4 + w, RLX_AGENT);
    tw0 = __hip_atomic_load((GAS unsigned*)(p.self + WS_AT + ((grp * NSTEP + nb) * 4 + w) * 4096), RLX_AGENT);
    tw1 = __hip_atomic_load((GAS unsigned*)(p.self + WS_AFLAG) + (grp * NSTEP + nb) * 32, RLX_AGENT);
    if (tid < 64) *(LAS unsigned*)(fl + tid * 4) = 0u;
    asm volatile("s_waitcnt lgkmcnt(0)" ::: "memory");
    SB();
  }
  __builtin_amdgcn_s_barrier();
  CFENCE();
  SB();
  const int rank = p.rank;
  unsigned char* ws = p.self;
  GAS unsigned* tmo = (GAS unsigned*)(ws + WS_TMO);
  GAS unsigned* afl = (GAS unsigned*)(ws + WS_AFLAG) + grp * NSTEP * 32;
  const int c = w & 3;
  const unsigned dst = (unsigned)(nb >> 2) & 7u;
  const int idx = ((nb & 3) * 2 + mb2) * 4 + kq;
  const int slot = idx * 4 + c;
  const bool remote = dst != (unsigned)rank;
  const unsigned me = (unsigned)rank;
  const int rowin = mb2 * 256 + c * 64 + kq * 16 + (lane & 15), colb = (nb & 3) * 128 + 16 * (lane >> 4);

  if (__builtin_expect(w < 4, 1)) {
    const unsigned call = UNI(cv) + 1u;
    {
      const unsigned sc = lds0 + L_SC + c * 4096;
      const unsigned tb = sc + ((lane & 7) >> 1) * 256 + (lane >> 3) * 16 + (lane & 1) * 8;
#pragma unroll
      for (int i = 0; i < 8; i++) {
        u32x2 o = {pk(xa[i][0], xa[i][1]), pk(xa[i][2], xa[i][3])};
        *(LAS u32x2*)(tb + (((unsigned)(i >> 1)) ^ (unsigned)kq) * 1024u + (i & 1) * 128) = o;
      }
      asm volatile("" ::"v"(tw0), "v"(tw1));
      u32x4 fr[4];
#pragma unroll
      for (int f = 0; f < 4; f++) fr[f] = *(LAS u32x4*)(sc + f * 1024 + lane * 16);
      unsigned char* pub = ws + WS_AT + ((grp * NSTEP + nb) * 4 + c) * 4096 + lane * 16;
#pragma unroll
      for (int f = 0; f < 4; f++) store_sys(pub + f * 1024, fr[f]);
      lds_put(fl + F_CALL * 4, call);
      lds_count(fl + F_ST * 4);
      asm volatile("s_waitcnt vmcnt(0)" ::: "memory");
      if (lds_count(fl + F_DR * 4) == 3u) {
        if (lane == 0) __hip_atomic_store(afl + nb * 32, call, RLX_SYS);
        lds_put(fl + F_FS * 4, 1u);
      }
      if (lane == 0) __hip_atomic_store((GAS unsigned*)(ws + WS_CNT) + b * 4 + c, call, RLX_AGENT);
      if (c == 0) {
        while (lds_get(fl + F_FS * 4) == 0u) {}
        wait8(afl, call, tmo);
        lds_put(fl + F_RDY * 4, 1u);
      }
    }
    f32x4 acc[4][8];
#pragma unroll
    for (int m = 0; m < 4; m++)
#pragma unroll
      for (int n = 0; n < 8; n++) { acc[m][n] = f32x4{0, 0, 0, 0}; asm volatile("" : "+v"(acc[m][n])); }
    const unsigned wBase = lds0 + lane * 16;
    unsigned pfw = fl + F_FULLW * 4, pdone = fl + (F_DONE + c) * 4, vdone = 0;
    asm volatile("" : "+v"(pfw), "+v"(pdone), "+v"(vdone));
    const __amdgpu_buffer_rsrc_t rA = __builtin_amdgcn_make_buffer_rsrc(
        (void*)(ws + WS_AT + (grp * NSTEP * 4 + c) * 4096), 0, (NSTEP - 1) * 16384 + 4096, 0x00020000);
    int vA[4];
#pragma unroll
    for (int m = 0; m < 4; m++) { vA[m] = lane * 16 + m * 1024; asm volatile("" : "+v"(vA[m])); }
    u32x4 A[4][4], B[4];
    unsigned fw;
    while (lds_get(fl + F_RDY * 4) == 0u) __builtin_amdgcn_s_sleep(1);
#pragma unroll
    for (int d = 0; d < 3; d++)
#pragma unroll
      for (int m = 0; m < 4; m++) { A[d][m] = __builtin_amdgcn_raw_buffer_load_b128(rA, vA[m], d * 16384, AUX_SYS); SB(); }
    while (lds_get(pfw) < 1u) {}
    __builtin_amdgcn_s_setprio(3);
    B[0] = *(LAS u32x4*)(wBase);
    B[1] = *(LAS u32x4*)(wBase + 1024);
    B[2] = *(LAS u32x4*)(wBase + 2048);
    CFENCE(); fw = *(LAS unsigned*)(pfw + 4); CFENCE();
    unsigned woff = 0, wc = wBase;
    auto mf = [&](int m, int n, int q) {
      asm volatile("v_mfma_f32_16x16x32_bf16 %0, %1, %2, %0" : "+v"(acc[m][n]) : "v"(B[n & 3]), "v"(A[q][m]));
    };
    auto step = [&](unsigned s, int q) {
      unsigned wn = 0, sf = 0;
#pragma unroll
      for (int n = 0; n < 8; n++) {
        mf(0, n, q); SB();
        if (n < 5) B[(n + 3) & 3] = *(LAS u32x4*)(wc + (n + 3) * 1024);
        else B[(n + 3) & 3] = *(LAS u32x4*)(wn + (n - 5) * 1024);
        SB(); mf(1, n, q); SB();
        if (n == 1) { woff = (woff == (NWS - 1) * 8192u) ? 0u : woff + 8192u; wn = wBase + woff; asm volatile("" : "+v"(wn)); }
        if (n == 2) sf = UNI(fw);
        if (n == 4) lds_put(pdone, vdone);
        if (n == 7) { CFENCE(); fw = *(LAS unsigned*)(pfw + ((q + 2) & 3) * 4); CFENCE(); }
        SB(); mf(2, n, q); SB();
        if (n == 0 && q == 1 && (s & 4u) && s < 24u) { while (lds_get(fl + F_RDY * 4 + ((s + 3u) >> 3) * 4) == 0u) {} }
        if ((n & 1) == 0) A[(q + 3) & 3][n >> 1] = __builtin_amdgcn_raw_buffer_load_b128(rA, vA[n >> 1], (s + 3u) * 16384u, AUX_SYS);
        if (n == 1) { vdone += 1; asm volatile("" : "+v"(vdone)); }
        SB(); mf(3, n, q); SB();
        if (n == 3) {
          const unsigned nd = s + 2u;
          if (__builtin_expect(sf < nd && nd <= (unsigned)NSTEP, 0))
            while (lds_get(pfw + ((q + 1) & 3) * 4) < nd) {}
          CFENCE();
          SB();
        }
      }
      wc = wn;
    };
    asm volatile("s_nop 7");
    for (unsigned s = 0; s < (unsigned)NSTEP; s += 4) {
      step(s, 0);
      step(s + 1, 1);
      step(s + 2, 2);
      step(s + 3, 3);
    }
    asm volatile("s_nop 15\n\ts_nop 15");
    SB();
    __builtin_amdgcn_s_setprio(0);
    const __amdgpu_buffer_rsrc_t rE = __builtin_amdgcn_make_buffer_rsrc((void*)(ws + WS_X), 0, X_BYTES, 0x00020000);
#pragma unroll
    for (int m = 1; m < 4; m++) {
      const int so = (((b ^ (m << 1)) * 3 + (m - 1)) * 4 + c) * 4096;
#pragma unroll
      for (int t = 0; t < 4; t++) {
        u32x4 o;
        o.x = pk(acc[m][2 * t][0], acc[m][2 * t][1]); o.y = pk(acc[m][2 * t][2], acc[m][2 * t][3]);
        o.z = pk(acc[m][2 * t + 1][0], acc[m][2 * t + 1][1]); o.w = pk(acc[m][2 * t + 1][2], acc[m][2 * t + 1][3]);
        __builtin_amdgcn_raw_buffer_store_b128(o, rE, lane * 16, so + t * 1024, AUX_SYS);
      }
    }
    GAS unsigned* xfl = (GAS unsigned*)(ws + WS_XFLAG);
    asm volatile("s_waitcnt vmcnt(0)" ::: "memory");
    if (lane < 3) __hip_atomic_store(xfl + (((b ^ ((lane + 1) << 1)) * 4 + c) * 3 + lane) * 32, call, RLX_SYS);
    {
      const u64 t0 = now();
      while (true) {
        const unsigned v = __hip_atomic_load(xfl + ((b * 4 + c) * 3 + min(lane, 2)) * 32, RLX_SYS);
        const unsigned tv = __hip_atomic_load(tmo, RLX_AGENT);
        if (__all(v >= call) || expired(tmo, tv, call, t0)) break;
      }
      __builtin_amdgcn_fence(__ATOMIC_ACQUIRE, "wavefront");
    }
    u32x4 in[3][4];
#pragma unroll
    for (int m = 0; m < 3; m++)
#pragma unroll
      for (int t = 0; t < 4; t++)
        in[m][t] = __builtin_amdgcn_raw_buffer_load_b128(rE, lane * 16, ((b * 3 + m) * 4 + c) * 4096 + t * 1024, AUX_SYS);
    float am = 0.f;
#pragma unroll
    for (int t = 0; t < 4; t++) {
#pragma unroll
      for (int m = 0; m < 3; m++)
#pragma unroll
        for (int e = 0; e < 4; e++) {
          const unsigned v = in[m][t][e];
          acc[0][2 * t + (e >> 1)][(e & 1) * 2] += bf_lo(v);
          acc[0][2 * t + (e >> 1)][(e & 1) * 2 + 1] += bf_hi(v);
        }
#pragma unroll
      for (int e = 0; e < 8; e++) am = fmaxf(am, fabsf(acc[0][2 * t + (e >> 2)][e & 3]));
    }
    lds_put(fl + (F_MAX + c) * 4, wave_max(__builtin_bit_cast(unsigned, am)));
    if (lds_count(fl + F_CNT * 4) == 3u) {
      const u32x4 m4 = lds_get4(fl + F_MAX * 4);
      const unsigned mx = max(max(m4.x, m4.y), max(m4.z, m4.w));
      if (lane == 0) __hip_atomic_store((GAS u64*)(ws + WS_LOC) + b * 16, ((u64)call << 32) | (u64)mx, RLX_SYS);
      lds_put(fl + F_GS * 4, 1u);
      if (remote) {
        GAS u64* bk = (GAS u64*)(ws + WS_BOOK) + dst * 16;
        u64 tb = 0;
        if (lane == 0) {
          __hip_atomic_fetch_max(bk, now(), RLX_AGENT);
          tb = __hip_atomic_fetch_add(bk, (u64)(4u * EGAP), RLX_AGENT);
        }
        lds_put(fl + F_TS * 4, UNI((unsigned)tb) | 1u);
      }
    }
    const unsigned par = call & 1u;
    u32x4 h0, h1;
    h0.x = pk(acc[0][0][0], acc[0][1][0]); h0.y = pk(acc[0][2][0], acc[0][3][0]);
    h0.z = pk(acc[0][0][1], acc[0][1][1]); h0.w = pk(acc[0][2][1], acc[0][3][1]);
    h1.x = pk(acc[0][0][2], acc[0][1][2]); h1.y = pk(acc[0][2][2], acc[0][3][2]);
    h1.z = pk(acc[0][0][3], acc[0][1][3]); h1.w = pk(acc[0][2][3], acc[0][3][3]);
    unsigned char* optr = p.out[dst] + (size_t)(rank * M + rowin) * 512 + colb;
    GAS unsigned* dnp = (GAS unsigned*)(p.ws[dst] + WS_DONE) + (slot * 8 + me) * 32;
    GAS unsigned* efp = (GAS unsigned*)(p.ws[dst] + WS_EF) + ((par * 128 + slot) * 8 + me) * 32;
    asm volatile("" : "+v"(optr), "+v"(dnp), "+v"(efp));
    bool sent = false;
    u32x2 gv;
    {
      unsigned ts = 0;
      while (gv = lds_get2(fl + F_GOK * 4), UNI(gv.x) == 0u) {
        if (remote && ts == 0u) ts = lds_get(fl + F_TS * 4);
        if (ts != 0u && !sent && (int)((unsigned)now() - (ts + (unsigned)c * EGAP)) >= 0) {
          unsigned char* ep = p.ws[dst] + WS_E + (size_t)((par * 8 + me) * 128 + slot) * 2048 + lane * 16;
          store_sys(ep, h0);
          store_sys(ep + 1024, h1);
          asm volatile("s_waitcnt vmcnt(0)" ::: "memory");
          if (lane == 0) __hip_atomic_store(efp, 2u * call + 1u, RLX_SYS);
          sent = true;
        }
      }
    }
    const unsigned rsb = gv.y;
    const float rs = __builtin_bit_cast(float, rsb);
    const bool failed = gv.x != 1u;
    const u32x4 poison = {0x80808080u, 0x80808080u, 0x80808080u, 0x80808080u};
    {
      u32x4 o;
#pragma unroll
      for (int rg = 0; rg < 4; rg++) o[rg] = q4(acc[0][4][rg], acc[0][5][rg], acc[0][6][rg], acc[0][7][rg], rs);
      store_sys(optr + 64, failed ? poison : o);
    }
    if (!sent) {
      u32x4 o;
      o.x = q4(bf_lo(h0.x), bf_hi(h0.x), bf_lo(h0.y), bf_hi(h0.y), rs);
      o.y = q4(bf_lo(h0.z), bf_hi(h0.z), bf_lo(h0.w), bf_hi(h0.w), rs);
      o.z = q4(bf_lo(h1.x), bf_hi(h1.x), bf_lo(h1.y), bf_hi(h1.y), rs);
      o.w = q4(bf_lo(h1.z), bf_hi(h1.z), bf_lo(h1.w), bf_hi(h1.w), rs);
      store_sys(optr, failed ? poison : o);
    }
    if (remote) {
      if (!sent && lane == 0) __hip_atomic_store(efp, 2u * call, RLX_SYS);
      asm volatile("s_waitcnt vmcnt(0)" ::: "memory");
      if (lane == 0) __hip_atomic_store(dnp, call, RLX_SYS);
    }
  } else {
    {
      Loader L;
      L.l = c;
      L.rW = __builtin_amdgcn_make_buffer_rsrc((void*)p.w, 0, K * N * 4, 0x00020000);
#pragma unroll
      for (int i = 0; i < 8; i++) L.vW[i] = ((lane >> 4) * 8 + i) * (N * 4) + (lane & 15) * 16;
      L.sW = (unsigned)(kq * 1024 + c * 32) * (N * 4u) + (unsigned)nb * 512u;
      L.own = (unsigned)mb2 * 256u;
      L.ldsW = lds0 + lane * 16;
      L.fl = fl;
      L.afl = afl;
      L.tmo = tmo;
      run_all(L, std::make_integer_sequence<int, NOPS>{});
    }
    const unsigned call = lds_get(fl + F_CALL * 4);
    const unsigned par = call & 1u;
    const u64 t0 = now();
    if (!remote && idx == 0) {
      GAS u64* g = (GAS u64*)(ws + WS_LOC) + (c * 64 + lane) * 16;
      const int srcid = rank;
      GAS u64* mb = (GAS u64*)(p.ws[lane >> 3] + WS_MBOX) + ((par * 8 + (lane & 7)) * 8 + srcid) * 16;
      asm volatile("" : "+v"(mb));
      u64 v;
      while (true) {
        v = __hip_atomic_load(g, RLX_SYS);
        const unsigned tv = __hip_atomic_load(tmo, RLX_AGENT);
        if (__all((unsigned)(v >> 32) == call) || expired(tmo, tv, call, t0)) break;
      }
      lds_put(fl + (F_LMAX + c) * 4, wave_max((unsigned)v));
      if (lds_count(fl + F_LCNT * 4) == 3u) {
        const u32x4 m4 = lds_get4(fl + F_LMAX * 4);
        const unsigned mx = max(max(m4.x, m4.y), max(m4.z, m4.w));
        __hip_atomic_store(mb, ((u64)call << 32) | (u64)mx, RLX_SYS);
      }
    }
    if (c == 1) {
      while (lds_get(fl + F_GS * 4) == 0u) __builtin_amdgcn_s_sleep(2);
      GAS u64* g = (GAS u64*)(ws + WS_MBOX) + ((par * 8 + xcd) * 8 + (lane & 7)) * 16;
      u64 v;
      bool bad;
      while (true) {
        v = __hip_atomic_load(g, RLX_SYS);
        const unsigned tv = __hip_atomic_load(tmo, RLX_AGENT);
        bad = expired(tmo, tv, call, t0);
        if (bad || __all((unsigned)(v >> 32) == call)) break;
      }
      const unsigned mx = wave_max((unsigned)v);
      const float amax = __builtin_bit_cast(float, mx);
      const float rs = amax > 0.f ? 127.0f / amax : 0.f;
      const u32x2 gw = {bad ? 2u : 1u, __builtin_bit_cast(unsigned, rs)};
      asm volatile("ds_write_b64 %0, %1" ::"v"(fl + F_GOK * 4), "v"(gw) : "memory");
    }
    if (remote) {
      GAS unsigned* ef = (GAS unsigned*)(ws + WS_EF) + ((par * 128 + slot) * 8 + dst) * 32;
      while (lds_get(fl + F_GS * 4) == 0u) __builtin_amdgcn_s_sleep(4);
      unsigned e;
      bool bad = false;
      while ((e = UNI(__hip_atomic_load(ef, RLX_SYS))) < 2u * call) {
        const unsigned ok = lds_get(fl + F_GOK * 4);
        if (ok == 2u || now() - t0 > DEADLINE) { __hip_atomic_store(tmo, call, RLX_AGENT); bad = true; break; }
        if (ok == 0u) __builtin_amdgcn_s_sleep(12);
      }
      if (bad || (e & 1u)) {
        __builtin_amdgcn_fence(__ATOMIC_ACQUIRE, "wavefront");
        const __amdgpu_buffer_rsrc_t rI = __builtin_amdgcn_make_buffer_rsrc(
            (void*)(ws + WS_E + (size_t)((par * 8 + dst) * 128 + slot) * 2048), 0, bad ? 0 : 2048, 0x00020000);
        const u32x4 h0 = __builtin_amdgcn_raw_buffer_load_b128(rI, lane * 16, 0, AUX_SYS);
        const u32x4 h1 = __builtin_amdgcn_raw_buffer_load_b128(rI, lane * 16, 1024, AUX_SYS);
        u32x2 gv;
        while (gv = lds_get2(fl + F_GOK * 4), UNI(gv.x) == 0u) __builtin_amdgcn_s_sleep(1);
        const unsigned rsb = gv.y;
        const float rs = __builtin_bit_cast(float, rsb);
        u32x4 o;
        o.x = q4(bf_lo(h0.x), bf_hi(h0.x), bf_lo(h0.y), bf_hi(h0.y), rs);
        o.y = q4(bf_lo(h0.z), bf_hi(h0.z), bf_lo(h0.w), bf_hi(h0.w), rs);
        o.z = q4(bf_lo(h1.x), bf_hi(h1.x), bf_lo(h1.y), bf_hi(h1.y), rs);
        o.w = q4(bf_lo(h1.z), bf_hi(h1.z), bf_lo(h1.w), bf_hi(h1.w), rs);
        if (bad || gv.x != 1u) o = u32x4{0x80808080u, 0x80808080u, 0x80808080u, 0x80808080u};
        store_sys(p.out[rank] + (size_t)(dst * M + rowin) * 512 + colb, o);
      }
    } else {
      while (lds_get(fl + F_GOK * 4) == 0u) __builtin_amdgcn_s_sleep(2);
      GAS unsigned* d = (GAS unsigned*)(ws + WS_DONE) + (slot * 8 + (lane & 7)) * 32;
      bool bad = false;
      while (true) {
        const unsigned v = __hip_atomic_load(d, RLX_SYS);
        const unsigned tv = __hip_atomic_load(tmo, RLX_AGENT);
        if (__all(v >= call || (lane & 7) == rank)) break;
        if (expired(tmo, tv, call, t0)) { bad = true; break; }
      }
      if (bad && lane == 0) p.out[rank][(size_t)(rank * M + rowin) * 512 + colb] = 0x80;
    }
  }
}

extern "C" void kernel_launch(int rank, int nranks, ncclComm_t comm, hipStream_t stream,
                              void* const* d_in, const int* in_sizes, int n_in,
                              void* d_out, int out_size, const int* in_dtypes, int out_dtype,
                              void* const* d_out_all, void* const* d_ws_all, size_t ws_bytes) {
  if (nranks != NR || rank < 0 || rank >= NR || n_in < 2 || ws_bytes < WS_NEED || in_sizes[0] != M * K ||
      in_sizes[1] != K * N || out_size != NR * M * (N / NR)) {
    fprintf(stderr, "unexpected problem shape\n");
    return;
  }
  static bool init = false;
  if (!init) {
    HIP_CHECK(hipFuncSetAttribute((const void*)fused_gemm_a2a, hipFuncAttributeMaxDynamicSharedMemorySize, LDS_BYTES));
    init = true;
  }
  Params p;
  p.x = (const float*)d_in[0];
  p.w = (const float*)d_in[1];
  for (int i = 0; i < NR; i++) { p.out[i] = (unsigned char*)d_out_all[i]; p.ws[i] = (unsigned char*)d_ws_all[i]; }
  p.self = (unsigned char*)d_ws_all[rank];
  p.rank = rank;
  p.pad = 0;
  hipLaunchKernelGGL(fused_gemm_a2a, dim3(NWG), dim3(512), LDS_BYTES, stream, p);
  HIP_CHECK(hipGetLastError());
}
```

```cpp
#include <hip/hip_runtime.h>
#include <rccl/rccl.h>
#include <cstdio>
#include <utility>

#define HIP_CHECK(expr) do { hipError_t _e = (expr); if (_e != hipSuccess) { \
    fprintf(stderr, "HIP error %s at %s:%d\n", hipGetErrorString(_e), __FILE__, __LINE__); \
    return; } } while (0)

typedef unsigned u32x4 __attribute__((ext_vector_type(4)));
typedef unsigned u32x2 __attribute__((ext_vector_type(2)));
typedef float f32x4 __attribute__((ext_vector_type(4)));
typedef float f32x2 __attribute__((ext_vector_type(2)));
typedef __bf16 bf16x2 __attribute__((ext_vector_type(2)));
typedef unsigned long long u64;
#define GAS __attribute__((address_space(1)))
#define LAS __attribute__((address_space(3)))
#define DI __device__ __forceinline__
#pragma clang diagnostic ignored "-Wint-to-pointer-cast"

constexpr int NR = 8;
constexpr int M = 512, N = 4096, K = 4096;
constexpr int NWG = 256;
constexpr int NSTEP = 32;
constexpr int WS_TMO = 0;
constexpr int WS_CNT = 4096;
constexpr int WS_LOC = 8192;
constexpr int WS_AFLAG = 64 * 1024;
constexpr int WS_MBOX = 96 * 1024;
constexpr int WS_BOOK = 120 * 1024;
constexpr int WS_DONE = 128 * 1024;
constexpr int WS_EF = 256 * 1024;
constexpr int WS_XFLAG = 576 * 1024;
constexpr int WS_AT = 1 << 20;
constexpr int WS_X = 5 << 20;
constexpr int X_BYTES = NWG * 3 * 4 * 4096;
constexpr int WS_E = 18 << 20;
constexpr size_t WS_NEED = (size_t)WS_E + 2 * 8 * 128 * 2048;
constexpr u64 DEADLINE = 40000000ull;
constexpr int NWS = 19;
constexpr int L_F = NWS * 8192;
constexpr int L_SC = L_F - 16384;
constexpr int LDS_BYTES = L_F + 256;
static_assert(LDS_BYTES <= 160 * 1024, "LDS");
enum : int {
  F_FULLW = 0,
  F_DONE = 4,
  F_RDY = 32,
  F_MAX = 12,
  F_LMAX = 16,
  F_GOK = 20,
  F_CALL = 22, F_ST, F_DR, F_FS, F_CNT, F_GS, F_LCNT, F_TS
};
constexpr int AUX_SYS = 17;
constexpr unsigned EGAP = 5;

struct Params {
  const float* x;
  const float* w;
  unsigned char* self;
  int rank;
  int pad;
  unsigned char* out[NR];
  unsigned char* ws[NR];
};

#define RLX_AGENT __ATOMIC_RELAXED, __HIP_MEMORY_SCOPE_AGENT
#define RLX_SYS __ATOMIC_RELAXED, __HIP_MEMORY_SCOPE_SYSTEM
#define CFENCE() asm volatile("" ::: "memory")
#define SB() __builtin_amdgcn_sched_barrier(0)
#define UNI(x) __builtin_amdgcn_readfirstlane(x)

DI unsigned pk(float a, float b) {
  f32x2 v = {a, b};
  bf16x2 r = __builtin_convertvector(v, bf16x2);
  return __builtin_bit_cast(unsigned, r);
}
DI float bf_lo(unsigned v) { return __builtin_bit_cast(float, v << 16); }
DI float bf_hi(unsigned v) { return __builtin_bit_cast(float, v & 0xffff0000u); }
DI unsigned q4(float a, float b, float c, float d, float rs) {
  const unsigned t0 = __builtin_bit_cast(unsigned, __builtin_fmaf(a, rs, 12582912.0f));
  const unsigned t1 = __builtin_bit_cast(unsigned, __builtin_fmaf(b, rs, 12582912.0f));
  const unsigned t2 = __builtin_bit_cast(unsigned, __builtin_fmaf(c, rs, 12582912.0f));
  const unsigned t3 = __builtin_bit_cast(unsigned, __builtin_fmaf(d, rs, 12582912.0f));
  return __builtin_amdgcn_perm(t1, t0, 0x0c0c0400u) | __builtin_amdgcn_perm(t3, t2, 0x04000c0cu);
}
DI u64 now() { return __builtin_amdgcn_s_memrealtime(); }
DI unsigned wave_max(unsigned v) {
  v = max(v, (unsigned)__builtin_amdgcn_update_dpp(0, (int)v, 0x111, 0xf, 0xf, true));
  v = max(v, (unsigned)__builtin_amdgcn_update_dpp(0, (int)v, 0x112, 0xf, 0xf, true));
  v = max(v, (unsigned)__builtin_amdgcn_update_dpp(0, (int)v, 0x114, 0xf, 0xf, true));
  v = max(v, (unsigned)__builtin_amdgcn_update_dpp(0, (int)v, 0x118, 0xf, 0xf, true));
  const unsigned a = __builtin_amdgcn_readlane(v, 15), b = __builtin_amdgcn_readlane(v, 31);
  const unsigned c = __builtin_amdgcn_readlane(v, 47), d = __builtin_amdgcn_readlane(v, 63);
  return max(max(a, b), max(c, d));
}
DI bool expired(GAS unsigned* tmo, unsigned tv, unsigned call, u64 t0) {
  if (UNI(tv) == call) return true;
  if (now() - t0 > DEADLINE) { __hip_atomic_store(tmo, call, RLX_AGENT); return true; }
  return false;
}
DI void store_sys(const void* p, u32x4 v) {
  asm volatile("global_store_dwordx4 %0, %1, off sc0 sc1\n\ts_nop 1" ::"v"(p), "v"(v) : "memory");
}
DI void lds_put(unsigned addr, unsigned v) { asm volatile("ds_write_b32 %0, %1" ::"v"(addr), "v"(v) : "memory"); }
DI unsigned lds_get(unsigned addr) {
  unsigned v;
  asm volatile("ds_read_b32 %0, %1\n\ts_waitcnt lgkmcnt(0)" : "=v"(v) : "v"(addr) : "memory");
  return UNI(v);
}
DI u32x2 lds_get2(unsigned addr) {
  u32x2 v;
  asm volatile("ds_read_b64 %0, %1\n\ts_waitcnt lgkmcnt(0)" : "=v"(v) : "v"(addr) : "memory");
  return v;
}
DI u32x4 lds_get4(unsigned addr) {
  u32x4 v;
  asm volatile("ds_read_b128 %0, %1\n\ts_waitcnt lgkmcnt(0)" : "=v"(v) : "v"(addr) : "memory");
  return v;
}
DI unsigned lds_count(unsigned cnt) {
  unsigned old;
  asm volatile("s_mov_b64 exec, 1\n\tds_add_rtn_u32 %0, %1, %2\n\ts_mov_b64 exec, -1\n\ts_waitcnt lgkmcnt(0)"
               : "=&v"(old) : "v"(cnt), "v"(1u) : "memory");
  return UNI(old);
}
DI void wait8(const GAS unsigned* f8, unsigned call, GAS unsigned* tmo) {
  const u64 t0 = now();
  for (unsigned it = 1;; it++) {
    unsigned a0, a1, a2, a3, a4, a5, a6, a7;
    asm volatile(
        "s_load_dword %0, %8, 0x0 glc\n\ts_load_dword %1, %8, 0x80 glc\n\ts_load_dword %2, %8, 0x100 glc\n\t"
        "s_load_dword %3, %8, 0x180 glc\n\ts_load_dword %4, %8, 0x200 glc\n\ts_load_dword %5, %8, 0x280 glc\n\t"
        "s_load_dword %6, %8, 0x300 glc\n\ts_load_dword %7, %8, 0x380 glc\n\ts_waitcnt lgkmcnt(0)"
        : "=&s"(a0), "=&s"(a1), "=&s"(a2), "=&s"(a3), "=&s"(a4), "=&s"(a5), "=&s"(a6), "=&s"(a7)
        : "s"(f8) : "memory");
    if (min(min(min(a0, a1), min(a2, a3)), min(min(a4, a5), min(a6, a7))) >= call) break;
    if ((it & 255u) == 0u) {
      unsigned tv;
      asm volatile("global_load_dword %0, %1, off sc1\n\ts_waitcnt vmcnt(0)" : "=&v"(tv) : "v"(tmo) : "memory");
      if (UNI(tv) == call) break;
      if (now() - t0 > DEADLINE) {
        asm volatile("global_store_dword %0, %1, off sc1\n\ts_waitcnt vmcnt(0)" ::"v"(tmo), "v"(call) : "memory");
        break;
      }
    }
    __builtin_amdgcn_s_sleep(2);
  }
}

enum : int { O_, P_, CW_, G_, WS_, WF_, Q_ };
struct Op { int k, a; };
#define RND(r) {G_, 4 * (r) - 6}, {P_, r}, {O_, (r) + 1}, {CW_, r}
constexpr Op SCHED[] = {
  {O_, 0}, {WS_, 0}, {P_, 0}, {WF_, 0}, {O_, 1}, {CW_, 0}, {P_, 1}, {O_, 2}, {CW_, 1}, {Q_, 0},
  RND(2), RND(3), RND(4), RND(5), RND(6), {G_, 22}, {P_, 7}, {CW_, 7},
};
constexpr int NOPS = sizeof(SCHED) / sizeof(Op);
constexpr int NSET = 5;
constexpr bool is_ld(int k) { return k == O_ || k == P_; }
constexpr int find_op(int k, int a) { for (int i = 0; i < NOPS; i++) if (SCHED[i].k == k && SCHED[i].a == a) return i; return -1; }
constexpr int younger(int t, int i) { int n = 0; for (int j = t + 1; j < i; j++) n += is_ld(SCHED[j].k) ? 8 : 0; return n; }
constexpr int set_of(int op) {
  int owner[NSET] = {-1, -1, -1, -1, -1};
  for (int i = 0; i < NOPS; i++) {
    if (is_ld(SCHED[i].k)) {
      int s = -1;
      for (int q = 0; q < NSET; q++) if (owner[q] < 0) { s = q; break; }
      if (s < 0) return -1;
      owner[s] = SCHED[i].a;
      if (i == op) return s;
    } else if (SCHED[i].k == CW_) {
      for (int q = 0; q < NSET; q++) if (owner[q] == SCHED[i].a) owner[q] = -1;
    }
  }
  return -1;
}
constexpr bool can_reach(int n, int i) { return n <= 0 || find_op(CW_, n / 4) < i; }
constexpr bool sched_ok() {
  for (int r = 0; r < 8; r++) {
    const int o = find_op(O_, r), p = find_op(P_, r), c = find_op(CW_, r);
    if (o < 0 || p < o || c < p || set_of(o) < 0 || set_of(p) < 0 || younger(o, c) > 63) return false;
    if (r > 0 && c < find_op(CW_, r - 1)) return false;
    if (!can_reach(4 * r + 3 - NWS + 1, c)) return false;
    if (4 * r + 3 >= NWS - 2 && c < find_op(WS_, 0)) return false;
  }
  for (int i = 0; i < NOPS; i++) if (SCHED[i].k == G_ && (!can_reach(SCHED[i].a, i) || i < find_op(Q_, 0))) return false;
  return true;
}
static_assert(sched_ok(), "loader schedule");

struct Loader {
  f32x4 wb[NSET][8];
  __amdgpu_buffer_rsrc_t rW;
  int vW[8];
  unsigned sW, own, ldsW, l, fl;
  const GAS unsigned* afl;
  GAS unsigned* tmo;
};
#define WAIT_SET(S, N)                                                                               \
  asm volatile("s_waitcnt vmcnt(%8)"                                                                 \
               : "+v"(L.wb[S][0]), "+v"(L.wb[S][1]), "+v"(L.wb[S][2]), "+v"(L.wb[S][3]),             \
                 "+v"(L.wb[S][4]), "+v"(L.wb[S][5]), "+v"(L.wb[S][6]), "+v"(L.wb[S][7])              \
               : "i"(N) : "memory")
template <int I> DI void run_op(Loader& L) {
  constexpr int k = SCHED[I].k, a = SCHED[I].a;
  if constexpr (is_ld(k)) {
    constexpr int s = set_of(I);
    const unsigned so = (unsigned)a * (128u * N * 4u) + L.sW + ((k == O_) ? L.own : (256u - L.own));
    asm volatile(
        "buffer_load_dwordx4 %0, %8, %16, %17 offen\n\t"
        "buffer_load_dwordx4 %1, %9, %16, %17 offen\n\t"
        "buffer_load_dwordx4 %2, %10, %16, %17 offen\n\t"
        "buffer_load_dwordx4 %3, %11, %16, %17 offen\n\t"
        "buffer_load_dwordx4 %4, %12, %16, %17 offen\n\t"
        "buffer_load_dwordx4 %5, %13, %16, %17 offen\n\t"
        "buffer_load_dwordx4 %6, %14, %16, %17 offen\n\t"
        "buffer_load_dwordx4 %7, %15, %16, %17 offen"
        : "=&v"(L.wb[s][0]), "=&v"(L.wb[s][1]), "=&v"(L.wb[s][2]), "=&v"(L.wb[s][3]),
          "=&v"(L.wb[s][4]), "=&v"(L.wb[s][5]), "=&v"(L.wb[s][6]), "=&v"(L.wb[s][7])
        : "v"(L.vW[0]), "v"(L.vW[1]), "v"(L.vW[2]), "v"(L.vW[3]), "v"(L.vW[4]), "v"(L.vW[5]), "v"(L.vW[6]), "v"(L.vW[7]),
          "s"(L.rW), "s"(so)
        : "memory");
  } else if constexpr (k == CW_) {
    constexpr int io = find_op(O_, a), ip = find_op(P_, a), so = set_of(io), sp = set_of(ip);
    const unsigned t = 4u * a + L.l;
    if constexpr (4 * a + 3 >= NWS) {
      const int need = (int)t - NWS + 1;
      while (true) {
        const u32x4 d = lds_get4(L.fl + F_DONE * 4);
        if ((int)UNI(min(min(d.x, d.y), min(d.z, d.w))) >= need) break;
        __builtin_amdgcn_s_sleep(1);
      }
    }
    unsigned slot = (4u * a) % NWS + L.l;
    if (slot >= (unsigned)NWS) slot -= NWS;
    const unsigned base = L.ldsW + slot * 8192u;
    auto conv = [&](f32x4(&w)[8], unsigned dst) {
#pragma unroll
      for (int c = 0; c < 4; c++) {
        u32x4 bf;
        bf.x = pk(w[0][c], w[1][c]); bf.y = pk(w[2][c], w[3][c]);
        bf.z = pk(w[4][c], w[5][c]); bf.w = pk(w[6][c], w[7][c]);
        *(LAS u32x4*)(dst + c * 1024) = bf;
      }
    };
    WAIT_SET(so, younger(io, I));
    conv(L.wb[so], base + L.own * 16u);
    WAIT_SET(sp, younger(ip, I));
    conv(L.wb[sp], base + (4096u - L.own * 16u));
    lds_put(L.fl + (F_FULLW + L.l) * 4, t + 1u);
  } else if constexpr (k == G_) {
    while (lds_get(L.fl + (F_DONE + L.l) * 4) < (unsigned)a) __builtin_amdgcn_s_sleep(1);
  } else if constexpr (k == WS_) {
    while (lds_get(L.fl + F_ST * 4) < 4u) __builtin_amdgcn_s_sleep(1);
  } else if constexpr (k == WF_) {
    while (lds_get(L.fl + F_FS * 4) == 0u) __builtin_amdgcn_s_sleep(1);
  } else if (L.l != 0u) {
    wait8(L.afl + L.l * 8u * 32u, lds_get(L.fl + F_CALL * 4), L.tmo);
    lds_put(L.fl + (F_RDY + L.l) * 4, 1u);
  }
}
template <int... I> DI void run_all(Loader& L, std::integer_sequence<int, I...>) { (run_op<I>(L), ...); }


__global__ void __launch_bounds__(512, 2) fused_gemm_a2a(Params p) {
  extern __shared__ __attribute__((aligned(16))) unsigned char smem[];
  const int tid = threadIdx.x, lane = tid & 63;
  const int w = UNI(tid >> 6);
  const int b = blockIdx.x;
  const int xcd = b & 7, j = b >> 3;
  const int kq = xcd >> 1, mb2 = j & 1;
  const int nb = (xcd & 1) * 16 + (j >> 1);
  const int grp = mb2 * 4 + kq;
  const unsigned lds0 = (unsigned)(size_t)(LAS unsigned char*)smem;
  const unsigned fl = lds0 + L_F;
  f32x4 xa[8];
  unsigned cv = 0, tw0 = 0, tw1 = 0;
  if (w < 4) {
    const __amdgpu_buffer_rsrc_t rX = __builtin_amdgcn_make_buffer_rsrc((void*)p.x, 0, M * K * 4, 0x00020000);
    const int vX = (lane >> 3) * (K * 4) + (lane & 7) * 16;
    const int so = (mb2 * 256 + w * 64) * (K * 4) + (kq * 1024 + nb * 32) * 4;
#pragma unroll
    for (int i = 0; i < 8; i++)
      xa[i] = __builtin_bit_cast(f32x4, __builtin_amdgcn_raw_buffer_load_b128(rX, vX, so + i * 8 * (K * 4), 0));
    SB();
    cv = __hip_atomic_load((GAS unsigned*)(p.self + WS_CNT) + b * 4 + w, RLX_AGENT);
    tw0 = __hip_atomic_load((GAS unsigned*)(p.self + WS_AT + ((grp * NSTEP + nb) * 4 + w) * 4096), RLX_AGENT);
    tw1 = __hip_atomic_load((GAS unsigned*)(p.self + WS_AFLAG) + (grp * NSTEP + nb) * 32, RLX_AGENT);
    if (tid < 64) *(LAS unsigned*)(fl + tid * 4) = 0u;
    asm volatile("s_waitcnt lgkmcnt(0)" ::: "memory");
    SB();
  }
  __builtin_amdgcn_s_barrier();
  CFENCE();
  SB();
  const int rank = p.rank;
  unsigned char* ws = p.self;
  GAS unsigned* tmo = (GAS unsigned*)(ws + WS_TMO);
  GAS unsigned* afl = (GAS unsigned*)(ws + WS_AFLAG) + grp * NSTEP * 32;
  const int c = w & 3;
  const unsigned dst = (unsigned)(nb >> 2) & 7u;
  const int idx = ((nb & 3) * 2 + mb2) * 4 + kq;
  const int slot = idx * 4 + c;
  const bool remote = dst != (unsigned)rank;
  const unsigned me = (unsigned)rank;
  const int rowin = mb2 * 256 + c * 64 + kq * 16 + (lane & 15), colb = (nb & 3) * 128 + 16 * (lane >> 4);

  if (__builtin_expect(w < 4, 1)) {
    const unsigned call = UNI(cv) + 1u;
    {
      const unsigned sc = lds0 + L_SC + c * 4096;
      const unsigned tb = sc + ((lane & 7) >> 1) * 256 + (lane >> 3) * 16 + (lane & 1) * 8;
#pragma unroll
      for (int i = 0; i < 8; i++) {
        u32x2 o = {pk(xa[i][0], xa[i][1]), pk(xa[i][2], xa[i][3])};
        *(LAS u32x2*)(tb + (((unsigned)(i >> 1)) ^ (unsigned)kq) * 1024u + (i & 1) * 128) = o;
      }
      asm volatile("" ::"v"(tw0), "v"(tw1));
      u32x4 fr[4];
#pragma unroll
      for (int f = 0; f < 4; f++) fr[f] = *(LAS u32x4*)(sc + f * 1024 + lane * 16);
      unsigned char* pub = ws + WS_AT + ((grp * NSTEP + nb) * 4 + c) * 4096 + lane * 16;
#pragma unroll
      for (int f = 0; f < 4; f++) store_sys(pub + f * 1024, fr[f]);
      lds_put(fl + F_CALL * 4, call);
      lds_count(fl + F_ST * 4);
      asm volatile("s_waitcnt vmcnt(0)" ::: "memory");
      if (lds_count(fl + F_DR * 4) == 3u) {
        if (lane == 0) __hip_atomic_store(afl + nb * 32, call, RLX_SYS);
        lds_put(fl + F_FS * 4, 1u);
      }
      if (lane == 0) __hip_atomic_store((GAS unsigned*)(ws + WS_CNT) + b * 4 + c, call, RLX_AGENT);
      if (c == 0) {
        while (lds_get(fl + F_FS * 4) == 0u) {}
        wait8(afl, call, tmo);
        lds_put(fl + (F_RDY + 4) * 4, 1u);
        lds_put(fl + F_RDY * 4, 1u);
      }
    }
    f32x4 acc[4][8];
#pragma unroll
    for (int m = 0; m < 4; m++)
#pragma unroll
      for (int n = 0; n < 8; n++) { acc[m][n] = f32x4{0, 0, 0, 0}; asm volatile("" : "+v"(acc[m][n])); }
    const unsigned wBase = lds0 + lane * 16;
    unsigned pfw = fl + F_FULLW * 4, pdone = fl + (F_DONE + c) * 4, vdone = 0;
    asm volatile("" : "+v"(pfw), "+v"(pdone), "+v"(vdone));
    const __amdgpu_buffer_rsrc_t rA = __builtin_amdgcn_make_buffer_rsrc(
        (void*)(ws + WS_AT + (grp * NSTEP * 4 + c) * 4096), 0, (NSTEP - 1) * 16384 + 4096, 0x00020000);
    int vA[4];
#pragma unroll
    for (int m = 0; m < 4; m++) { vA[m] = lane * 16 + m * 1024; asm volatile("" : "+v"(vA[m])); }
    auto ldA = [&](int m, unsigned t) { return __builtin_amdgcn_raw_buffer_load_b128(rA, vA[m], t * 16384u, AUX_SYS); };
    u32x4 A[5][4], B[4];
#define SETOF(t) ((((t) & 7) == 5) ? 4 : ((t) & 3))
    unsigned fw;
    while (lds_get(fl + F_RDY * 4) == 0u) __builtin_amdgcn_s_sleep(1);
#pragma unroll
    for (int d = 0; d < 2; d++)
#pragma unroll
      for (int m = 0; m < 4; m++) { A[d][m] = ldA(m, d); SB(); }
    while (lds_get(pfw) < 1u) {}
    __builtin_amdgcn_s_setprio(3);
    B[0] = *(LAS u32x4*)(wBase);
    B[1] = *(LAS u32x4*)(wBase + 1024);
    B[2] = *(LAS u32x4*)(wBase + 2048);
    CFENCE(); fw = *(LAS unsigned*)(pfw + 4); CFENCE();
    unsigned woff = 0, wc = wBase, rdy = 0, prdy = fl + (F_RDY + 1) * 4;
    asm volatile("" : "+v"(prdy));
    auto mf = [&](int m, int n, int q) {
      asm volatile("v_mfma_f32_16x16x32_bf16 %0, %1, %2, %0" : "+v"(acc[m][n]) : "v"(B[n & 3]), "v"(A[q][m]));
    };
    auto step = [&](unsigned s, int q8) {
      const int q = q8 & 3;
      unsigned wn = 0, sf = 0, nd = 0;
#pragma unroll
      for (int n = 0; n < 8; n++) {
        mf(0, n, SETOF(q8)); SB();
        if (n == 0 && q8 == 5) {
          if (__builtin_expect(UNI(rdy) == 0u, 0)) while (lds_get(prdy) == 0u) {}
          prdy += 4u;
          CFENCE();
        }
        if (q == 1 && n < 4) { A[SETOF(q8 + 3)][n] = ldA(n, s + 3u); SB(); }
        if (n < 5) B[(n + 3) & 3] = *(LAS u32x4*)(wc + (n + 3) * 1024);
        else B[(n + 3) & 3] = *(LAS u32x4*)(wn + (n - 5) * 1024);
        SB(); mf(1, n, SETOF(q8)); SB();
        if (n == 1) { woff = (woff == (NWS - 1) * 8192u) ? 0u : woff + 8192u; wn = wBase + woff; asm volatile("" : "+v"(wn)); }
        if (n == 2) sf = UNI(fw);
        if (n == 4) lds_put(pdone, vdone);
        if (n == 7) { CFENCE(); fw = *(LAS unsigned*)(pfw + ((q + 2) & 3) * 4); CFENCE(); }
        SB(); mf(2, n, SETOF(q8)); SB();
        if (q == 0) A[SETOF(q8 + 2 + (n >> 2))][n & 3] = ldA(n & 3, s + 2u + (n >> 2));
        if (q == 1 && n < 4) A[SETOF(q8 + 4)][n] = ldA(n, s + 4u);
        if (n == 1) { vdone += 1; asm volatile("" : "+v"(vdone)); }
        if (n == 3) { nd = (q8 == 7 && s == NSTEP - 1u) ? 0u : s + 2u; asm volatile("" : "+s"(nd)); }
        if (n == 5 && q8 == 4) { CFENCE(); rdy = *(LAS unsigned*)(prdy); CFENCE(); }
        SB(); mf(3, n, SETOF(q8)); SB();
        if (n == 3) {
          if (__builtin_expect(sf < nd, 0))
            while (lds_get(pfw + ((q + 1) & 3) * 4) < nd) {}
          CFENCE();
          SB();
        }
      }
      wc = wn;
    };
    asm volatile("s_nop 7");
    for (unsigned s = 0; s < (unsigned)NSTEP; s += 8) {
      step(s, 0); step(s + 1, 1); step(s + 2, 2); step(s + 3, 3);
      step(s + 4, 4); step(s + 5, 5); step(s + 6, 6); step(s + 7, 7);
    }
    asm volatile("s_nop 15\n\ts_nop 15");
    SB();
    __builtin_amdgcn_s_setprio(0);
    const __amdgpu_buffer_rsrc_t rE = __builtin_amdgcn_make_buffer_rsrc((void*)(ws + WS_X), 0, X_BYTES, 0x00020000);
#pragma unroll
    for (int m = 1; m < 4; m++) {
      const int so = (((b ^ (m << 1)) * 3 + (m - 1)) * 4 + c) * 4096;
#pragma unroll
      for (int t = 0; t < 4; t++) {
        u32x4 o;
        o.x = pk(acc[m][2 * t][0], acc[m][2 * t][1]); o.y = pk(acc[m][2 * t][2], acc[m][2 * t][3]);
        o.z = pk(acc[m][2 * t + 1][0], acc[m][2 * t + 1][1]); o.w = pk(acc[m][2 * t + 1][2], acc[m][2 * t + 1][3]);
        __builtin_amdgcn_raw_buffer_store_b128(o, rE, lane * 16, so + t * 1024, AUX_SYS);
      }
    }
    GAS unsigned* xfl = (GAS unsigned*)(ws + WS_XFLAG);
    asm volatile("s_waitcnt vmcnt(0)" ::: "memory");
    if (lane < 3) __hip_atomic_store(xfl + (((b ^ ((lane + 1) << 1)) * 4 + c) * 3 + lane) * 32, call, RLX_SYS);
    {
      const u64 t0 = now();
      while (true) {
        const unsigned v = __hip_atomic_load(xfl + ((b * 4 + c) * 3 + min(lane, 2)) * 32, RLX_SYS);
        const unsigned tv = __hip_atomic_load(tmo, RLX_AGENT);
        if (__all(v >= call) || expired(tmo, tv, call, t0)) break;
      }
      __builtin_amdgcn_fence(__ATOMIC_ACQUIRE, "wavefront");
    }
    u32x4 in[3][4];
#pragma unroll
    for (int m = 0; m < 3; m++)
#pragma unroll
      for (int t = 0; t < 4; t++)
        in[m][t] = __builtin_amdgcn_raw_buffer_load_b128(rE, lane * 16, ((b * 3 + m) * 4 + c) * 4096 + t * 1024, AUX_SYS);
    float am = 0.f;
#pragma unroll
    for (int t = 0; t < 4; t++) {
#pragma unroll
      for (int m = 0; m < 3; m++)
#pragma unroll
        for (int e = 0; e < 4; e++) {
          const unsigned v = in[m][t][e];
          acc[0][2 * t + (e >> 1)][(e & 1) * 2] += bf_lo(v);
          acc[0][2 * t + (e >> 1)][(e & 1) * 2 + 1] += bf_hi(v);
        }
#pragma unroll
      for (int e = 0; e < 8; e++) am = fmaxf(am, fabsf(acc[0][2 * t + (e >> 2)][e & 3]));
    }
    lds_put(fl + (F_MAX + c) * 4, wave_max(__builtin_bit_cast(unsigned, am)));
    if (lds_count(fl + F_CNT * 4) == 3u) {
      const u32x4 m4 = lds_get4(fl + F_MAX * 4);
      const unsigned mx = max(max(m4.x, m4.y), max(m4.z, m4.w));
      if (lane == 0) __hip_atomic_store((GAS u64*)(ws + WS_LOC) + b * 16, ((u64)call << 32) | (u64)mx, RLX_SYS);
      lds_put(fl + F_GS * 4, 1u);
      if (remote) {
        GAS u64* bk = (GAS u64*)(ws + WS_BOOK) + dst * 16;
        u64 tb = 0;
        if (lane == 0) {
          __hip_atomic_fetch_max(bk, now(), RLX_AGENT);
          tb = __hip_atomic_fetch_add(bk, (u64)(4u * EGAP), RLX_AGENT);
        }
        lds_put(fl + F_TS * 4, UNI((unsigned)tb) | 1u);
      }
    }
    const unsigned par = call & 1u;
    u32x4 h0, h1;
    h0.x = pk(acc[0][0][0], acc[0][1][0]); h0.y = pk(acc[0][2][0], acc[0][3][0]);
    h0.z = pk(acc[0][0][1], acc[0][1][1]); h0.w = pk(acc[0][2][1], acc[0][3][1]);
    h1.x = pk(acc[0][0][2], acc[0][1][2]); h1.y = pk(acc[0][2][2], acc[0][3][2]);
    h1.z = pk(acc[0][0][3], acc[0][1][3]); h1.w = pk(acc[0][2][3], acc[0][3][3]);
    unsigned char* optr = p.out[dst] + (size_t)(rank * M + rowin) * 512 + colb;
    GAS unsigned* dnp = (GAS unsigned*)(p.ws[dst] + WS_DONE) + (slot * 8 + me) * 32;
    GAS unsigned* efp = (GAS unsigned*)(p.ws[dst] + WS_EF) + ((par * 128 + slot) * 8 + me) * 32;
    asm volatile("" : "+v"(optr), "+v"(dnp), "+v"(efp));
    bool sent = false;
    u32x2 gv;
    {
      unsigned ts = 0;
      while (gv = lds_get2(fl + F_GOK * 4), UNI(gv.x) == 0u) {
        if (remote && ts == 0u) ts = lds_get(fl + F_TS * 4);
        if (ts != 0u && !sent && (int)((unsigned)now() - (ts + (unsigned)c * EGAP)) >= 0) {
          unsigned char* ep = p.ws[dst] + WS_E + (size_t)((par * 8 + me) * 128 + slot) * 2048 + lane * 16;
          store_sys(ep, h0);
          store_sys(ep + 1024, h1);
          asm volatile("s_waitcnt vmcnt(0)" ::: "memory");
          if (lane == 0) __hip_atomic_store(efp, 2u * call + 1u, RLX_SYS);
          sent = true;
        }
      }
    }
    const unsigned rsb = gv.y;
    const float rs = __builtin_bit_cast(float, rsb);
    const bool failed = gv.x != 1u;
    const u32x4 poison = {0x80808080u, 0x80808080u, 0x80808080u, 0x80808080u};
    {
      u32x4 o;
#pragma unroll
      for (int rg = 0; rg < 4; rg++) o[rg] = q4(acc[0][4][rg], acc[0][5][rg], acc[0][6][rg], acc[0][7][rg], rs);
      store_sys(optr + 64, failed ? poison : o);
    }
    if (!sent) {
      u32x4 o;
      o.x = q4(bf_lo(h0.x), bf_hi(h0.x), bf_lo(h0.y), bf_hi(h0.y), rs);
      o.y = q4(bf_lo(h0.z), bf_hi(h0.z), bf_lo(h0.w), bf_hi(h0.w), rs);
      o.z = q4(bf_lo(h1.x), bf_hi(h1.x), bf_lo(h1.y), bf_hi(h1.y), rs);
      o.w = q4(bf_lo(h1.z), bf_hi(h1.z), bf_lo(h1.w), bf_hi(h1.w), rs);
      store_sys(optr, failed ? poison : o);
    }
    if (remote) {
      if (!sent && lane == 0) __hip_atomic_store(efp, 2u * call, RLX_SYS);
      asm volatile("s_waitcnt vmcnt(0)" ::: "memory");
      if (lane == 0) __hip_atomic_store(dnp, call, RLX_SYS);
    }
  } else {
    {
      Loader L;
      L.l = c;
            L.rW = __builtin_amdgcn_make_buffer_rsrc((void*)p.w, 0, K * N * 4, 0x00020000);
#pragma unroll
      for (int i = 0; i < 8; i++) L.vW[i] = ((lane >> 4) * 8 + i) * (N * 4) + (lane & 15) * 16;
      L.sW = (unsigned)(kq * 1024 + c * 32) * (N * 4u) + (unsigned)nb * 512u;
      L.own = (unsigned)mb2 * 256u;
      L.ldsW = lds0 + lane * 16;
      L.fl = fl;
      L.afl = afl;
      L.tmo = tmo;
      run_all(L, std::make_integer_sequence<int, NOPS>{});
    }
    const unsigned call = lds_get(fl + F_CALL * 4);
    const unsigned par = call & 1u;
    const u64 t0 = now();
    if (!remote && idx == 0) {
      GAS u64* g = (GAS u64*)(ws + WS_LOC) + (c * 64 + lane) * 16;
      const int srcid = rank;
      GAS u64* mb = (GAS u64*)(p.ws[lane >> 3] + WS_MBOX) + ((par * 8 + (lane & 7)) * 8 + srcid) * 16;
      asm volatile("" : "+v"(mb));
      u64 v;
      while (true) {
        v = __hip_atomic_load(g, RLX_SYS);
        const unsigned tv = __hip_atomic_load(tmo, RLX_AGENT);
        if (__all((unsigned)(v >> 32) == call) || expired(tmo, tv, call, t0)) break;
      }
      lds_put(fl + (F_LMAX + c) * 4, wave_max((unsigned)v));
      if (lds_count(fl + F_LCNT * 4) == 3u) {
        const u32x4 m4 = lds_get4(fl + F_LMAX * 4);
        const unsigned mx = max(max(m4.x, m4.y), max(m4.z, m4.w));
        __hip_atomic_store(mb, ((u64)call << 32) | (u64)mx, RLX_SYS);
      }
    }
    if (c == 1) {
      while (lds_get(fl + F_GS * 4) == 0u) __builtin_amdgcn_s_sleep(2);
      GAS u64* g = (GAS u64*)(ws + WS_MBOX) + ((par * 8 + xcd) * 8 + (lane & 7)) * 16;
      u64 v;
      bool bad;
      while (true) {
        v = __hip_atomic_load(g, RLX_SYS);
        const unsigned tv = __hip_atomic_load(tmo, RLX_AGENT);
        bad = expired(tmo, tv, call, t0);
        if (bad || __all((unsigned)(v >> 32) == call)) break;
      }
      const unsigned mx = wave_max((unsigned)v);
      const float amax = __builtin_bit_cast(float, mx);
      const float rs = amax > 0.f ? 127.0f / amax : 0.f;
      const u32x2 gw = {bad ? 2u : 1u, __builtin_bit_cast(unsigned, rs)};
      asm volatile("ds_write_b64 %0, %1" ::"v"(fl + F_GOK * 4), "v"(gw) : "memory");
    }
    if (remote) {
      GAS unsigned* ef = (GAS unsigned*)(ws + WS_EF) + ((par * 128 + slot) * 8 + dst) * 32;
      while (lds_get(fl + F_GS * 4) == 0u) __builtin_amdgcn_s_sleep(4);
      unsigned e;
      bool bad = false;
      while ((e = UNI(__hip_atomic_load(ef, RLX_SYS))) < 2u * call) {
        const unsigned ok = lds_get(fl + F_GOK * 4);
        if (ok == 2u || now() - t0 > DEADLINE) { __hip_atomic_store(tmo, call, RLX_AGENT); bad = true; break; }
        if (ok == 0u) __builtin_amdgcn_s_sleep(12);
      }
      if (bad || (e & 1u)) {
        __builtin_amdgcn_fence(__ATOMIC_ACQUIRE, "wavefront");
        const __amdgpu_buffer_rsrc_t rI = __builtin_amdgcn_make_buffer_rsrc(
            (void*)(ws + WS_E + (size_t)((par * 8 + dst) * 128 + slot) * 2048), 0, bad ? 0 : 2048, 0x00020000);
        const u32x4 h0 = __builtin_amdgcn_raw_buffer_load_b128(rI, lane * 16, 0, AUX_SYS);
        const u32x4 h1 = __builtin_amdgcn_raw_buffer_load_b128(rI, lane * 16, 1024, AUX_SYS);
        u32x2 gv;
        while (gv = lds_get2(fl + F_GOK * 4), UNI(gv.x) == 0u) __builtin_amdgcn_s_sleep(1);
        const unsigned rsb = gv.y;
        const float rs = __builtin_bit_cast(float, rsb);
        u32x4 o;
        o.x = q4(bf_lo(h0.x), bf_hi(h0.x), bf_lo(h0.y), bf_hi(h0.y), rs);
        o.y = q4(bf_lo(h0.z), bf_hi(h0.z), bf_lo(h0.w), bf_hi(h0.w), rs);
        o.z = q4(bf_lo(h1.x), bf_hi(h1.x), bf_lo(h1.y), bf_hi(h1.y), rs);
        o.w = q4(bf_lo(h1.z), bf_hi(h1.z), bf_lo(h1.w), bf_hi(h1.w), rs);
        if (bad || gv.x != 1u) o = u32x4{0x80808080u, 0x80808080u, 0x80808080u, 0x80808080u};
        store_sys(p.out[rank] + (size_t)(dst * M + rowin) * 512 + colb, o);
      }
    } else {
      while (lds_get(fl + F_GOK * 4) == 0u) __builtin_amdgcn_s_sleep(2);
      GAS unsigned* d = (GAS unsigned*)(ws + WS_DONE) + (slot * 8 + (lane & 7)) * 32;
      bool bad = false;
      while (true) {
        const unsigned v = __hip_atomic_load(d, RLX_SYS);
        const unsigned tv = __hip_atomic_load(tmo, RLX_AGENT);
        if (__all(v >= call || (lane & 7) == rank)) break;
        if (expired(tmo, tv, call, t0)) { bad = true; break; }
      }
      if (bad && lane == 0) p.out[rank][(size_t)(rank * M + rowin) * 512 + colb] = 0x80;
    }
  }
}

extern "C" void kernel_launch(int rank, int nranks, ncclComm_t comm, hipStream_t stream,
                              void* const* d_in, const int* in_sizes, int n_in,
                              void* d_out, int out_size, const int* in_dtypes, int out_dtype,
                              void* const* d_out_all, void* const* d_ws_all, size_t ws_bytes) {
  if (nranks != NR || rank < 0 || rank >= NR || n_in < 2 || ws_bytes < WS_NEED || in_sizes[0] != M * K ||
      in_sizes[1] != K * N || out_size != NR * M * (N / NR)) {
    fprintf(stderr, "unexpected problem shape\n");
    return;
  }
  static bool init = false;
  if (!init) {
    HIP_CHECK(hipFuncSetAttribute((const void*)fused_gemm_a2a, hipFuncAttributeMaxDynamicSharedMemorySize, LDS_BYTES));
    init = true;
  }
  Params p;
  p.x = (const float*)d_in[0];
  p.w = (const float*)d_in[1];
  for (int i = 0; i < NR; i++) { p.out[i] = (unsigned char*)d_out_all[i]; p.ws[i] = (unsigned char*)d_ws_all[i]; }
  p.self = (unsigned char*)d_ws_all[rank];
  p.rank = rank;
  p.pad = 0;
  hipLaunchKernelGGL(fused_gemm_a2a, dim3(NWG), dim3(512), LDS_BYTES, stream, p);
  HIP_CHECK(hipGetLastError());
}
```

```cpp
#include <hip/hip_runtime.h>
#include <rccl/rccl.h>
#include <cstdio>
#include <utility>

#define HIP_CHECK(expr) do { hipError_t _e = (expr); if (_e != hipSuccess) { \
    fprintf(stderr, "HIP error %s at %s:%d\n", hipGetErrorString(_e), __FILE__, __LINE__); \
    return; } } while (0)

typedef unsigned u32x4 __attribute__((ext_vector_type(4)));
typedef unsigned u32x2 __attribute__((ext_vector_type(2)));
typedef float f32x4 __attribute__((ext_vector_type(4)));
typedef float f32x2 __attribute__((ext_vector_type(2)));
typedef __bf16 bf16x2 __attribute__((ext_vector_type(2)));
typedef unsigned long long u64;
#define GAS __attribute__((address_space(1)))
#define LAS __attribute__((address_space(3)))
#define DI __device__ __forceinline__
#pragma clang diagnostic ignored "-Wint-to-pointer-cast"

constexpr int NR = 8;
constexpr int M = 512, N = 4096, K = 4096;
constexpr int NWG = 256;
constexpr int NSTEP = 32;
constexpr int WS_TMO = 0;
constexpr int WS_CNT = 4096;
constexpr int WS_LOC = 8192;
constexpr int WS_AFLAG = 64 * 1024;
constexpr int WS_MBOX = 96 * 1024;
constexpr int WS_BOOK = 120 * 1024;
constexpr int WS_DONE = 128 * 1024;
constexpr int WS_EF = 256 * 1024;
constexpr int WS_XFLAG = 576 * 1024;
constexpr int WS_AT = 1 << 20;
constexpr int WS_X = 5 << 20;
constexpr int X_BYTES = NWG * 3 * 4 * 4096;
constexpr int WS_E = 18 << 20;
constexpr size_t WS_NEED = (size_t)WS_E + 2 * 8 * 128 * 2048;
constexpr u64 DEADLINE = 40000000ull;
constexpr int NWS = 19;
constexpr int L_F = NWS * 8192;
constexpr int L_SC = L_F - 16384;
constexpr int LDS_BYTES = L_F + 256;
static_assert(LDS_BYTES <= 160 * 1024, "LDS");
enum : int {
  F_FULLW = 0,
  F_DONE = 4,
  F_RDY = 32,
  F_MAX = 12,
  F_LMAX = 16,
  F_GOK = 20,
  F_CALL = 22, F_ST, F_DR, F_FS, F_CNT, F_GS, F_LCNT, F_TS
};
constexpr int AUX_SYS = 17;
constexpr unsigned EGAP = 5;

struct Params {
  const float* x;
  const float* w;
  unsigned char* self;
  int rank;
  int pad;
  unsigned char* out[NR];
  unsigned char* ws[NR];
};

#define RLX_AGENT __ATOMIC_RELAXED, __HIP_MEMORY_SCOPE_AGENT
#define RLX_SYS __ATOMIC_RELAXED, __HIP_MEMORY_SCOPE_SYSTEM
#define CFENCE() asm volatile("" ::: "memory")
#define SB() __builtin_amdgcn_sched_barrier(0)
#define UNI(x) __builtin_amdgcn_readfirstlane(x)

DI unsigned pk(float a, float b) {
  f32x2 v = {a, b};
  bf16x2 r = __builtin_convertvector(v, bf16x2);
  return __builtin_bit_cast(unsigned, r);
}
DI float bf_lo(unsigned v) { return __builtin_bit_cast(float, v << 16); }
DI float bf_hi(unsigned v) { return __builtin_bit_cast(float, v & 0xffff0000u); }
DI unsigned q4(float a, float b, float c, float d, float rs) {
  const unsigned t0 = __builtin_bit_cast(unsigned, __builtin_fmaf(a, rs, 12582912.0f));
  const unsigned t1 = __builtin_bit_cast(unsigned, __builtin_fmaf(b, rs, 12582912.0f));
  const unsigned t2 = __builtin_bit_cast(unsigned, __builtin_fmaf(c, rs, 12582912.0f));
  const unsigned t3 = __builtin_bit_cast(unsigned, __builtin_fmaf(d, rs, 12582912.0f));
  return __builtin_amdgcn_perm(t1, t0, 0x0c0c0400u) | __builtin_amdgcn_perm(t3, t2, 0x04000c0cu);
}
DI u64 now() { return __builtin_amdgcn_s_memrealtime(); }
DI unsigned wave_max(unsigned v) {
  v = max(v, (unsigned)__builtin_amdgcn_update_dpp(0, (int)v, 0x111, 0xf, 0xf, true));
  v = max(v, (unsigned)__builtin_amdgcn_update_dpp(0, (int)v, 0x112, 0xf, 0xf, true));
  v = max(v, (unsigned)__builtin_amdgcn_update_dpp(0, (int)v, 0x114, 0xf, 0xf, true));
  v = max(v, (unsigned)__builtin_amdgcn_update_dpp(0, (int)v, 0x118, 0xf, 0xf, true));
  const unsigned a = __builtin_amdgcn_readlane(v, 15), b = __builtin_amdgcn_readlane(v, 31);
  const unsigned c = __builtin_amdgcn_readlane(v, 47), d = __builtin_amdgcn_readlane(v, 63);
  return max(max(a, b), max(c, d));
}
DI bool expired(GAS unsigned* tmo, unsigned tv, unsigned call, u64 t0) {
  if (UNI(tv) == call) return true;
  if (now() - t0 > DEADLINE) { __hip_atomic_store(tmo, call, RLX_AGENT); return true; }
  return false;
}
DI void store_sys(const void* p, u32x4 v) {
  asm volatile("global_store_dwordx4 %0, %1, off sc0 sc1\n\ts_nop 1" ::"v"(p), "v"(v) : "memory");
}
DI void lds_put(unsigned addr, unsigned v) { asm volatile("ds_write_b32 %0, %1" ::"v"(addr), "v"(v) : "memory"); }
DI unsigned lds_get(unsigned addr) {
  unsigned v;
  asm volatile("ds_read_b32 %0, %1\n\ts_waitcnt lgkmcnt(0)" : "=v"(v) : "v"(addr) : "memory");
  return UNI(v);
}
DI u32x2 lds_get2(unsigned addr) {
  u32x2 v;
  asm volatile("ds_read_b64 %0, %1\n\ts_waitcnt lgkmcnt(0)" : "=v"(v) : "v"(addr) : "memory");
  return v;
}
DI u32x4 lds_get4(unsigned addr) {
  u32x4 v;
  asm volatile("ds_read_b128 %0, %1\n\ts_waitcnt lgkmcnt(0)" : "=v"(v) : "v"(addr) : "memory");
  return v;
}
DI unsigned lds_count(unsigned cnt) {
  unsigned old;
  asm volatile("s_mov_b64 exec, 1\n\tds_add_rtn_u32 %0, %1, %2\n\ts_mov_b64 exec, -1\n\ts_waitcnt lgkmcnt(0)"
               : "=&v"(old) : "v"(cnt), "v"(1u) : "memory");
  return UNI(old);
}
DI void wait8(const GAS unsigned* f8, unsigned call, GAS unsigned* tmo) {
  const u64 t0 = now();
  for (unsigned it = 1;; it++) {
    unsigned a0, a1, a2, a3, a4, a5, a6, a7;
    asm volatile(
        "s_load_dword %0, %8, 0x0 glc\n\ts_load_dword %1, %8, 0x80 glc\n\ts_load_dword %2, %8, 0x100 glc\n\t"
        "s_load_dword %3, %8, 0x180 glc\n\ts_load_dword %4, %8, 0x200 glc\n\ts_load_dword %5, %8, 0x280 glc\n\t"
        "s_load_dword %6, %8, 0x300 glc\n\ts_load_dword %7, %8, 0x380 glc\n\ts_waitcnt lgkmcnt(0)"
        : "=&s"(a0), "=&s"(a1), "=&s"(a2), "=&s"(a3), "=&s"(a4), "=&s"(a5), "=&s"(a6), "=&s"(a7)
        : "s"(f8) : "memory");
    if (min(min(min(a0, a1), min(a2, a3)), min(min(a4, a5), min(a6, a7))) >= call) break;
    if ((it & 255u) == 0u) {
      unsigned tv;
      asm volatile("global_load_dword %0, %1, off sc1\n\ts_waitcnt vmcnt(0)" : "=&v"(tv) : "v"(tmo) : "memory");
      if (UNI(tv) == call) break;
      if (now() - t0 > DEADLINE) {
        asm volatile("global_store_dword %0, %1, off sc1\n\ts_waitcnt vmcnt(0)" ::"v"(tmo), "v"(call) : "memory");
        break;
      }
    }
    __builtin_amdgcn_s_sleep(2);
  }
}

enum : int { O_, P_, CW_, G_, WS_, WF_, Q_ };
struct Op { int k, a; };
#define RND(r) {G_, 4 * (r) - 6}, {P_, r}, {O_, (r) + 1}, {CW_, r}
constexpr Op SCHED[] = {
  {O_, 0}, {WS_, 0}, {P_, 0}, {WF_, 0}, {O_, 1}, {CW_, 0}, {P_, 1}, {O_, 2}, {CW_, 1}, {Q_, 0},
  RND(2), RND(3), RND(4), RND(5), RND(6), {G_, 22}, {P_, 7}, {CW_, 7},
};
constexpr int NOPS = sizeof(SCHED) / sizeof(Op);
constexpr int NSET = 5;
constexpr bool is_ld(int k) { return k == O_ || k == P_; }
constexpr int find_op(int k, int a) { for (int i = 0; i < NOPS; i++) if (SCHED[i].k == k && SCHED[i].a == a) return i; return -1; }
constexpr int younger(int t, int i) { int n = 0; for (int j = t + 1; j < i; j++) n += is_ld(SCHED[j].k) ? 8 : 0; return n; }
constexpr int set_of(int op) {
  int owner[NSET] = {-1, -1, -1, -1, -1};
  for (int i = 0; i < NOPS; i++) {
    if (is_ld(SCHED[i].k)) {
      int s = -1;
      for (int q = 0; q < NSET; q++) if (owner[q] < 0) { s = q; break; }
      if (s < 0) return -1;
      owner[s] = SCHED[i].a;
      if (i == op) return s;
    } else if (SCHED[i].k == CW_) {
      for (int q = 0; q < NSET; q++) if (owner[q] == SCHED[i].a) owner[q] = -1;
    }
  }
  return -1;
}
constexpr bool can_reach(int n, int i) { return n <= 0 || find_op(CW_, n / 4) < i; }
constexpr bool sched_ok() {
  for (int r = 0; r < 8; r++) {
    const int o = find_op(O_, r), p = find_op(P_, r), c = find_op(CW_, r);
    if (o < 0 || p < o || c < p || set_of(o) < 0 || set_of(p) < 0 || younger(o, c) > 63) return false;
    if (r > 0 && c < find_op(CW_, r - 1)) return false;
    if (!can_reach(4 * r + 3 - NWS + 1, c)) return false;
    if (4 * r + 3 >= NWS - 2 && c < find_op(WS_, 0)) return false;
  }
  for (int i = 0; i < NOPS; i++) if (SCHED[i].k == G_ && (!can_reach(SCHED[i].a, i) || i < find_op(Q_, 0))) return false;
  return true;
}
static_assert(sched_ok(), "loader schedule");

struct Loader {
  f32x4 wb[NSET][8];
  __amdgpu_buffer_rsrc_t rW;
  int vW[8];
  unsigned sW, own, ldsW, l, fl;
  const GAS unsigned* afl;
  GAS unsigned* tmo;
  const unsigned char *po, *pw, *pe;
  bool rt;
};
#define WAIT_SET(S, N)                                                                               \
  asm volatile("s_waitcnt vmcnt(%8)"                                                                 \
               : "+v"(L.wb[S][0]), "+v"(L.wb[S][1]), "+v"(L.wb[S][2]), "+v"(L.wb[S][3]),             \
                 "+v"(L.wb[S][4]), "+v"(L.wb[S][5]), "+v"(L.wb[S][6]), "+v"(L.wb[S][7])              \
               : "i"(N) : "memory")
template <int I> DI void run_op(Loader& L) {
  constexpr int k = SCHED[I].k, a = SCHED[I].a;
  if constexpr (is_ld(k)) {
    constexpr int s = set_of(I);
    const unsigned so = (unsigned)a * (128u * N * 4u) + L.sW + ((k == O_) ? L.own : (256u - L.own));
    asm volatile(
        "buffer_load_dwordx4 %0, %8, %16, %17 offen\n\t"
        "buffer_load_dwordx4 %1, %9, %16, %17 offen\n\t"
        "buffer_load_dwordx4 %2, %10, %16, %17 offen\n\t"
        "buffer_load_dwordx4 %3, %11, %16, %17 offen\n\t"
        "buffer_load_dwordx4 %4, %12, %16, %17 offen\n\t"
        "buffer_load_dwordx4 %5, %13, %16, %17 offen\n\t"
        "buffer_load_dwordx4 %6, %14, %16, %17 offen\n\t"
        "buffer_load_dwordx4 %7, %15, %16, %17 offen"
        : "=&v"(L.wb[s][0]), "=&v"(L.wb[s][1]), "=&v"(L.wb[s][2]), "=&v"(L.wb[s][3]),
          "=&v"(L.wb[s][4]), "=&v"(L.wb[s][5]), "=&v"(L.wb[s][6]), "=&v"(L.wb[s][7])
        : "v"(L.vW[0]), "v"(L.vW[1]), "v"(L.vW[2]), "v"(L.vW[3]), "v"(L.vW[4]), "v"(L.vW[5]), "v"(L.vW[6]), "v"(L.vW[7]),
          "s"(L.rW), "s"(so)
        : "memory");
  } else if constexpr (k == CW_) {
    constexpr int io = find_op(O_, a), ip = find_op(P_, a), so = set_of(io), sp = set_of(ip);
    const unsigned t = 4u * a + L.l;
    if constexpr (4 * a + 3 >= NWS) {
      const int need = (int)t - NWS + 1;
      while (true) {
        const u32x4 d = lds_get4(L.fl + F_DONE * 4);
        if ((int)UNI(min(min(d.x, d.y), min(d.z, d.w))) >= need) break;
        __builtin_amdgcn_s_sleep(1);
      }
    }
    unsigned slot = (4u * a) % NWS + L.l;
    if (slot >= (unsigned)NWS) slot -= NWS;
    const unsigned base = L.ldsW + slot * 8192u;
    auto conv = [&](f32x4(&w)[8], unsigned dst) {
#pragma unroll
      for (int c = 0; c < 4; c++) {
        u32x4 bf;
        bf.x = pk(w[0][c], w[1][c]); bf.y = pk(w[2][c], w[3][c]);
        bf.z = pk(w[4][c], w[5][c]); bf.w = pk(w[6][c], w[7][c]);
        *(LAS u32x4*)(dst + c * 1024) = bf;
      }
    };
    WAIT_SET(so, younger(io, I));
    conv(L.wb[so], base + L.own * 16u);
    WAIT_SET(sp, younger(ip, I));
    conv(L.wb[sp], base + (4096u - L.own * 16u));
    lds_put(L.fl + (F_FULLW + L.l) * 4, t + 1u);
  } else if constexpr (k == G_) {
    while (lds_get(L.fl + (F_DONE + L.l) * 4) < (unsigned)a) __builtin_amdgcn_s_sleep(1);
  } else if constexpr (k == WS_) {
    if (L.rt) {
      unsigned r0, r1, r2, r3;
      asm volatile("s_load_dword %0, %4, 0x0\n\ts_load_dword %1, %5, 0x20000\n\ts_load_dword %2, %6, 0x0\n\t"
                   "s_load_dword %3, %7, 0x0\n\ts_waitcnt lgkmcnt(0)"
                   : "=&s"(r0), "=&s"(r1), "=&s"(r2), "=&s"(r3) : "s"(L.po), "s"(L.pw), "s"(L.pe), "s"(L.pe + (2 << 20)) : "memory");
    }
    while (lds_get(L.fl + F_ST * 4) < 4u) __builtin_amdgcn_s_sleep(1);
  } else if constexpr (k == WF_) {
    while (lds_get(L.fl + F_FS * 4) == 0u) __builtin_amdgcn_s_sleep(1);
  } else if (L.l != 0u) {
    wait8(L.afl + L.l * 8u * 32u, lds_get(L.fl + F_CALL * 4), L.tmo);
    lds_put(L.fl + (F_RDY + L.l) * 4, 1u);
  }
}
template <int... I> DI void run_all(Loader& L, std::integer_sequence<int, I...>) { (run_op<I>(L), ...); }


__global__ void __launch_bounds__(512, 2) fused_gemm_a2a(Params p) {
  extern __shared__ __attribute__((aligned(16))) unsigned char smem[];
  const int tid = threadIdx.x, lane = tid & 63;
  const int w = UNI(tid >> 6);
  const int b = blockIdx.x;
  const int xcd = b & 7, j = b >> 3;
  const int kq = xcd >> 1, mb2 = j & 1;
  const int nb = (xcd & 1) * 16 + (j >> 1);
  const int grp = mb2 * 4 + kq;
  const unsigned lds0 = (unsigned)(size_t)(LAS unsigned char*)smem;
  const unsigned fl = lds0 + L_F;
  f32x4 xa[8];
  unsigned cv = 0, tw = 0;
  if (w < 4) {
    const __amdgpu_buffer_rsrc_t rX = __builtin_amdgcn_make_buffer_rsrc((void*)p.x, 0, M * K * 4, 0x00020000);
    const int vX = (lane >> 3) * (K * 4) + (lane & 7) * 16;
    const int so = (mb2 * 256 + w * 64) * (K * 4) + (kq * 1024 + nb * 32) * 4;
#pragma unroll
    for (int i = 0; i < 8; i++)
      xa[i] = __builtin_bit_cast(f32x4, __builtin_amdgcn_raw_buffer_load_b128(rX, vX, so + i * 8 * (K * 4), 0));
    SB();
    cv = __hip_atomic_load((GAS unsigned*)(p.self + WS_CNT) + b * 4 + w, RLX_AGENT);
    {
      const int t = lane & 15;
      const unsigned char* a = p.self + WS_AT + ((grp * NSTEP + nb) * 4 + w) * 4096;
      if (t == 1) a = p.self + WS_AFLAG + (grp * NSTEP + nb) * 128;
      if (t == 2) a = p.self + WS_XFLAG + b * 12 * 128;
      if (t == 3) a = p.self + WS_AT + grp * NSTEP * 16384;
      if (t >= 4 && t < 8) a = p.self + WS_X + (size_t)((b ^ ((t & 3) << 1)) * 12) * 4096;
      if (t >= 8) a = (const unsigned char*)p.w + (size_t)(kq * 1024 + (t - 8) * 128) * (N * 4) + nb * 512 + mb2 * 256;
      tw = __hip_atomic_load((GAS unsigned*)a, RLX_AGENT);
    }
    if (tid < 64) *(LAS unsigned*)(fl + tid * 4) = 0u;
    asm volatile("s_waitcnt lgkmcnt(0)" ::: "memory");
    SB();
  }
  __builtin_amdgcn_s_barrier();
  CFENCE();
  SB();
  const int rank = p.rank;
  unsigned char* ws = p.self;
  GAS unsigned* tmo = (GAS unsigned*)(ws + WS_TMO);
  GAS unsigned* afl = (GAS unsigned*)(ws + WS_AFLAG) + grp * NSTEP * 32;
  const int c = w & 3;
  const unsigned dst = (unsigned)(nb >> 2) & 7u;
  const int idx = ((nb & 3) * 2 + mb2) * 4 + kq;
  const int slot = idx * 4 + c;
  const bool remote = dst != (unsigned)rank;
  const unsigned me = (unsigned)rank;
  const int rowin = mb2 * 256 + c * 64 + kq * 16 + (lane & 15), colb = (nb & 3) * 128 + 16 * (lane >> 4);

  if (__builtin_expect(w < 4, 1)) {
    const unsigned call = UNI(cv) + 1u;
    {
      const unsigned sc = lds0 + L_SC + c * 4096;
      const unsigned tb = sc + ((lane & 7) >> 1) * 256 + (lane >> 3) * 16 + (lane & 1) * 8;
#pragma unroll
      for (int i = 0; i < 8; i++) {
        u32x2 o = {pk(xa[i][0], xa[i][1]), pk(xa[i][2], xa[i][3])};
        *(LAS u32x2*)(tb + (((unsigned)(i >> 1)) ^ (unsigned)kq) * 1024u + (i & 1) * 128) = o;
      }
      asm volatile("" ::"v"(tw));
      u32x4 fr[4];
#pragma unroll
      for (int f = 0; f < 4; f++) fr[f] = *(LAS u32x4*)(sc + f * 1024 + lane * 16);
      unsigned char* pub = ws + WS_AT + ((grp * NSTEP + nb) * 4 + c) * 4096 + lane * 16;
#pragma unroll
      for (int f = 0; f < 4; f++) store_sys(pub + f * 1024, fr[f]);
      lds_put(fl + F_CALL * 4, call);
      lds_count(fl + F_ST * 4);
      asm volatile("s_waitcnt vmcnt(0)" ::: "memory");
      if (lds_count(fl + F_DR * 4) == 3u) {
        if (lane == 0) __hip_atomic_store(afl + nb * 32, call, RLX_SYS);
        lds_put(fl + F_FS * 4, 1u);
      }
      if (lane == 0) __hip_atomic_store((GAS unsigned*)(ws + WS_CNT) + b * 4 + c, call, RLX_AGENT);
      if (c == 0) {
        while (lds_get(fl + F_FS * 4) == 0u) {}
        wait8(afl, call, tmo);
        lds_put(fl + (F_RDY + 4) * 4, 1u);
        lds_put(fl + F_RDY * 4, 1u);
      }
    }
    f32x4 acc[4][8];
#pragma unroll
    for (int m = 0; m < 4; m++)
#pragma unroll
      for (int n = 0; n < 8; n++) { acc[m][n] = f32x4{0, 0, 0, 0}; asm volatile("" : "+v"(acc[m][n])); }
    const unsigned wBase = lds0 + lane * 16;
    unsigned pfw = fl + F_FULLW * 4, pdone = fl + (F_DONE + c) * 4, vdone = 0;
    asm volatile("" : "+v"(pfw), "+v"(pdone), "+v"(vdone));
    const __amdgpu_buffer_rsrc_t rA = __builtin_amdgcn_make_buffer_rsrc(
        (void*)(ws + WS_AT + (grp * NSTEP * 4 + c) * 4096), 0, (NSTEP - 1) * 16384 + 4096, 0x00020000);
    int vA[4];
#pragma unroll
    for (int m = 0; m < 4; m++) { vA[m] = lane * 16 + m * 1024; asm volatile("" : "+v"(vA[m])); }
    auto ldA = [&](int m, unsigned t) { return __builtin_amdgcn_raw_buffer_load_b128(rA, vA[m], t * 16384u, AUX_SYS); };
    u32x4 A[5][4], B[4];
#define SETOF(t) ((((t) & 7) == 5) ? 4 : ((t) & 3))
    unsigned fw;
    while (lds_get(fl + F_RDY * 4) == 0u) __builtin_amdgcn_s_sleep(1);
#pragma unroll
    for (int d = 0; d < 2; d++)
#pragma unroll
      for (int m = 0; m < 4; m++) { A[d][m] = ldA(m, d); SB(); }
    while (lds_get(pfw) < 1u) {}
    __builtin_amdgcn_s_setprio(3);
    B[0] = *(LAS u32x4*)(wBase);
    B[1] = *(LAS u32x4*)(wBase + 1024);
    B[2] = *(LAS u32x4*)(wBase + 2048);
    CFENCE(); fw = *(LAS unsigned*)(pfw + 4); CFENCE();
    unsigned woff = 0, wc = wBase, rdy = 0, prdy = fl + (F_RDY + 1) * 4;
    asm volatile("" : "+v"(prdy));
    auto mf = [&](int m, int n, int q) {
      asm volatile("v_mfma_f32_16x16x32_bf16 %0, %1, %2, %0" : "+v"(acc[m][n]) : "v"(B[n & 3]), "v"(A[q][m]));
    };
    auto step = [&](unsigned s, int q8) {
      const int q = q8 & 3;
      unsigned wn = 0, sf = 0, nd = 0;
#pragma unroll
      for (int n = 0; n < 8; n++) {
        mf(0, n, SETOF(q8)); SB();
        if (n == 0 && q8 == 5) {
          if (__builtin_expect(UNI(rdy) == 0u, 0)) while (lds_get(prdy) == 0u) {}
          prdy += 4u;
          CFENCE();
        }
        if (q == 1 && n < 4) { A[SETOF(q8 + 3)][n] = ldA(n, s + 3u); SB(); }
        if (n < 5) B[(n + 3) & 3] = *(LAS u32x4*)(wc + (n + 3) * 1024);
        else B[(n + 3) & 3] = *(LAS u32x4*)(wn + (n - 5) * 1024);
        SB(); mf(1, n, SETOF(q8)); SB();
        if (n == 1) { woff = (woff == (NWS - 1) * 8192u) ? 0u : woff + 8192u; wn = wBase + woff; asm volatile("" : "+v"(wn)); }
        if (n == 2) sf = UNI(fw);
        if (n == 4) lds_put(pdone, vdone);
        if (n == 7) { CFENCE(); fw = *(LAS unsigned*)(pfw + ((q + 2) & 3) * 4); CFENCE(); }
        SB(); mf(2, n, SETOF(q8)); SB();
        if (q == 0) A[SETOF(q8 + 2 + (n >> 2))][n & 3] = ldA(n & 3, s + 2u + (n >> 2));
        if (q == 1 && n < 4) A[SETOF(q8 + 4)][n] = ldA(n, s + 4u);
        if (n == 1) { vdone += 1; asm volatile("" : "+v"(vdone)); }
        if (n == 3) { nd = (q8 == 7 && s == NSTEP - 1u) ? 0u : s + 2u; asm volatile("" : "+s"(nd)); }
        if (n == 5 && q8 == 4) { CFENCE(); rdy = *(LAS unsigned*)(prdy); CFENCE(); }
        SB(); mf(3, n, SETOF(q8)); SB();
        if (n == 3) {
          if (__builtin_expect(sf < nd, 0))
            while (lds_get(pfw + ((q + 1) & 3) * 4) < nd) {}
          CFENCE();
          SB();
        }
      }
      wc = wn;
    };
    asm volatile("s_nop 7");
    for (unsigned s = 0; s < (unsigned)NSTEP; s += 8) {
      step(s, 0); step(s + 1, 1); step(s + 2, 2); step(s + 3, 3);
      step(s + 4, 4); step(s + 5, 5); step(s + 6, 6); step(s + 7, 7);
    }
    asm volatile("s_nop 15\n\ts_nop 15");
    SB();
    __builtin_amdgcn_s_setprio(0);
    const __amdgpu_buffer_rsrc_t rE = __builtin_amdgcn_make_buffer_rsrc((void*)(ws + WS_X), 0, X_BYTES, 0x00020000);
#pragma unroll
    for (int m = 1; m < 4; m++) {
      const int so = (((b ^ (m << 1)) * 3 + (m - 1)) * 4 + c) * 4096;
#pragma unroll
      for (int t = 0; t < 4; t++) {
        u32x4 o;
        o.x = pk(acc[m][2 * t][0], acc[m][2 * t][1]); o.y = pk(acc[m][2 * t][2], acc[m][2 * t][3]);
        o.z = pk(acc[m][2 * t + 1][0], acc[m][2 * t + 1][1]); o.w = pk(acc[m][2 * t + 1][2], acc[m][2 * t + 1][3]);
        __builtin_amdgcn_raw_buffer_store_b128(o, rE, lane * 16, so + t * 1024, AUX_SYS);
      }
    }
    GAS unsigned* xfl = (GAS unsigned*)(ws + WS_XFLAG);
    asm volatile("s_waitcnt vmcnt(0)" ::: "memory");
    if (lane < 3) __hip_atomic_store(xfl + (((b ^ ((lane + 1) << 1)) * 4 + c) * 3 + lane) * 32, call, RLX_SYS);
    {
      const u64 t0 = now();
      while (true) {
        const unsigned v = __hip_atomic_load(xfl + ((b * 4 + c) * 3 + min(lane, 2)) * 32, RLX_SYS);
        const unsigned tv = __hip_atomic_load(tmo, RLX_AGENT);
        if (__all(v >= call) || expired(tmo, tv, call, t0)) break;
      }
      __builtin_amdgcn_fence(__ATOMIC_ACQUIRE, "wavefront");
    }
    u32x4 in[3][4];
#pragma unroll
    for (int m = 0; m < 3; m++)
#pragma unroll
      for (int t = 0; t < 4; t++)
        in[m][t] = __builtin_amdgcn_raw_buffer_load_b128(rE, lane * 16, ((b * 3 + m) * 4 + c) * 4096 + t * 1024, AUX_SYS);
    float am = 0.f;
#pragma unroll
    for (int t = 0; t < 4; t++) {
#pragma unroll
      for (int m = 0; m < 3; m++)
#pragma unroll
        for (int e = 0; e < 4; e++) {
          const unsigned v = in[m][t][e];
          acc[0][2 * t + (e >> 1)][(e & 1) * 2] += bf_lo(v);
          acc[0][2 * t + (e >> 1)][(e & 1) * 2 + 1] += bf_hi(v);
        }
#pragma unroll
      for (int e = 0; e < 8; e++) am = fmaxf(am, fabsf(acc[0][2 * t + (e >> 2)][e & 3]));
    }
    lds_put(fl + (F_MAX + c) * 4, wave_max(__builtin_bit_cast(unsigned, am)));
    if (lds_count(fl + F_CNT * 4) == 3u) {
      const u32x4 m4 = lds_get4(fl + F_MAX * 4);
      const unsigned mx = max(max(m4.x, m4.y), max(m4.z, m4.w));
      if (lane == 0) __hip_atomic_store((GAS u64*)(ws + WS_LOC) + b * 16, ((u64)call << 32) | (u64)mx, RLX_SYS);
      lds_put(fl + F_GS * 4, 1u);
      if (remote) {
        GAS u64* bk = (GAS u64*)(ws + WS_BOOK) + dst * 16;
        u64 tb = 0;
        if (lane == 0) {
          __hip_atomic_fetch_max(bk, now(), RLX_AGENT);
          tb = __hip_atomic_fetch_add(bk, (u64)(4u * EGAP), RLX_AGENT);
        }
        lds_put(fl + F_TS * 4, UNI((unsigned)tb) | 1u);
      }
    }
    const unsigned par = call & 1u;
    u32x4 h0, h1;
    h0.x = pk(acc[0][0][0], acc[0][1][0]); h0.y = pk(acc[0][2][0], acc[0][3][0]);
    h0.z = pk(acc[0][0][1], acc[0][1][1]); h0.w = pk(acc[0][2][1], acc[0][3][1]);
    h1.x = pk(acc[0][0][2], acc[0][1][2]); h1.y = pk(acc[0][2][2], acc[0][3][2]);
    h1.z = pk(acc[0][0][3], acc[0][1][3]); h1.w = pk(acc[0][2][3], acc[0][3][3]);
    unsigned char* optr = p.out[dst] + (size_t)(rank * M + rowin) * 512 + colb;
    GAS unsigned* dnp = (GAS unsigned*)(p.ws[dst] + WS_DONE) + (slot * 8 + me) * 32;
    GAS unsigned* efp = (GAS unsigned*)(p.ws[dst] + WS_EF) + ((par * 128 + slot) * 8 + me) * 32;
    asm volatile("" : "+v"(optr), "+v"(dnp), "+v"(efp));
    bool sent = false;
    u32x2 gv;
    {
      unsigned ts = 0;
      while (gv = lds_get2(fl + F_GOK * 4), UNI(gv.x) == 0u) {
        if (remote && ts == 0u) ts = lds_get(fl + F_TS * 4);
        if (ts != 0u && !sent && (int)((unsigned)now() - (ts + (unsigned)c * EGAP)) >= 0) {
          unsigned char* ep = p.ws[dst] + WS_E + (size_t)((par * 8 + me) * 128 + slot) * 2048 + lane * 16;
          store_sys(ep, h0);
          store_sys(ep + 1024, h1);
          asm volatile("s_waitcnt vmcnt(0)" ::: "memory");
          if (lane == 0) __hip_atomic_store(efp, 2u * call + 1u, RLX_SYS);
          sent = true;
        }
      }
    }
    const unsigned rsb = gv.y;
    const float rs = __builtin_bit_cast(float, rsb);
    const bool failed = gv.x != 1u;
    const u32x4 poison = {0x80808080u, 0x80808080u, 0x80808080u, 0x80808080u};
    {
      u32x4 o;
#pragma unroll
      for (int rg = 0; rg < 4; rg++) o[rg] = q4(acc[0][4][rg], acc[0][5][rg], acc[0][6][rg], acc[0][7][rg], rs);
      store_sys(optr + 64, failed ? poison : o);
    }
    if (!sent) {
      u32x4 o;
      o.x = q4(bf_lo(h0.x), bf_hi(h0.x), bf_lo(h0.y), bf_hi(h0.y), rs);
      o.y = q4(bf_lo(h0.z), bf_hi(h0.z), bf_lo(h0.w), bf_hi(h0.w), rs);
      o.z = q4(bf_lo(h1.x), bf_hi(h1.x), bf_lo(h1.y), bf_hi(h1.y), rs);
      o.w = q4(bf_lo(h1.z), bf_hi(h1.z), bf_lo(h1.w), bf_hi(h1.w), rs);
      store_sys(optr, failed ? poison : o);
    }
    if (remote) {
      if (!sent && lane == 0) __hip_atomic_store(efp, 2u * call, RLX_SYS);
      asm volatile("s_waitcnt vmcnt(0)" ::: "memory");
      if (lane == 0) __hip_atomic_store(dnp, call, RLX_SYS);
    }
  } else {
    {
      Loader L;
      L.l = c;
            L.rW = __builtin_amdgcn_make_buffer_rsrc((void*)p.w, 0, K * N * 4, 0x00020000);
#pragma unroll
      for (int i = 0; i < 8; i++) L.vW[i] = ((lane >> 4) * 8 + i) * (N * 4) + (lane & 15) * 16;
      L.sW = (unsigned)(kq * 1024 + c * 32) * (N * 4u) + (unsigned)nb * 512u;
      L.own = (unsigned)mb2 * 256u;
      L.ldsW = lds0 + lane * 16;
      L.fl = fl;
      L.afl = afl;
      L.tmo = tmo;
      L.po = p.out[j & 7] + (size_t)rank * M * 512;
      L.pw = p.ws[j & 7];
      L.pe = p.ws[j & 7] + WS_E + (size_t)rank * 128 * 2048;
      L.rt = c == 3 && j < NR;
      run_all(L, std::make_integer_sequence<int, NOPS>{});
    }
    const unsigned call = lds_get(fl + F_CALL * 4);
    const unsigned par = call & 1u;
    const u64 t0 = now();
    if (!remote && idx == 0) {
      GAS u64* g = (GAS u64*)(ws + WS_LOC) + (c * 64 + lane) * 16;
      const int srcid = rank;
      GAS u64* mb = (GAS u64*)(p.ws[lane >> 3] + WS_MBOX) + ((par * 8 + (lane & 7)) * 8 + srcid) * 16;
      asm volatile("" : "+v"(mb));
      u64 v;
      while (true) {
        v = __hip_atomic_load(g, RLX_SYS);
        const unsigned tv = __hip_atomic_load(tmo, RLX_AGENT);
        if (__all((unsigned)(v >> 32) == call) || expired(tmo, tv, call, t0)) break;
      }
      lds_put(fl + (F_LMAX + c) * 4, wave_max((unsigned)v));
      if (lds_count(fl + F_LCNT * 4) == 3u) {
        const u32x4 m4 = lds_get4(fl + F_LMAX * 4);
        const unsigned mx = max(max(m4.x, m4.y), max(m4.z, m4.w));
        __hip_atomic_store(mb, ((u64)call << 32) | (u64)mx, RLX_SYS);
      }
    }
    if (c == 1) {
      while (lds_get(fl + F_GS * 4) == 0u) __builtin_amdgcn_s_sleep(2);
      GAS u64* g = (GAS u64*)(ws + WS_MBOX) + ((par * 8 + xcd) * 8 + (lane & 7)) * 16;
      u64 v;
      bool bad;
      while (true) {
        v = __hip_atomic_load(g, RLX_SYS);
        const unsigned tv = __hip_atomic_load(tmo, RLX_AGENT);
        bad = expired(tmo, tv, call, t0);
        if (bad || __all((unsigned)(v >> 32) == call)) break;
      }
      const unsigned mx = wave_max((unsigned)v);
      const float amax = __builtin_bit_cast(float, mx);
      const float rs = amax > 0.f ? 127.0f / amax : 0.f;
      const u32x2 gw = {bad ? 2u : 1u, __builtin_bit_cast(unsigned, rs)};
      asm volatile("ds_write_b64 %0, %1" ::"v"(fl + F_GOK * 4), "v"(gw) : "memory");
    }
    if (remote) {
      GAS unsigned* ef = (GAS unsigned*)(ws + WS_EF) + ((par * 128 + slot) * 8 + dst) * 32;
      while (lds_get(fl + F_GS * 4) == 0u) __builtin_amdgcn_s_sleep(4);
      unsigned e;
      bool bad = false;
      while ((e = UNI(__hip_atomic_load(ef, RLX_SYS))) < 2u * call) {
        const unsigned ok = lds_get(fl + F_GOK * 4);
        if (ok == 2u || now() - t0 > DEADLINE) { __hip_atomic_store(tmo, call, RLX_AGENT); bad = true; break; }
        if (ok == 0u) __builtin_amdgcn_s_sleep(12);
      }
      if (bad || (e & 1u)) {
        __builtin_amdgcn_fence(__ATOMIC_ACQUIRE, "wavefront");
        const __amdgpu_buffer_rsrc_t rI = __builtin_amdgcn_make_buffer_rsrc(
            (void*)(ws + WS_E + (size_t)((par * 8 + dst) * 128 + slot) * 2048), 0, bad ? 0 : 2048, 0x00020000);
        const u32x4 h0 = __builtin_amdgcn_raw_buffer_load_b128(rI, lane * 16, 0, AUX_SYS);
        const u32x4 h1 = __builtin_amdgcn_raw_buffer_load_b128(rI, lane * 16, 1024, AUX_SYS);
        u32x2 gv;
        while (gv = lds_get2(fl + F_GOK * 4), UNI(gv.x) == 0u) __builtin_amdgcn_s_sleep(1);
        const unsigned rsb = gv.y;
        const float rs = __builtin_bit_cast(float, rsb);
        u32x4 o;
        o.x = q4(bf_lo(h0.x), bf_hi(h0.x), bf_lo(h0.y), bf_hi(h0.y), rs);
        o.y = q4(bf_lo(h0.z), bf_hi(h0.z), bf_lo(h0.w), bf_hi(h0.w), rs);
        o.z = q4(bf_lo(h1.x), bf_hi(h1.x), bf_lo(h1.y), bf_hi(h1.y), rs);
        o.w = q4(bf_lo(h1.z), bf_hi(h1.z), bf_lo(h1.w), bf_hi(h1.w), rs);
        if (bad || gv.x != 1u) o = u32x4{0x80808080u, 0x80808080u, 0x80808080u, 0x80808080u};
        store_sys(p.out[rank] + (size_t)(dst * M + rowin) * 512 + colb, o);
      }
    } else {
      while (lds_get(fl + F_GOK * 4) == 0u) __builtin_amdgcn_s_sleep(2);
      GAS unsigned* d = (GAS unsigned*)(ws + WS_DONE) + (slot * 8 + (lane & 7)) * 32;
      bool bad = false;
      while (true) {
        const unsigned v = __hip_atomic_load(d, RLX_SYS);
        const unsigned tv = __hip_atomic_load(tmo, RLX_AGENT);
        if (__all(v >= call || (lane & 7) == rank)) break;
        if (expired(tmo, tv, call, t0)) { bad = true; break; }
      }
      if (bad && lane == 0) p.out[rank][(size_t)(rank * M + rowin) * 512 + colb] = 0x80;
    }
  }
}

extern "C" void kernel_launch(int rank, int nranks, ncclComm_t comm, hipStream_t stream,
                              void* const* d_in, const int* in_sizes, int n_in,
                              void* d_out, int out_size, const int* in_dtypes, int out_dtype,
                              void* const* d_out_all, void* const* d_ws_all, size_t ws_bytes) {
  if (nranks != NR || rank < 0 || rank >= NR || n_in < 2 || ws_bytes < WS_NEED || in_sizes[0] != M * K ||
      in_sizes[1] != K * N || out_size != NR * M * (N / NR)) {
    fprintf(stderr, "unexpected problem shape\n");
    return;
  }
  static bool init = false;
  if (!init) {
    HIP_CHECK(hipFuncSetAttribute((const void*)fused_gemm_a2a, hipFuncAttributeMaxDynamicSharedMemorySize, LDS_BYTES));
    init = true;
  }
  Params p;
  p.x = (const float*)d_in[0];
  p.w = (const float*)d_in[1];
  for (int i = 0; i < NR; i++) { p.out[i] = (unsigned char*)d_out_all[i]; p.ws[i] = (unsigned char*)d_ws_all[i]; }
  p.self = (unsigned char*)d_ws_all[rank];
  p.rank = rank;
  p.pad = 0;
  hipLaunchKernelGGL(fused_gemm_a2a, dim3(NWG), dim3(512), LDS_BYTES, stream, p);
  HIP_CHECK(hipGetLastError());
}
```

```cpp
#include <hip/hip_runtime.h>
#include <rccl/rccl.h>
#include <cstdio>
#include <utility>

#define HIP_CHECK(expr) do { hipError_t _e = (expr); if (_e != hipSuccess) { \
    fprintf(stderr, "HIP error %s at %s:%d\n", hipGetErrorString(_e), __FILE__, __LINE__); \
    return; } } while (0)

typedef unsigned u32x4 __attribute__((ext_vector_type(4)));
typedef unsigned u32x2 __attribute__((ext_vector_type(2)));
typedef float f32x4 __attribute__((ext_vector_type(4)));
typedef float f32x2 __attribute__((ext_vector_type(2)));
typedef __bf16 bf16x2 __attribute__((ext_vector_type(2)));
typedef unsigned long long u64;
#define GAS __attribute__((address_space(1)))
#define LAS __attribute__((address_space(3)))
#define DI __device__ __forceinline__
#pragma clang diagnostic ignored "-Wint-to-pointer-cast"

constexpr int NR = 8;
constexpr int M = 512, N = 4096, K = 4096;
constexpr int NWG = 256;
constexpr int NSTEP = 32;
constexpr int WS_TMO = 0;
constexpr int WS_CNT = 4096;
constexpr int WS_LOC = 8192;
constexpr int WS_AFLAG = 64 * 1024;
constexpr int WS_MBOX = 96 * 1024;
constexpr int WS_BOOK = 120 * 1024;
constexpr int WS_DONE = 128 * 1024;
constexpr int WS_EF = 256 * 1024;
constexpr int WS_XFLAG = 576 * 1024;
constexpr int WS_AT = 1 << 20;
constexpr int WS_X = 5 << 20;
constexpr int X_BYTES = NWG * 3 * 4 * 4096;
constexpr int WS_E = 18 << 20;
constexpr size_t WS_NEED = (size_t)WS_E + 2 * 8 * 128 * 2048;
constexpr u64 DEADLINE = 100000000ull;
constexpr int NWS = 19;
constexpr int L_F = NWS * 8192;
constexpr int L_SC = L_F - 16384;
constexpr int LDS_BYTES = L_F + 256;
static_assert(LDS_BYTES <= 160 * 1024, "LDS");
enum : int {
  F_FULLW = 0,
  F_DONE = 4,
  F_RDY = 32,
  F_MAX = 12,
  F_LMAX = 16,
  F_GOK = 20,
  F_CALL = 22, F_ST, F_DR, F_FS, F_CNT, F_GS, F_LCNT, F_TS, F_XS
};
constexpr int AUX_SYS = 17;
constexpr unsigned XBOOK = 70;
constexpr unsigned EGAP = 4;

struct Params {
  const float* x;
  const float* w;
  unsigned char* self;
  int rank;
  int pad;
  unsigned char* out[NR];
  unsigned char* ws[NR];
};

#define RLX_AGENT __ATOMIC_RELAXED, __HIP_MEMORY_SCOPE_AGENT
#define RLX_SYS __ATOMIC_RELAXED, __HIP_MEMORY_SCOPE_SYSTEM
#define CFENCE() asm volatile("" ::: "memory")
#define SB() __builtin_amdgcn_sched_barrier(0)
#define UNI(x) __builtin_amdgcn_readfirstlane(x)

DI unsigned pk(float a, float b) {
  f32x2 v = {a, b};
  bf16x2 r = __builtin_convertvector(v, bf16x2);
  return __builtin_bit_cast(unsigned, r);
}
DI float bf_lo(unsigned v) { return __builtin_bit_cast(float, v << 16); }
DI float bf_hi(unsigned v) { return __builtin_bit_cast(float, v & 0xffff0000u); }
DI unsigned q4(float a, float b, float c, float d, float rs) {
  const unsigned t0 = __builtin_bit_cast(unsigned, __builtin_fmaf(a, rs, 12582912.0f));
  const unsigned t1 = __builtin_bit_cast(unsigned, __builtin_fmaf(b, rs, 12582912.0f));
  const unsigned t2 = __builtin_bit_cast(unsigned, __builtin_fmaf(c, rs, 12582912.0f));
  const unsigned t3 = __builtin_bit_cast(unsigned, __builtin_fmaf(d, rs, 12582912.0f));
  return __builtin_amdgcn_perm(t1, t0, 0x0c0c0400u) | __builtin_amdgcn_perm(t3, t2, 0x04000c0cu);
}
DI u64 now() { return __builtin_amdgcn_s_memrealtime(); }
DI unsigned wave_max(unsigned v) {
  v = max(v, (unsigned)__builtin_amdgcn_update_dpp(0, (int)v, 0x111, 0xf, 0xf, true));
  v = max(v, (unsigned)__builtin_amdgcn_update_dpp(0, (int)v, 0x112, 0xf, 0xf, true));
  v = max(v, (unsigned)__builtin_amdgcn_update_dpp(0, (int)v, 0x114, 0xf, 0xf, true));
  v = max(v, (unsigned)__builtin_amdgcn_update_dpp(0, (int)v, 0x118, 0xf, 0xf, true));
  const unsigned a = __builtin_amdgcn_readlane(v, 15), b = __builtin_amdgcn_readlane(v, 31);
  const unsigned c = __builtin_amdgcn_readlane(v, 47), d = __builtin_amdgcn_readlane(v, 63);
  return max(max(a, b), max(c, d));
}
DI bool expired(GAS unsigned* tmo, unsigned tv, unsigned call, u64 t0) {
  if (UNI(tv) == call) return true;
  if (now() - t0 > DEADLINE) { __hip_atomic_store(tmo, call, RLX_AGENT); return true; }
  return false;
}
DI void store_sys(const void* p, u32x4 v) {
  asm volatile("global_store_dwordx4 %0, %1, off sc0 sc1\n\ts_nop 1" ::"v"(p), "v"(v) : "memory");
}
DI void lds_put(unsigned addr, unsigned v) { asm volatile("ds_write_b32 %0, %1" ::"v"(addr), "v"(v) : "memory"); }
DI unsigned lds_get(unsigned addr) {
  unsigned v;
  asm volatile("ds_read_b32 %0, %1\n\ts_waitcnt lgkmcnt(0)" : "=v"(v) : "v"(addr) : "memory");
  return UNI(v);
}
DI u32x2 lds_get2(unsigned addr) {
  u32x2 v;
  asm volatile("ds_read_b64 %0, %1\n\ts_waitcnt lgkmcnt(0)" : "=v"(v) : "v"(addr) : "memory");
  return v;
}
DI u32x4 lds_get4(unsigned addr) {
  u32x4 v;
  asm volatile("ds_read_b128 %0, %1\n\ts_waitcnt lgkmcnt(0)" : "=v"(v) : "v"(addr) : "memory");
  return v;
}
DI unsigned lds_count(unsigned cnt) {
  unsigned old;
  asm volatile("s_mov_b64 exec, 1\n\tds_add_rtn_u32 %0, %1, %2\n\ts_mov_b64 exec, -1\n\ts_waitcnt lgkmcnt(0)"
               : "=&v"(old) : "v"(cnt), "v"(1u) : "memory");
  return UNI(old);
}
DI void wait8(const GAS unsigned* f8, unsigned call, GAS unsigned* tmo) {
  const u64 t0 = now();
  for (unsigned it = 1;; it++) {
    unsigned a0, a1, a2, a3, a4, a5, a6, a7;
    asm volatile(
        "s_load_dword %0, %8, 0x0 glc\n\ts_load_dword %1, %8, 0x80 glc\n\ts_load_dword %2, %8, 0x100 glc\n\t"
        "s_load_dword %3, %8, 0x180 glc\n\ts_load_dword %4, %8, 0x200 glc\n\ts_load_dword %5, %8, 0x280 glc\n\t"
        "s_load_dword %6, %8, 0x300 glc\n\ts_load_dword %7, %8, 0x380 glc\n\ts_waitcnt lgkmcnt(0)"
        : "=&s"(a0), "=&s"(a1), "=&s"(a2), "=&s"(a3), "=&s"(a4), "=&s"(a5), "=&s"(a6), "=&s"(a7)
        : "s"(f8) : "memory");
    if (min(min(min(a0, a1), min(a2, a3)), min(min(a4, a5), min(a6, a7))) >= call) break;
    if ((it & 255u) == 0u) {
      unsigned tv;
      asm volatile("global_load_dword %0, %1, off sc1\n\ts_waitcnt vmcnt(0)" : "=&v"(tv) : "v"(tmo) : "memory");
      if (UNI(tv) == call) break;
      if (now() - t0 > DEADLINE) {
        asm volatile("global_store_dword %0, %1, off sc1\n\ts_waitcnt vmcnt(0)" ::"v"(tmo), "v"(call) : "memory");
        break;
      }
    }
    __builtin_amdgcn_s_sleep(2);
  }
}

enum : int { O_, P_, CW_, G_, WS_, WF_, Q_ };
struct Op { int k, a; };
#define RND(r) {G_, 4 * (r) - 6}, {P_, r}, {O_, (r) + 1}, {CW_, r}
constexpr Op SCHED[] = {
  {O_, 0}, {WS_, 0}, {P_, 0}, {WF_, 0}, {O_, 1}, {CW_, 0}, {P_, 1}, {O_, 2}, {CW_, 1}, {Q_, 0},
  RND(2), RND(3), RND(4), RND(5), RND(6), {G_, 22}, {P_, 7}, {CW_, 7},
};
constexpr int NOPS = sizeof(SCHED) / sizeof(Op);
constexpr int NSET = 5;
constexpr bool is_ld(int k) { return k == O_ || k == P_; }
constexpr int find_op(int k, int a) { for (int i = 0; i < NOPS; i++) if (SCHED[i].k == k && SCHED[i].a == a) return i; return -1; }
constexpr int younger(int t, int i) { int n = 0; for (int j = t + 1; j < i; j++) n += is_ld(SCHED[j].k) ? 8 : 0; return n; }
constexpr int set_of(int op) {
  int owner[NSET] = {-1, -1, -1, -1, -1};
  for (int i = 0; i < NOPS; i++) {
    if (is_ld(SCHED[i].k)) {
      int s = -1;
      for (int q = 0; q < NSET; q++) if (owner[q] < 0) { s = q; break; }
      if (s < 0) return -1;
      owner[s] = SCHED[i].a;
      if (i == op) return s;
    } else if (SCHED[i].k == CW_) {
      for (int q = 0; q < NSET; q++) if (owner[q] == SCHED[i].a) owner[q] = -1;
    }
  }
  return -1;
}
constexpr bool can_reach(int n, int i) { return n <= 0 || find_op(CW_, n / 4) < i; }
constexpr bool sched_ok() {
  for (int r = 0; r < 8; r++) {
    const int o = find_op(O_, r), p = find_op(P_, r), c = find_op(CW_, r);
    if (o < 0 || p < o || c < p || set_of(o) < 0 || set_of(p) < 0 || younger(o, c) > 63) return false;
    if (r > 0 && c < find_op(CW_, r - 1)) return false;
    if (!can_reach(4 * r + 3 - NWS + 1, c)) return false;
    if (4 * r + 3 >= NWS - 2 && c < find_op(WS_, 0)) return false;
  }
  for (int i = 0; i < NOPS; i++) if (SCHED[i].k == G_ && (!can_reach(SCHED[i].a, i) || i < find_op(Q_, 0))) return false;
  return true;
}
static_assert(sched_ok(), "loader schedule");

struct Loader {
  f32x4 wb[NSET][8];
  __amdgpu_buffer_rsrc_t rW;
  int vW[8];
  unsigned sW, own, ldsW, l, fl;
  const GAS unsigned* afl;
  GAS unsigned* tmo;
  const unsigned char *po, *pw, *pe;
  bool rt;
};
#define WAIT_SET(S, N)                                                                               \
  asm volatile("s_waitcnt vmcnt(%8)"                                                                 \
               : "+v"(L.wb[S][0]), "+v"(L.wb[S][1]), "+v"(L.wb[S][2]), "+v"(L.wb[S][3]),             \
                 "+v"(L.wb[S][4]), "+v"(L.wb[S][5]), "+v"(L.wb[S][6]), "+v"(L.wb[S][7])              \
               : "i"(N) : "memory")
template <int I> DI void run_op(Loader& L) {
  constexpr int k = SCHED[I].k, a = SCHED[I].a;
  if constexpr (is_ld(k)) {
    constexpr int s = set_of(I);
    const unsigned so = (unsigned)a * (128u * N * 4u) + L.sW + ((k == O_) ? L.own : (256u - L.own));
    asm volatile(
        "buffer_load_dwordx4 %0, %8, %16, %17 offen\n\t"
        "buffer_load_dwordx4 %1, %9, %16, %17 offen\n\t"
        "buffer_load_dwordx4 %2, %10, %16, %17 offen\n\t"
        "buffer_load_dwordx4 %3, %11, %16, %17 offen\n\t"
        "buffer_load_dwordx4 %4, %12, %16, %17 offen\n\t"
        "buffer_load_dwordx4 %5, %13, %16, %17 offen\n\t"
        "buffer_load_dwordx4 %6, %14, %16, %17 offen\n\t"
        "buffer_load_dwordx4 %7, %15, %16, %17 offen"
        : "=&v"(L.wb[s][0]), "=&v"(L.wb[s][1]), "=&v"(L.wb[s][2]), "=&v"(L.wb[s][3]),
          "=&v"(L.wb[s][4]), "=&v"(L.wb[s][5]), "=&v"(L.wb[s][6]), "=&v"(L.wb[s][7])
        : "v"(L.vW[0]), "v"(L.vW[1]), "v"(L.vW[2]), "v"(L.vW[3]), "v"(L.vW[4]), "v"(L.vW[5]), "v"(L.vW[6]), "v"(L.vW[7]),
          "s"(L.rW), "s"(so)
        : "memory");
  } else if constexpr (k == CW_) {
    constexpr int io = find_op(O_, a), ip = find_op(P_, a), so = set_of(io), sp = set_of(ip);
    const unsigned t = 4u * a + L.l;
    if constexpr (4 * a + 3 >= NWS) {
      const int need = (int)t - NWS + 1;
      while (true) {
        const u32x4 d = lds_get4(L.fl + F_DONE * 4);
        if ((int)UNI(min(min(d.x, d.y), min(d.z, d.w))) >= need) break;
        __builtin_amdgcn_s_sleep(1);
      }
    }
    unsigned slot = (4u * a) % NWS + L.l;
    if (slot >= (unsigned)NWS) slot -= NWS;
    const unsigned base = L.ldsW + slot * 8192u;
    auto conv = [&](f32x4(&w)[8], unsigned dst) {
#pragma unroll
      for (int c = 0; c < 4; c++) {
        u32x4 bf;
        bf.x = pk(w[0][c], w[1][c]); bf.y = pk(w[2][c], w[3][c]);
        bf.z = pk(w[4][c], w[5][c]); bf.w = pk(w[6][c], w[7][c]);
        *(LAS u32x4*)(dst + c * 1024) = bf;
      }
    };
    WAIT_SET(so, younger(io, I));
    conv(L.wb[so], base + L.own * 16u);
    WAIT_SET(sp, younger(ip, I));
    conv(L.wb[sp], base + (4096u - L.own * 16u));
    lds_put(L.fl + (F_FULLW + L.l) * 4, t + 1u);
  } else if constexpr (k == G_) {
    while (lds_get(L.fl + (F_DONE + L.l) * 4) < (unsigned)a) __builtin_amdgcn_s_sleep(1);
  } else if constexpr (k == WS_) {
    if (L.rt) {
      unsigned r0, r1, r2, r3;
      asm volatile("s_load_dword %0, %4, 0x0\n\ts_load_dword %1, %5, 0x20000\n\ts_load_dword %2, %6, 0x0\n\t"
                   "s_load_dword %3, %7, 0x0\n\ts_waitcnt lgkmcnt(0)"
                   : "=&s"(r0), "=&s"(r1), "=&s"(r2), "=&s"(r3) : "s"(L.po), "s"(L.pw), "s"(L.pe), "s"(L.pe + (2 << 20)) : "memory");
    }
    while (lds_get(L.fl + F_ST * 4) < 4u) __builtin_amdgcn_s_sleep(1);
  } else if constexpr (k == WF_) {
    while (lds_get(L.fl + F_FS * 4) == 0u) __builtin_amdgcn_s_sleep(1);
  } else if (L.l != 0u) {
    wait8(L.afl + L.l * 8u * 32u, lds_get(L.fl + F_CALL * 4), L.tmo);
    lds_put(L.fl + (F_RDY + L.l) * 4, 1u);
  }
}
template <int... I> DI void run_all(Loader& L, std::integer_sequence<int, I...>) { (run_op<I>(L), ...); }


__global__ void __launch_bounds__(512, 2) fused_gemm_a2a(Params p) {
  extern __shared__ __attribute__((aligned(16))) unsigned char smem[];
  const int tid = threadIdx.x, lane = tid & 63;
  const int w = UNI(tid >> 6);
  const int b = blockIdx.x;
  const int xcd = b & 7, j = b >> 3;
  const int kq = xcd >> 1, mb2 = j & 1;
  const int nb = (xcd & 1) * 16 + (j >> 1);
  const int grp = mb2 * 4 + kq;
  const unsigned lds0 = (unsigned)(size_t)(LAS unsigned char*)smem;
  const unsigned fl = lds0 + L_F;
  f32x4 xa[8];
  unsigned cv = 0, tw = 0;
  if (w < 4) {
    const __amdgpu_buffer_rsrc_t rX = __builtin_amdgcn_make_buffer_rsrc((void*)p.x, 0, M * K * 4, 0x00020000);
    const int vX = (lane >> 3) * (K * 4) + (lane & 7) * 16;
    const int so = (mb2 * 256 + w * 64) * (K * 4) + (kq * 1024 + nb * 32) * 4;
#pragma unroll
    for (int i = 0; i < 8; i++)
      xa[i] = __builtin_bit_cast(f32x4, __builtin_amdgcn_raw_buffer_load_b128(rX, vX, so + i * 8 * (K * 4), 0));
    SB();
    cv = __hip_atomic_load((GAS unsigned*)(p.self + WS_CNT) + b * 4 + w, RLX_AGENT);
    {
      const int t = lane & 15;
      const unsigned char* a = p.self + WS_AT + ((grp * NSTEP + nb) * 4 + w) * 4096;
      if (t == 1) a = p.self + WS_AFLAG + (grp * NSTEP + nb) * 128;
      if (t == 2) a = p.self + WS_XFLAG + b * 12 * 128;
      if (t == 3) a = p.self + WS_AT + grp * NSTEP * 16384;
      if (t >= 4 && t < 8) a = p.self + WS_X + (size_t)((b ^ ((t & 3) << 1)) * 12) * 4096;
      if (t >= 8) a = (const unsigned char*)p.w + (size_t)(kq * 1024 + (t - 8) * 128) * (N * 4) + nb * 512 + mb2 * 256;
      tw = __hip_atomic_load((GAS unsigned*)a, RLX_AGENT);
    }
    if (tid < 64) *(LAS unsigned*)(fl + tid * 4) = 0u;
    asm volatile("s_waitcnt lgkmcnt(0)" ::: "memory");
    SB();
  }
  __builtin_amdgcn_s_barrier();
  CFENCE();
  SB();
  const int rank = p.rank;
  unsigned char* ws = p.self;
  GAS unsigned* tmo = (GAS unsigned*)(ws + WS_TMO);
  GAS unsigned* afl = (GAS unsigned*)(ws + WS_AFLAG) + grp * NSTEP * 32;
  const int c = w & 3;
  const unsigned dst = (unsigned)(nb >> 2) & 7u;
  const int idx = ((nb & 3) * 2 + mb2) * 4 + kq;
  const int slot = idx * 4 + c;
  const bool remote = dst != (unsigned)rank;
  const unsigned me = (unsigned)rank;
  const int rowin = mb2 * 256 + c * 64 + kq * 16 + (lane & 15), colb = (nb & 3) * 128 + 16 * (lane >> 4);

  if (__builtin_expect(w < 4, 1)) {
    const unsigned call = UNI(cv) + 1u;
    {
      const unsigned sc = lds0 + L_SC + c * 4096;
      const unsigned tb = sc + ((lane & 7) >> 1) * 256 + (lane >> 3) * 16 + (lane & 1) * 8;
#pragma unroll
      for (int i = 0; i < 8; i++) {
        u32x2 o = {pk(xa[i][0], xa[i][1]), pk(xa[i][2], xa[i][3])};
        *(LAS u32x2*)(tb + (((unsigned)(i >> 1)) ^ (unsigned)kq) * 1024u + (i & 1) * 128) = o;
      }
      asm volatile("" ::"v"(tw));
      u32x4 fr[4];
#pragma unroll
      for (int f = 0; f < 4; f++) fr[f] = *(LAS u32x4*)(sc + f * 1024 + lane * 16);
      unsigned char* pub = ws + WS_AT + ((grp * NSTEP + nb) * 4 + c) * 4096 + lane * 16;
#pragma unroll
      for (int f = 0; f < 4; f++) store_sys(pub + f * 1024, fr[f]);
      lds_put(fl + F_CALL * 4, call);
      lds_count(fl + F_ST * 4);
      asm volatile("s_waitcnt vmcnt(0)" ::: "memory");
      if (lds_count(fl + F_DR * 4) == 3u) {
        if (lane == 0) __hip_atomic_store(afl + nb * 32, call, RLX_SYS);
        lds_put(fl + F_FS * 4, 1u);
      }
      if (lane == 0) __hip_atomic_store((GAS unsigned*)(ws + WS_CNT) + b * 4 + c, call, RLX_AGENT);
      if (c == 0) {
        while (lds_get(fl + F_FS * 4) == 0u) {}
        wait8(afl, call, tmo);
        lds_put(fl + (F_RDY + 4) * 4, 1u);
        lds_put(fl + F_RDY * 4, 1u);
      }
    }
    f32x4 acc[4][8];
#pragma unroll
    for (int m = 0; m < 4; m++)
#pragma unroll
      for (int n = 0; n < 8; n++) { acc[m][n] = f32x4{0, 0, 0, 0}; asm volatile("" : "+v"(acc[m][n])); }
    const unsigned wBase = lds0 + lane * 16;
    unsigned pfw = fl + F_FULLW * 4, pdone = fl + (F_DONE + c) * 4, vdone = 0;
    asm volatile("" : "+v"(pfw), "+v"(pdone), "+v"(vdone));
    const __amdgpu_buffer_rsrc_t rA = __builtin_amdgcn_make_buffer_rsrc(
        (void*)(ws + WS_AT + (grp * NSTEP * 4 + c) * 4096), 0, (NSTEP - 1) * 16384 + 4096, 0x00020000);
    int vA[4];
#pragma unroll
    for (int m = 0; m < 4; m++) { vA[m] = lane * 16 + m * 1024; asm volatile("" : "+v"(vA[m])); }
    auto ldA = [&](int m, unsigned t) { return __builtin_amdgcn_raw_buffer_load_b128(rA, vA[m], t * 16384u, AUX_SYS); };
    u32x4 A[5][4], B[4];
#define SETOF(t) ((((t) & 7) == 5) ? 4 : ((t) & 3))
    unsigned fw;
    while (lds_get(fl + F_RDY * 4) == 0u) __builtin_amdgcn_s_sleep(1);
#pragma unroll
    for (int d = 0; d < 2; d++)
#pragma unroll
      for (int m = 0; m < 4; m++) { A[d][m] = ldA(m, d); SB(); }
    while (lds_get(pfw) < 1u) {}
    __builtin_amdgcn_s_setprio(3);
    B[0] = *(LAS u32x4*)(wBase);
    B[1] = *(LAS u32x4*)(wBase + 1024);
    B[2] = *(LAS u32x4*)(wBase + 2048);
    CFENCE(); fw = *(LAS unsigned*)(pfw + 4); CFENCE();
    unsigned woff = 0, wc = wBase, rdy = 0, prdy = fl + (F_RDY + 1) * 4;
    asm volatile("" : "+v"(prdy));
    auto mf = [&](int m, int n, int q) {
      asm volatile("v_mfma_f32_16x16x32_bf16 %0, %1, %2, %0" : "+v"(acc[m][n]) : "v"(B[n & 3]), "v"(A[q][m]));
    };
    auto step = [&](unsigned s, int q8) {
      const int q = q8 & 3;
      unsigned wn = 0, sf = 0, nd = 0;
#pragma unroll
      for (int n = 0; n < 8; n++) {
        mf(0, n, SETOF(q8)); SB();
        if (n == 0 && q8 == 5) {
          if (__builtin_expect(UNI(rdy) == 0u, 0)) while (lds_get(prdy) == 0u) {}
          prdy += 4u;
          CFENCE();
        }
        if (q == 1 && n < 4) { A[SETOF(q8 + 3)][n] = ldA(n, s + 3u); SB(); }
        if (n < 5) B[(n + 3) & 3] = *(LAS u32x4*)(wc + (n + 3) * 1024);
        else B[(n + 3) & 3] = *(LAS u32x4*)(wn + (n - 5) * 1024);
        SB(); mf(1, n, SETOF(q8)); SB();
        if (n == 1) { woff = (woff == (NWS - 1) * 8192u) ? 0u : woff + 8192u; wn = wBase + woff; asm volatile("" : "+v"(wn)); }
        if (n == 2) sf = UNI(fw);
        if (n == 4) lds_put(pdone, vdone);
        if (n == 7) { CFENCE(); fw = *(LAS unsigned*)(pfw + ((q + 2) & 3) * 4); CFENCE(); }
        SB(); mf(2, n, SETOF(q8)); SB();
        if (q == 0) A[SETOF(q8 + 2 + (n >> 2))][n & 3] = ldA(n & 3, s + 2u + (n >> 2));
        if (q == 1 && n < 4) A[SETOF(q8 + 4)][n] = ldA(n, s + 4u);
        if (n == 1) { vdone += 1; asm volatile("" : "+v"(vdone)); }
        if (n == 3) { nd = (q8 == 7 && s == NSTEP - 1u) ? 0u : s + 2u; asm volatile("" : "+s"(nd)); }
        if (n == 5 && q8 == 4) { CFENCE(); rdy = *(LAS unsigned*)(prdy); CFENCE(); }
        SB(); mf(3, n, SETOF(q8)); SB();
        if (n == 3) {
          if (__builtin_expect(sf < nd, 0))
            while (lds_get(pfw + ((q + 1) & 3) * 4) < nd) {}
          CFENCE();
          SB();
        }
      }
      wc = wn;
    };
    asm volatile("s_nop 7");
    for (unsigned s = 0; s < (unsigned)NSTEP; s += 8) {
      step(s, 0); step(s + 1, 1); step(s + 2, 2); step(s + 3, 3);
      step(s + 4, 4); step(s + 5, 5); step(s + 6, 6); step(s + 7, 7);
    }
    asm volatile("s_nop 15\n\ts_nop 15");
    SB();
    __builtin_amdgcn_s_setprio(0);
    const __amdgpu_buffer_rsrc_t rE = __builtin_amdgcn_make_buffer_rsrc((void*)(ws + WS_X), 0, X_BYTES, 0x00020000);
#pragma unroll
    for (int m = 1; m < 4; m++) {
      const int so = (((b ^ (m << 1)) * 3 + (m - 1)) * 4 + c) * 4096;
#pragma unroll
      for (int t = 0; t < 4; t++) {
        u32x4 o;
        o.x = pk(acc[m][2 * t][0], acc[m][2 * t][1]); o.y = pk(acc[m][2 * t][2], acc[m][2 * t][3]);
        o.z = pk(acc[m][2 * t + 1][0], acc[m][2 * t + 1][1]); o.w = pk(acc[m][2 * t + 1][2], acc[m][2 * t + 1][3]);
        __builtin_amdgcn_raw_buffer_store_b128(o, rE, lane * 16, so + t * 1024, AUX_SYS);
      }
    }
    GAS unsigned* xfl = (GAS unsigned*)(ws + WS_XFLAG);
    asm volatile("s_waitcnt vmcnt(0)" ::: "memory");
    if (lane < 3) __hip_atomic_store(xfl + (((b ^ ((lane + 1) << 1)) * 4 + c) * 3 + lane) * 32, call, RLX_SYS);
    {
      const u64 t0 = now();
      while (true) {
        const unsigned v = __hip_atomic_load(xfl + ((b * 4 + c) * 3 + min(lane, 2)) * 32, RLX_SYS);
        const unsigned tv = __hip_atomic_load(tmo, RLX_AGENT);
        if (__all(v >= call) || expired(tmo, tv, call, t0)) break;
      }
      __builtin_amdgcn_fence(__ATOMIC_ACQUIRE, "wavefront");
    }
    lds_count(fl + F_XS * 4);
    u32x4 in[3][4];
#pragma unroll
    for (int m = 0; m < 3; m++)
#pragma unroll
      for (int t = 0; t < 4; t++)
        in[m][t] = __builtin_amdgcn_raw_buffer_load_b128(rE, lane * 16, ((b * 3 + m) * 4 + c) * 4096 + t * 1024, AUX_SYS);
    float am = 0.f;
#pragma unroll
    for (int t = 0; t < 4; t++) {
#pragma unroll
      for (int m = 0; m < 3; m++)
#pragma unroll
        for (int e = 0; e < 4; e++) {
          const unsigned v = in[m][t][e];
          acc[0][2 * t + (e >> 1)][(e & 1) * 2] += bf_lo(v);
          acc[0][2 * t + (e >> 1)][(e & 1) * 2 + 1] += bf_hi(v);
        }
#pragma unroll
      for (int e = 0; e < 8; e++) am = fmaxf(am, fabsf(acc[0][2 * t + (e >> 2)][e & 3]));
    }
    lds_put(fl + (F_MAX + c) * 4, wave_max(__builtin_bit_cast(unsigned, am)));
    if (lds_count(fl + F_CNT * 4) == 3u) {
      const u32x4 m4 = lds_get4(fl + F_MAX * 4);
      const unsigned mx = max(max(m4.x, m4.y), max(m4.z, m4.w));
      if (lane == 0) __hip_atomic_store((GAS u64*)(ws + WS_LOC) + b * 16, ((u64)call << 32) | (u64)mx, RLX_SYS);
      lds_put(fl + F_GS * 4, 1u);
    }
    const unsigned par = call & 1u;
    u32x4 h0, h1;
    h0.x = pk(acc[0][0][0], acc[0][1][0]); h0.y = pk(acc[0][2][0], acc[0][3][0]);
    h0.z = pk(acc[0][0][1], acc[0][1][1]); h0.w = pk(acc[0][2][1], acc[0][3][1]);
    h1.x = pk(acc[0][0][2], acc[0][1][2]); h1.y = pk(acc[0][2][2], acc[0][3][2]);
    h1.z = pk(acc[0][0][3], acc[0][1][3]); h1.w = pk(acc[0][2][3], acc[0][3][3]);
    unsigned char* optr = p.out[dst] + (size_t)(rank * M + rowin) * 512 + colb;
    GAS unsigned* dnp = (GAS unsigned*)(p.ws[dst] + WS_DONE) + (slot * 8 + me) * 32;
    GAS unsigned* efp = (GAS unsigned*)(p.ws[dst] + WS_EF) + ((par * 128 + slot) * 8 + me) * 32;
    asm volatile("" : "+v"(optr), "+v"(dnp), "+v"(efp));
    bool sent = false;
    u32x2 gv;
    {
      unsigned ts = 0;
      while (gv = lds_get2(fl + F_GOK * 4), UNI(gv.x) == 0u) {
        if (remote && ts == 0u && lds_get(fl + F_GS * 4) != 0u) ts = lds_get(fl + F_TS * 4);
        if (ts != 0u && !sent && (int)((unsigned)now() - (ts + (unsigned)c * EGAP)) >= 0) {
          unsigned char* ep = p.ws[dst] + WS_E + (size_t)((par * 8 + me) * 128 + slot) * 2048 + lane * 16;
          store_sys(ep, h0);
          store_sys(ep + 1024, h1);
          asm volatile("s_waitcnt vmcnt(0)" ::: "memory");
          if (lane == 0) __hip_atomic_store(efp, 2u * call + 1u, RLX_SYS);
          sent = true;
        }
      }
    }
    const unsigned rsb = gv.y;
    const float rs = __builtin_bit_cast(float, rsb);
    const bool failed = gv.x != 1u;
    const u32x4 poison = {0x80808080u, 0x80808080u, 0x80808080u, 0x80808080u};
    {
      u32x4 o;
#pragma unroll
      for (int rg = 0; rg < 4; rg++) o[rg] = q4(acc[0][4][rg], acc[0][5][rg], acc[0][6][rg], acc[0][7][rg], rs);
      store_sys(optr + 64, failed ? poison : o);
    }
    if (!sent) {
      u32x4 o;
      o.x = q4(bf_lo(h0.x), bf_hi(h0.x), bf_lo(h0.y), bf_hi(h0.y), rs);
      o.y = q4(bf_lo(h0.z), bf_hi(h0.z), bf_lo(h0.w), bf_hi(h0.w), rs);
      o.z = q4(bf_lo(h1.x), bf_hi(h1.x), bf_lo(h1.y), bf_hi(h1.y), rs);
      o.w = q4(bf_lo(h1.z), bf_hi(h1.z), bf_lo(h1.w), bf_hi(h1.w), rs);
      store_sys(optr, failed ? poison : o);
    }
    if (remote) {
      if (!sent && lane == 0) __hip_atomic_store(efp, 2u * call, RLX_SYS);
      asm volatile("s_waitcnt vmcnt(0)" ::: "memory");
      if (lane == 0) __hip_atomic_store(dnp, call, RLX_SYS);
    }
  } else {
    {
      Loader L;
      L.l = c;
            L.rW = __builtin_amdgcn_make_buffer_rsrc((void*)p.w, 0, K * N * 4, 0x00020000);
#pragma unroll
      for (int i = 0; i < 8; i++) L.vW[i] = ((lane >> 4) * 8 + i) * (N * 4) + (lane & 15) * 16;
      L.sW = (unsigned)(kq * 1024 + c * 32) * (N * 4u) + (unsigned)nb * 512u;
      L.own = (unsigned)mb2 * 256u;
      L.ldsW = lds0 + lane * 16;
      L.fl = fl;
      L.afl = afl;
      L.tmo = tmo;
      L.po = p.out[j & 7] + (size_t)rank * M * 512;
      L.pw = p.ws[j & 7];
      L.pe = p.ws[j & 7] + WS_E + (size_t)rank * 128 * 2048;
      L.rt = c == 3 && j < NR;
      run_all(L, std::make_integer_sequence<int, NOPS>{});
    }
    const unsigned call = lds_get(fl + F_CALL * 4);
    const unsigned par = call & 1u;
    const u64 t0 = now();
    if (c == 2 && remote) {
      while (lds_get(fl + F_XS * 4) < 4u) __builtin_amdgcn_s_sleep(2);
      GAS u64* bk = (GAS u64*)(ws + WS_BOOK) + dst * 16;
      u64 tb = 0;
      if (lane == 0) {
        __hip_atomic_fetch_max(bk, now() + XBOOK, RLX_AGENT);
        tb = __hip_atomic_fetch_add(bk, (u64)(4u * EGAP), RLX_AGENT);
      }
      lds_put(fl + F_TS * 4, UNI((unsigned)tb) | 1u);
    }
    if (!remote && idx == 0) {
      GAS u64* g = (GAS u64*)(ws + WS_LOC) + (c * 64 + lane) * 16;
      const int srcid = rank;
      GAS u64* mb = (GAS u64*)(p.ws[lane >> 3] + WS_MBOX) + ((par * 8 + (lane & 7)) * 8 + srcid) * 16;
      asm volatile("" : "+v"(mb));
      u64 v;
      while (true) {
        v = __hip_atomic_load(g, RLX_SYS);
        const unsigned tv = __hip_atomic_load(tmo, RLX_AGENT);
        if (__all((unsigned)(v >> 32) == call) || expired(tmo, tv, call, t0)) break;
      }
      lds_put(fl + (F_LMAX + c) * 4, wave_max((unsigned)v));
      if (lds_count(fl + F_LCNT * 4) == 3u) {
        const u32x4 m4 = lds_get4(fl + F_LMAX * 4);
        const unsigned mx = max(max(m4.x, m4.y), max(m4.z, m4.w));
        __hip_atomic_store(mb, ((u64)call << 32) | (u64)mx, RLX_SYS);
      }
    }
    if (c == 1) {
      while (lds_get(fl + F_GS * 4) == 0u) __builtin_amdgcn_s_sleep(2);
      GAS u64* g = (GAS u64*)(ws + WS_MBOX) + ((par * 8 + xcd) * 8 + (lane & 7)) * 16;
      u64 v;
      bool bad;
      while (true) {
        v = __hip_atomic_load(g, RLX_SYS);
        const unsigned tv = __hip_atomic_load(tmo, RLX_AGENT);
        bad = UNI(tv) == call;
        if (__all((unsigned)(v >> 32) == call)) break;
        if (expired(tmo, tv, call, t0)) { bad = true; break; }
      }
      const unsigned mx = wave_max((unsigned)v);
      const float amax = __builtin_bit_cast(float, mx);
      const float rs = amax > 0.f ? 127.0f / amax : 0.f;
      const u32x2 gw = {bad ? 2u : 1u, __builtin_bit_cast(unsigned, rs)};
      asm volatile("ds_write_b64 %0, %1" ::"v"(fl + F_GOK * 4), "v"(gw) : "memory");
    }
    if (remote) {
      GAS unsigned* ef = (GAS unsigned*)(ws + WS_EF) + ((par * 128 + slot) * 8 + dst) * 32;
      while (lds_get(fl + F_GS * 4) == 0u) __builtin_amdgcn_s_sleep(4);
      unsigned e;
      bool bad = false;
      while ((e = UNI(__hip_atomic_load(ef, RLX_SYS))) < 2u * call) {
        const unsigned ok = lds_get(fl + F_GOK * 4);
        if (ok == 2u || now() - t0 > DEADLINE) { __hip_atomic_store(tmo, call, RLX_AGENT); bad = true; break; }
        if (ok == 0u) __builtin_amdgcn_s_sleep(12);
      }
      if (bad || (e & 1u)) {
        __builtin_amdgcn_fence(__ATOMIC_ACQUIRE, "wavefront");
        const __amdgpu_buffer_rsrc_t rI = __builtin_amdgcn_make_buffer_rsrc(
            (void*)(ws + WS_E + (size_t)((par * 8 + dst) * 128 + slot) * 2048), 0, bad ? 0 : 2048, 0x00020000);
        const u32x4 h0 = __builtin_amdgcn_raw_buffer_load_b128(rI, lane * 16, 0, AUX_SYS);
        const u32x4 h1 = __builtin_amdgcn_raw_buffer_load_b128(rI, lane * 16, 1024, AUX_SYS);
        u32x2 gv;
        while (gv = lds_get2(fl + F_GOK * 4), UNI(gv.x) == 0u) __builtin_amdgcn_s_sleep(1);
        const unsigned rsb = gv.y;
        const float rs = __builtin_bit_cast(float, rsb);
        u32x4 o;
        o.x = q4(bf_lo(h0.x), bf_hi(h0.x), bf_lo(h0.y), bf_hi(h0.y), rs);
        o.y = q4(bf_lo(h0.z), bf_hi(h0.z), bf_lo(h0.w), bf_hi(h0.w), rs);
        o.z = q4(bf_lo(h1.x), bf_hi(h1.x), bf_lo(h1.y), bf_hi(h1.y), rs);
        o.w = q4(bf_lo(h1.z), bf_hi(h1.z), bf_lo(h1.w), bf_hi(h1.w), rs);
        if (bad || gv.x != 1u) o = u32x4{0x80808080u, 0x80808080u, 0x80808080u, 0x80808080u};
        store_sys(p.out[rank] + (size_t)(dst * M + rowin) * 512 + colb, o);
      }
    } else {
      while (lds_get(fl + F_GOK * 4) == 0u) __builtin_amdgcn_s_sleep(2);
      GAS unsigned* d = (GAS unsigned*)(ws + WS_DONE) + (slot * 8 + (lane & 7)) * 32;
      bool bad = false;
      while (true) {
        const unsigned v = __hip_atomic_load(d, RLX_SYS);
        const unsigned tv = __hip_atomic_load(tmo, RLX_AGENT);
        if (__all(v >= call || (lane & 7) == rank)) break;
        if (expired(tmo, tv, call, t0)) { bad = true; break; }
      }
      if (bad && lane == 0) p.out[rank][(size_t)(rank * M + rowin) * 512 + colb] = 0x80;
    }
  }
}

extern "C" void kernel_launch(int rank, int nranks, ncclComm_t comm, hipStream_t stream,
                              void* const* d_in, const int* in_sizes, int n_in,
                              void* d_out, int out_size, const int* in_dtypes, int out_dtype,
                              void* const* d_out_all, void* const* d_ws_all, size_t ws_bytes) {
  if (nranks != NR || rank < 0 || rank >= NR || n_in < 2 || ws_bytes < WS_NEED || in_sizes[0] != M * K ||
      in_sizes[1] != K * N || out_size != NR * M * (N / NR)) {
    fprintf(stderr, "unexpected problem shape\n");
    return;
  }
  static bool init = false;
  if (!init) {
    HIP_CHECK(hipFuncSetAttribute((const void*)fused_gemm_a2a, hipFuncAttributeMaxDynamicSharedMemorySize, LDS_BYTES));
    init = true;
  }
  Params p;
  p.x = (const float*)d_in[0];
  p.w = (const float*)d_in[1];
  for (int i = 0; i < NR; i++) { p.out[i] = (unsigned char*)d_out_all[i]; p.ws[i] = (unsigned char*)d_ws_all[i]; }
  p.self = (unsigned char*)d_ws_all[rank];
  p.rank = rank;
  p.pad = 0;
  hipLaunchKernelGGL(fused_gemm_a2a, dim3(NWG), dim3(512), LDS_BYTES, stream, p);
  HIP_CHECK(hipGetLastError());
}
```

```cpp
#include <hip/hip_runtime.h>
#include <rccl/rccl.h>
#include <cstdio>
#include <utility>

#define HIP_CHECK(expr) do { hipError_t _e = (expr); if (_e != hipSuccess) { \
    fprintf(stderr, "HIP error %s at %s:%d\n", hipGetErrorString(_e), __FILE__, __LINE__); \
    return; } } while (0)

typedef unsigned u32x4 __attribute__((ext_vector_type(4)));
typedef unsigned u32x2 __attribute__((ext_vector_type(2)));
typedef float f32x4 __attribute__((ext_vector_type(4)));
typedef float f32x2 __attribute__((ext_vector_type(2)));
typedef __bf16 bf16x2 __attribute__((ext_vector_type(2)));
typedef unsigned long long u64;
#define GAS __attribute__((address_space(1)))
#define LAS __attribute__((address_space(3)))
#define DI __device__ __forceinline__
#pragma clang diagnostic ignored "-Wint-to-pointer-cast"

constexpr int NR = 8;
constexpr int M = 512, N = 4096, K = 4096;
constexpr int NWG = 256;
constexpr int NSTEP = 32;
constexpr int WS_TMO = 0;
constexpr int WS_CNT = 4096;
constexpr int WS_LOC = 8192;
constexpr int WS_AFLAG = 64 * 1024;
constexpr int WS_MBOX = 96 * 1024;
constexpr int WS_BOOK = 120 * 1024;
constexpr int WS_DONE = 128 * 1024;
constexpr int WS_EF = 256 * 1024;
constexpr int WS_XFLAG = 576 * 1024;
constexpr int WS_AT = 1 << 20;
constexpr int WS_X = 5 << 20;
constexpr int X_BYTES = NWG * 3 * 4 * 4096;
constexpr int WS_E = 18 << 20;
constexpr size_t WS_NEED = (size_t)WS_E + 2 * 8 * 128 * 2048;
constexpr u64 DEADLINE = 100000000ull;
constexpr int NWS = 19;
constexpr int L_F = NWS * 8192;
constexpr int L_SC = L_F - 16384;
constexpr int LDS_BYTES = L_F + 256;
static_assert(LDS_BYTES <= 160 * 1024, "LDS");
enum : int {
  F_FULLW = 0,
  F_DONE = 4,
  F_RDY = 32,
  F_MAX = 12,
  F_LMAX = 16,
  F_GOK = 20,
  F_CALL = 22, F_ST, F_DR, F_FS, F_CNT, F_GS, F_LCNT, F_TS, F_XS
};
constexpr int AUX_SYS = 17;
constexpr unsigned XBOOK = 70;
constexpr unsigned EGAP = 4;

struct Params {
  const float* x;
  const float* w;
  unsigned char* self;
  int rank;
  int pad;
  unsigned char* out[NR];
  unsigned char* ws[NR];
};

#define RLX_AGENT __ATOMIC_RELAXED, __HIP_MEMORY_SCOPE_AGENT
#define RLX_SYS __ATOMIC_RELAXED, __HIP_MEMORY_SCOPE_SYSTEM
#define CFENCE() asm volatile("" ::: "memory")
#define SB() __builtin_amdgcn_sched_barrier(0)
#define UNI(x) __builtin_amdgcn_readfirstlane(x)

DI unsigned pk(float a, float b) {
  f32x2 v = {a, b};
  bf16x2 r = __builtin_convertvector(v, bf16x2);
  return __builtin_bit_cast(unsigned, r);
}
DI float bf_lo(unsigned v) { return __builtin_bit_cast(float, v << 16); }
DI float bf_hi(unsigned v) { return __builtin_bit_cast(float, v & 0xffff0000u); }
DI unsigned q4(float a, float b, float c, float d, float rs) {
  const unsigned t0 = __builtin_bit_cast(unsigned, __builtin_fmaf(a, rs, 12582912.0f));
  const unsigned t1 = __builtin_bit_cast(unsigned, __builtin_fmaf(b, rs, 12582912.0f));
  const unsigned t2 = __builtin_bit_cast(unsigned, __builtin_fmaf(c, rs, 12582912.0f));
  const unsigned t3 = __builtin_bit_cast(unsigned, __builtin_fmaf(d, rs, 12582912.0f));
  return __builtin_amdgcn_perm(t1, t0, 0x0c0c0400u) | __builtin_amdgcn_perm(t3, t2, 0x04000c0cu);
}
DI u64 now() { return __builtin_amdgcn_s_memrealtime(); }
DI unsigned wave_max(unsigned v) {
  v = max(v, (unsigned)__builtin_amdgcn_update_dpp(0, (int)v, 0x111, 0xf, 0xf, true));
  v = max(v, (unsigned)__builtin_amdgcn_update_dpp(0, (int)v, 0x112, 0xf, 0xf, true));
  v = max(v, (unsigned)__builtin_amdgcn_update_dpp(0, (int)v, 0x114, 0xf, 0xf, true));
  v = max(v, (unsigned)__builtin_amdgcn_update_dpp(0, (int)v, 0x118, 0xf, 0xf, true));
  const unsigned a = __builtin_amdgcn_readlane(v, 15), b = __builtin_amdgcn_readlane(v, 31);
  const unsigned c = __builtin_amdgcn_readlane(v, 47), d = __builtin_amdgcn_readlane(v, 63);
  return max(max(a, b), max(c, d));
}
DI bool expired(GAS unsigned* tmo, unsigned tv, unsigned call, u64 t0) {
  if (UNI(tv) == call) return true;
  if (now() - t0 > DEADLINE) { __hip_atomic_store(tmo, call, RLX_AGENT); return true; }
  return false;
}
DI void poll32(const GAS unsigned* f, const GAS unsigned* tmo, unsigned& v, unsigned& tv) {
  asm volatile("global_load_dword %0, %2, off sc0 sc1\n\tglobal_load_dword %1, %3, off sc1\n\ts_waitcnt vmcnt(0)"
               : "=&v"(v), "=&v"(tv) : "v"(f), "v"(tmo) : "memory");
}
DI void poll64(const GAS u64* f, const GAS unsigned* tmo, u64& v, unsigned& tv) {
  asm volatile("global_load_dwordx2 %0, %2, off sc0 sc1\n\tglobal_load_dword %1, %3, off sc1\n\ts_waitcnt vmcnt(0)"
               : "=&v"(v), "=&v"(tv) : "v"(f), "v"(tmo) : "memory");
}
DI void store_sys(const void* p, u32x4 v) {
  asm volatile("global_store_dwordx4 %0, %1, off sc0 sc1\n\ts_nop 1" ::"v"(p), "v"(v) : "memory");
}
DI void lds_put(unsigned addr, unsigned v) { asm volatile("ds_write_b32 %0, %1" ::"v"(addr), "v"(v) : "memory"); }
DI unsigned lds_get(unsigned addr) {
  unsigned v;
  asm volatile("ds_read_b32 %0, %1\n\ts_waitcnt lgkmcnt(0)" : "=v"(v) : "v"(addr) : "memory");
  return UNI(v);
}
DI u32x2 lds_get2(unsigned addr) {
  u32x2 v;
  asm volatile("ds_read_b64 %0, %1\n\ts_waitcnt lgkmcnt(0)" : "=v"(v) : "v"(addr) : "memory");
  return v;
}
DI u32x4 lds_get4(unsigned addr) {
  u32x4 v;
  asm volatile("ds_read_b128 %0, %1\n\ts_waitcnt lgkmcnt(0)" : "=v"(v) : "v"(addr) : "memory");
  return v;
}
DI unsigned lds_count(unsigned cnt) {
  unsigned old;
  asm volatile("s_mov_b64 exec, 1\n\tds_add_rtn_u32 %0, %1, %2\n\ts_mov_b64 exec, -1\n\ts_waitcnt lgkmcnt(0)"
               : "=&v"(old) : "v"(cnt), "v"(1u) : "memory");
  return UNI(old);
}
DI void wait8(const GAS unsigned* f8, unsigned call, GAS unsigned* tmo) {
  const u64 t0 = now();
  for (unsigned it = 1;; it++) {
    unsigned a0, a1, a2, a3, a4, a5, a6, a7;
    asm volatile(
        "s_load_dword %0, %8, 0x0 glc\n\ts_load_dword %1, %8, 0x80 glc\n\ts_load_dword %2, %8, 0x100 glc\n\t"
        "s_load_dword %3, %8, 0x180 glc\n\ts_load_dword %4, %8, 0x200 glc\n\ts_load_dword %5, %8, 0x280 glc\n\t"
        "s_load_dword %6, %8, 0x300 glc\n\ts_load_dword %7, %8, 0x380 glc\n\ts_waitcnt lgkmcnt(0)"
        : "=&s"(a0), "=&s"(a1), "=&s"(a2), "=&s"(a3), "=&s"(a4), "=&s"(a5), "=&s"(a6), "=&s"(a7)
        : "s"(f8) : "memory");
    if (min(min(min(a0, a1), min(a2, a3)), min(min(a4, a5), min(a6, a7))) >= call) break;
    if ((it & 255u) == 0u) {
      unsigned tv;
      asm volatile("global_load_dword %0, %1, off sc1\n\ts_waitcnt vmcnt(0)" : "=&v"(tv) : "v"(tmo) : "memory");
      if (UNI(tv) == call) break;
      if (now() - t0 > DEADLINE) {
        asm volatile("global_store_dword %0, %1, off sc1\n\ts_waitcnt vmcnt(0)" ::"v"(tmo), "v"(call) : "memory");
        break;
      }
    }
    __builtin_amdgcn_s_sleep(2);
  }
}

enum : int { O_, P_, CW_, G_, WS_, WF_, Q_ };
struct Op { int k, a; };
#define RND(r) {G_, 4 * (r) - 6}, {P_, r}, {O_, (r) + 1}, {CW_, r}
constexpr Op SCHED[] = {
  {O_, 0}, {WS_, 0}, {P_, 0}, {WF_, 0}, {O_, 1}, {CW_, 0}, {P_, 1}, {O_, 2}, {CW_, 1}, {Q_, 0},
  RND(2), RND(3), RND(4), RND(5), RND(6), {G_, 22}, {P_, 7}, {CW_, 7},
};
constexpr int NOPS = sizeof(SCHED) / sizeof(Op);
constexpr int NSET = 5;
constexpr bool is_ld(int k) { return k == O_ || k == P_; }
constexpr int find_op(int k, int a) { for (int i = 0; i < NOPS; i++) if (SCHED[i].k == k && SCHED[i].a == a) return i; return -1; }
constexpr int younger(int t, int i) { int n = 0; for (int j = t + 1; j < i; j++) n += is_ld(SCHED[j].k) ? 8 : 0; return n; }
constexpr int set_of(int op) {
  int owner[NSET] = {-1, -1, -1, -1, -1};
  for (int i = 0; i < NOPS; i++) {
    if (is_ld(SCHED[i].k)) {
      int s = -1;
      for (int q = 0; q < NSET; q++) if (owner[q] < 0) { s = q; break; }
      if (s < 0) return -1;
      owner[s] = SCHED[i].a;
      if (i == op) return s;
    } else if (SCHED[i].k == CW_) {
      for (int q = 0; q < NSET; q++) if (owner[q] == SCHED[i].a) owner[q] = -1;
    }
  }
  return -1;
}
constexpr bool can_reach(int n, int i) { return n <= 0 || find_op(CW_, n / 4) < i; }
constexpr bool sched_ok() {
  for (int r = 0; r < 8; r++) {
    const int o = find_op(O_, r), p = find_op(P_, r), c = find_op(CW_, r);
    if (o < 0 || p < o || c < p || set_of(o) < 0 || set_of(p) < 0 || younger(o, c) > 63) return false;
    if (r > 0 && c < find_op(CW_, r - 1)) return false;
    if (!can_reach(4 * r + 3 - NWS + 1, c)) return false;
    if (4 * r + 3 >= NWS - 2 && c < find_op(WS_, 0)) return false;
  }
  for (int i = 0; i < NOPS; i++) if (SCHED[i].k == G_ && (!can_reach(SCHED[i].a, i) || i < find_op(Q_, 0))) return false;
  return true;
}
static_assert(sched_ok(), "loader schedule");

struct Loader {
  f32x4 wb[NSET][8];
  __amdgpu_buffer_rsrc_t rW;
  int vW[8];
  unsigned sW, own, ldsW, l, fl;
  const GAS unsigned* afl;
  GAS unsigned* tmo;
  const unsigned char *po, *pw, *pe;
  bool rt;
};
#define WAIT_SET(S, N)                                                                               \
  asm volatile("s_waitcnt vmcnt(%8)"                                                                 \
               : "+v"(L.wb[S][0]), "+v"(L.wb[S][1]), "+v"(L.wb[S][2]), "+v"(L.wb[S][3]),             \
                 "+v"(L.wb[S][4]), "+v"(L.wb[S][5]), "+v"(L.wb[S][6]), "+v"(L.wb[S][7])              \
               : "i"(N) : "memory")
template <int I> DI void run_op(Loader& L) {
  constexpr int k = SCHED[I].k, a = SCHED[I].a;
  if constexpr (is_ld(k)) {
    constexpr int s = set_of(I);
    const unsigned so = (unsigned)a * (128u * N * 4u) + L.sW + ((k == O_) ? L.own : (256u - L.own));
    asm volatile(
        "buffer_load_dwordx4 %0, %8, %16, %17 offen\n\t"
        "buffer_load_dwordx4 %1, %9, %16, %17 offen\n\t"
        "buffer_load_dwordx4 %2, %10, %16, %17 offen\n\t"
        "buffer_load_dwordx4 %3, %11, %16, %17 offen\n\t"
        "buffer_load_dwordx4 %4, %12, %16, %17 offen\n\t"
        "buffer_load_dwordx4 %5, %13, %16, %17 offen\n\t"
        "buffer_load_dwordx4 %6, %14, %16, %17 offen\n\t"
        "buffer_load_dwordx4 %7, %15, %16, %17 offen"
        : "=&v"(L.wb[s][0]), "=&v"(L.wb[s][1]), "=&v"(L.wb[s][2]), "=&v"(L.wb[s][3]),
          "=&v"(L.wb[s][4]), "=&v"(L.wb[s][5]), "=&v"(L.wb[s][6]), "=&v"(L.wb[s][7])
        : "v"(L.vW[0]), "v"(L.vW[1]), "v"(L.vW[2]), "v"(L.vW[3]), "v"(L.vW[4]), "v"(L.vW[5]), "v"(L.vW[6]), "v"(L.vW[7]),
          "s"(L.rW), "s"(so)
        : "memory");
  } else if constexpr (k == CW_) {
    constexpr int io = find_op(O_, a), ip = find_op(P_, a), so = set_of(io), sp = set_of(ip);
    const unsigned t = 4u * a + L.l;
    if constexpr (4 * a + 3 >= NWS) {
      const int need = (int)t - NWS + 1;
      while (true) {
        const u32x4 d = lds_get4(L.fl + F_DONE * 4);
        if ((int)UNI(min(min(d.x, d.y), min(d.z, d.w))) >= need) break;
        __builtin_amdgcn_s_sleep(1);
      }
    }
    unsigned slot = (4u * a) % NWS + L.l;
    if (slot >= (unsigned)NWS) slot -= NWS;
    const unsigned base = L.ldsW + slot * 8192u;
    auto conv = [&](f32x4(&w)[8], unsigned dst) {
#pragma unroll
      for (int c = 0; c < 4; c++) {
        u32x4 bf;
        bf.x = pk(w[0][c], w[1][c]); bf.y = pk(w[2][c], w[3][c]);
        bf.z = pk(w[4][c], w[5][c]); bf.w = pk(w[6][c], w[7][c]);
        *(LAS u32x4*)(dst + c * 1024) = bf;
      }
    };
    WAIT_SET(so, younger(io, I));
    conv(L.wb[so], base + L.own * 16u);
    WAIT_SET(sp, younger(ip, I));
    conv(L.wb[sp], base + (4096u - L.own * 16u));
    lds_put(L.fl + (F_FULLW + L.l) * 4, t + 1u);
  } else if constexpr (k == G_) {
    while (lds_get(L.fl + (F_DONE + L.l) * 4) < (unsigned)a) __builtin_amdgcn_s_sleep(1);
  } else if constexpr (k == WS_) {
    if (L.rt) {
      unsigned r0, r1, r2, r3;
      asm volatile("s_load_dword %0, %4, 0x0\n\ts_load_dword %1, %5, 0x20000\n\ts_load_dword %2, %6, 0x0\n\t"
                   "s_load_dword %3, %7, 0x0\n\ts_waitcnt lgkmcnt(0)"
                   : "=&s"(r0), "=&s"(r1), "=&s"(r2), "=&s"(r3) : "s"(L.po), "s"(L.pw), "s"(L.pe), "s"(L.pe + (2 << 20)) : "memory");
    }
    while (lds_get(L.fl + F_ST * 4) < 4u) __builtin_amdgcn_s_sleep(1);
  } else if constexpr (k == WF_) {
    while (lds_get(L.fl + F_FS * 4) == 0u) __builtin_amdgcn_s_sleep(1);
  } else if (L.l != 0u) {
    wait8(L.afl + L.l * 8u * 32u, lds_get(L.fl + F_CALL * 4), L.tmo);
    lds_put(L.fl + (F_RDY + L.l) * 4, 1u);
  }
}
template <int... I> DI void run_rest(Loader& L, std::integer_sequence<int, I...>) { (run_op<I + 1>(L), ...); }


#define ME (unsigned)rank
#define COMMON_DEFS \
  const int rank = p.rank; \
  unsigned char* ws = p.self; \
  GAS unsigned* tmo = (GAS unsigned*)(ws + WS_TMO); \
  GAS unsigned* afl = (GAS unsigned*)(ws + WS_AFLAG) + grp * NSTEP * 32; \
  const int c = w & 3; \
  const unsigned dst = (unsigned)(nb >> 2) & 7u; \
  const int idx = ((nb & 3) * 2 + mb2) * 4 + kq; \
  const int slot = idx * 4 + c; \
  const bool remote = dst != (unsigned)rank; \
  const unsigned me = ME; \
  const int rowin = mb2 * 256 + c * 64 + kq * 16 + (lane & 15), colb = (nb & 3) * 128 + 16 * (lane >> 4);

__global__ void __launch_bounds__(512, 2) fused_gemm_a2a(Params p) {
  extern __shared__ __attribute__((aligned(16))) unsigned char smem[];
  const int tid = threadIdx.x, lane = tid & 63;
  const int w = UNI(tid >> 6);
  const int b = blockIdx.x;
  const int xcd = b & 7, j = b >> 3;
  const int kq = xcd >> 1, mb2 = j & 1;
  const int nb = (xcd & 1) * 16 + (j >> 1);
  const int grp = mb2 * 4 + kq;
  const unsigned lds0 = (unsigned)(size_t)(LAS unsigned char*)smem;
  const unsigned fl = lds0 + L_F;
  if (__builtin_expect(w < 4, 1)) {
    f32x4 xa[8];
    const __amdgpu_buffer_rsrc_t rX = __builtin_amdgcn_make_buffer_rsrc((void*)p.x, 0, M * K * 4, 0x00020000);
    const int vX = (lane >> 3) * (K * 4) + (lane & 7) * 16;
    const int so = (mb2 * 256 + w * 64) * (K * 4) + (kq * 1024 + nb * 32) * 4;
#pragma unroll
    for (int i = 0; i < 8; i++)
      xa[i] = __builtin_bit_cast(f32x4, __builtin_amdgcn_raw_buffer_load_b128(rX, vX, so + i * 8 * (K * 4), 0));
    SB();
    const unsigned cv = __hip_atomic_load((GAS unsigned*)(p.self + WS_CNT) + b * 4 + w, RLX_AGENT);
    unsigned tw;
    {
      const int t = lane & 15;
      const unsigned char* a = p.self + WS_AT + ((grp * NSTEP + nb) * 4 + w) * 4096;
      if (t == 1) a = p.self + WS_AFLAG + (grp * NSTEP + nb) * 128;
      if (t == 2) a = p.self + WS_XFLAG + b * 12 * 128;
      if (t == 3) a = p.self + WS_AT + grp * NSTEP * 16384;
      if (t >= 4 && t < 8) a = p.self + WS_X + (size_t)((b ^ ((t & 3) << 1)) * 12) * 4096;
      if (t >= 8) a = (const unsigned char*)p.w + (size_t)(kq * 1024 + (t - 8) * 128) * (N * 4) + nb * 512 + mb2 * 256;
      tw = __hip_atomic_load((GAS unsigned*)a, RLX_AGENT);
    }
    if (tid < 64) *(LAS unsigned*)(fl + tid * 4) = 0u;
    asm volatile("s_waitcnt lgkmcnt(0)" ::: "memory");
    SB();
    __builtin_amdgcn_s_barrier();
    CFENCE();
    SB();
    COMMON_DEFS
    unsigned call;
    {
      const unsigned sc = lds0 + L_SC + c * 4096;
      const unsigned tb = sc + ((lane & 7) >> 1) * 256 + (lane >> 3) * 16 + (lane & 1) * 8;
#pragma unroll
      for (int i = 0; i < 8; i++) {
        u32x2 o = {pk(xa[i][0], xa[i][1]), pk(xa[i][2], xa[i][3])};
        *(LAS u32x2*)(tb + (((unsigned)(i >> 1)) ^ (unsigned)kq) * 1024u + (i & 1) * 128) = o;
      }
      call = UNI(cv) + 1u;
      asm volatile("" ::"v"(tw));
      u32x4 fr[4];
#pragma unroll
      for (int f = 0; f < 4; f++) fr[f] = *(LAS u32x4*)(sc + f * 1024 + lane * 16);
      unsigned char* pub = ws + WS_AT + ((grp * NSTEP + nb) * 4 + c) * 4096 + lane * 16;
#pragma unroll
      for (int f = 0; f < 4; f++) store_sys(pub + f * 1024, fr[f]);
      lds_put(fl + F_CALL * 4, call);
      lds_count(fl + F_ST * 4);
      asm volatile("s_waitcnt vmcnt(0)" ::: "memory");
      if (lds_count(fl + F_DR * 4) == 3u) {
        if (lane == 0) __hip_atomic_store(afl + nb * 32, call, RLX_SYS);
        lds_put(fl + F_FS * 4, 1u);
      }
      if (lane == 0) __hip_atomic_store((GAS unsigned*)(ws + WS_CNT) + b * 4 + c, call, RLX_AGENT);
      if (c == 0) {
        while (lds_get(fl + F_FS * 4) == 0u) {}
        wait8(afl, call, tmo);
        lds_put(fl + (F_RDY + 4) * 4, 1u);
        lds_put(fl + F_RDY * 4, 1u);
      }
    }
    f32x4 acc[4][8];
#pragma unroll
    for (int m = 0; m < 4; m++)
#pragma unroll
      for (int n = 0; n < 8; n++) { acc[m][n] = f32x4{0, 0, 0, 0}; asm volatile("" : "+v"(acc[m][n])); }
    const unsigned wBase = lds0 + lane * 16;
    unsigned pfw = fl + F_FULLW * 4, pdone = fl + (F_DONE + c) * 4, vdone = 0;
    asm volatile("" : "+v"(pfw), "+v"(pdone), "+v"(vdone));
    const __amdgpu_buffer_rsrc_t rA = __builtin_amdgcn_make_buffer_rsrc(
        (void*)(ws + WS_AT + (grp * NSTEP * 4 + c) * 4096), 0, (NSTEP - 1) * 16384 + 4096, 0x00020000);
    int vA[4];
#pragma unroll
    for (int m = 0; m < 4; m++) { vA[m] = lane * 16 + m * 1024; asm volatile("" : "+v"(vA[m])); }
    auto ldA = [&](int m, unsigned t) { return __builtin_amdgcn_raw_buffer_load_b128(rA, vA[m], t * 16384u, AUX_SYS); };
    u32x4 A[5][4], B[4];
#define SETOF(t) ((((t) & 7) == 5) ? 4 : ((t) & 3))
    unsigned fw;
    while (lds_get(fl + F_RDY * 4) == 0u) __builtin_amdgcn_s_sleep(1);
#pragma unroll
    for (int d = 0; d < 2; d++)
#pragma unroll
      for (int m = 0; m < 4; m++) { A[d][m] = ldA(m, d); SB(); }
    while (lds_get(pfw) < 1u) {}
    __builtin_amdgcn_s_setprio(3);
    B[0] = *(LAS u32x4*)(wBase);
    B[1] = *(LAS u32x4*)(wBase + 1024);
    B[2] = *(LAS u32x4*)(wBase + 2048);
    CFENCE(); fw = *(LAS unsigned*)(pfw + 4); CFENCE();
    unsigned woff = 0, wc = wBase, rdy = 0, prdy = fl + (F_RDY + 1) * 4;
    asm volatile("" : "+v"(prdy));
    auto mf = [&](int m, int n, int q) {
      asm volatile("v_mfma_f32_16x16x32_bf16 %0, %1, %2, %0" : "+v"(acc[m][n]) : "v"(B[n & 3]), "v"(A[q][m]));
    };
    auto step = [&](unsigned s, int q8) {
      const int q = q8 & 3;
      unsigned wn = 0, sf = 0, nd = 0;
#pragma unroll
      for (int n = 0; n < 8; n++) {
        mf(0, n, SETOF(q8)); SB();
        if (n == 0 && q8 == 5) {
          if (__builtin_expect(UNI(rdy) == 0u, 0)) while (lds_get(prdy) == 0u) {}
          prdy += 4u;
          CFENCE();
        }
        if (q == 1 && n < 4) { A[SETOF(q8 + 3)][n] = ldA(n, s + 3u); SB(); }
        if (n < 5) B[(n + 3) & 3] = *(LAS u32x4*)(wc + (n + 3) * 1024);
        else B[(n + 3) & 3] = *(LAS u32x4*)(wn + (n - 5) * 1024);
        SB(); mf(1, n, SETOF(q8)); SB();
        if (n == 1) { woff = (woff == (NWS - 1) * 8192u) ? 0u : woff + 8192u; wn = wBase + woff; asm volatile("" : "+v"(wn)); }
        if (n == 2) sf = UNI(fw);
        if (n == 4) lds_put(pdone, vdone);
        if (n == 7) { CFENCE(); fw = *(LAS unsigned*)(pfw + ((q + 2) & 3) * 4); CFENCE(); }
        SB(); mf(2, n, SETOF(q8)); SB();
        if (q == 0) A[SETOF(q8 + 2 + (n >> 2))][n & 3] = ldA(n & 3, s + 2u + (n >> 2));
        if (q == 1 && n < 4) A[SETOF(q8 + 4)][n] = ldA(n, s + 4u);
        if (n == 1) { vdone += 1; asm volatile("" : "+v"(vdone)); }
        if (n == 3) { nd = (q8 == 7 && s == NSTEP - 1u) ? 0u : s + 2u; asm volatile("" : "+s"(nd)); }
        if (n == 5 && q8 == 4) { CFENCE(); rdy = *(LAS unsigned*)(prdy); CFENCE(); }
        SB(); mf(3, n, SETOF(q8)); SB();
        if (n == 3) {
          if (__builtin_expect(sf < nd, 0))
            while (lds_get(pfw + ((q + 1) & 3) * 4) < nd) {}
          CFENCE();
          SB();
        }
      }
      wc = wn;
    };
    asm volatile("s_nop 7");
    for (unsigned s = 0; s < (unsigned)NSTEP; s += 8) {
      step(s, 0); step(s + 1, 1); step(s + 2, 2); step(s + 3, 3);
      step(s + 4, 4); step(s + 5, 5); step(s + 6, 6); step(s + 7, 7);
    }
    asm volatile("s_nop 15\n\ts_nop 15");
    SB();
    __builtin_amdgcn_s_setprio(0);
    const __amdgpu_buffer_rsrc_t rE = __builtin_amdgcn_make_buffer_rsrc((void*)(ws + WS_X), 0, X_BYTES, 0x00020000);
#pragma unroll
    for (int m = 1; m < 4; m++) {
      const int so = (((b ^ (m << 1)) * 3 + (m - 1)) * 4 + c) * 4096;
#pragma unroll
      for (int t = 0; t < 4; t++) {
        u32x4 o;
        o.x = pk(acc[m][2 * t][0], acc[m][2 * t][1]); o.y = pk(acc[m][2 * t][2], acc[m][2 * t][3]);
        o.z = pk(acc[m][2 * t + 1][0], acc[m][2 * t + 1][1]); o.w = pk(acc[m][2 * t + 1][2], acc[m][2 * t + 1][3]);
        __builtin_amdgcn_raw_buffer_store_b128(o, rE, lane * 16, so + t * 1024, AUX_SYS);
      }
    }
    GAS unsigned* xfl = (GAS unsigned*)(ws + WS_XFLAG);
    asm volatile("s_waitcnt vmcnt(0)" ::: "memory");
    if (lane < 3) __hip_atomic_store(xfl + (((b ^ ((lane + 1) << 1)) * 4 + c) * 3 + lane) * 32, call, RLX_SYS);
    {
      const u64 t0 = now();
      while (true) {
        unsigned v, tv;
        poll32(xfl + ((b * 4 + c) * 3 + min(lane, 2)) * 32, tmo, v, tv);
        if (__all(v >= call) || expired(tmo, tv, call, t0)) break;
      }
      __builtin_amdgcn_fence(__ATOMIC_ACQUIRE, "wavefront");
    }
    lds_count(fl + F_XS * 4);
    u32x4 in[3][4];
#pragma unroll
    for (int m = 0; m < 3; m++)
#pragma unroll
      for (int t = 0; t < 4; t++)
        in[m][t] = __builtin_amdgcn_raw_buffer_load_b128(rE, lane * 16, ((b * 3 + m) * 4 + c) * 4096 + t * 1024, AUX_SYS);
    float am = 0.f;
#pragma unroll
    for (int t = 0; t < 4; t++) {
#pragma unroll
      for (int m = 0; m < 3; m++)
#pragma unroll
        for (int e = 0; e < 4; e++) {
          const unsigned v = in[m][t][e];
          acc[0][2 * t + (e >> 1)][(e & 1) * 2] += bf_lo(v);
          acc[0][2 * t + (e >> 1)][(e & 1) * 2 + 1] += bf_hi(v);
        }
#pragma unroll
      for (int e = 0; e < 8; e++) am = fmaxf(am, fabsf(acc[0][2 * t + (e >> 2)][e & 3]));
    }
    lds_put(fl + (F_MAX + c) * 4, wave_max(__builtin_bit_cast(unsigned, am)));
    if (lds_count(fl + F_CNT * 4) == 3u) {
      const u32x4 m4 = lds_get4(fl + F_MAX * 4);
      const unsigned mx = max(max(m4.x, m4.y), max(m4.z, m4.w));
      if (lane == 0) __hip_atomic_store((GAS u64*)(ws + WS_LOC) + b * 16, ((u64)call << 32) | (u64)mx, RLX_SYS);
      lds_put(fl + F_GS * 4, 1u);
    }
    const unsigned par = call & 1u;
    u32x4 h0, h1;
    h0.x = pk(acc[0][0][0], acc[0][1][0]); h0.y = pk(acc[0][2][0], acc[0][3][0]);
    h0.z = pk(acc[0][0][1], acc[0][1][1]); h0.w = pk(acc[0][2][1], acc[0][3][1]);
    h1.x = pk(acc[0][0][2], acc[0][1][2]); h1.y = pk(acc[0][2][2], acc[0][3][2]);
    h1.z = pk(acc[0][0][3], acc[0][1][3]); h1.w = pk(acc[0][2][3], acc[0][3][3]);
    unsigned char* optr = p.out[dst] + (size_t)(rank * M + rowin) * 512 + colb;
    GAS unsigned* dnp = (GAS unsigned*)(p.ws[dst] + WS_DONE) + (slot * 8 + me) * 32;
    GAS unsigned* efp = (GAS unsigned*)(p.ws[dst] + WS_EF) + ((par * 128 + slot) * 8 + me) * 32;
    asm volatile("" : "+v"(optr), "+v"(dnp), "+v"(efp));
    bool sent = false;
    u32x2 gv;
    {
      unsigned ts = 0;
      while (gv = lds_get2(fl + F_GOK * 4), UNI(gv.x) == 0u) {
        if (remote && ts == 0u && lds_get(fl + F_GS * 4) != 0u) ts = lds_get(fl + F_TS * 4);
        if (ts != 0u && !sent && (int)((unsigned)now() - (ts + (unsigned)c * EGAP)) >= 0) {
          unsigned char* ep = p.ws[dst] + WS_E + (size_t)((par * 8 + me) * 128 + slot) * 2048 + lane * 16;
          store_sys(ep, h0);
          store_sys(ep + 1024, h1);
          asm volatile("s_waitcnt vmcnt(0)" ::: "memory");
          if (lane == 0) __hip_atomic_store(efp, 2u * call + 1u, RLX_SYS);
          sent = true;
        }
      }
    }
    const unsigned rsb = gv.y;
    const float rs = __builtin_bit_cast(float, rsb);
    const bool failed = gv.x != 1u;
    const u32x4 poison = {0x80808080u, 0x80808080u, 0x80808080u, 0x80808080u};
    {
      u32x4 o;
#pragma unroll
      for (int rg = 0; rg < 4; rg++) o[rg] = q4(acc[0][4][rg], acc[0][5][rg], acc[0][6][rg], acc[0][7][rg], rs);
      store_sys(optr + 64, failed ? poison : o);
    }
    if (!sent) {
      u32x4 o;
      o.x = q4(bf_lo(h0.x), bf_hi(h0.x), bf_lo(h0.y), bf_hi(h0.y), rs);
      o.y = q4(bf_lo(h0.z), bf_hi(h0.z), bf_lo(h0.w), bf_hi(h0.w), rs);
      o.z = q4(bf_lo(h1.x), bf_hi(h1.x), bf_lo(h1.y), bf_hi(h1.y), rs);
      o.w = q4(bf_lo(h1.z), bf_hi(h1.z), bf_lo(h1.w), bf_hi(h1.w), rs);
      store_sys(optr, failed ? poison : o);
    }
    if (remote) {
      if (!sent && lane == 0) __hip_atomic_store(efp, 2u * call, RLX_SYS);
      asm volatile("s_waitcnt vmcnt(0)" ::: "memory");
      if (lane == 0) __hip_atomic_store(dnp, call, RLX_SYS);
    }
  } else {
    COMMON_DEFS
    {
      Loader L;
      L.l = c;
      L.rW = __builtin_amdgcn_make_buffer_rsrc((void*)p.w, 0, K * N * 4, 0x00020000);
#pragma unroll
      for (int i = 0; i < 8; i++) L.vW[i] = ((lane >> 4) * 8 + i) * (N * 4) + (lane & 15) * 16;
      L.sW = (unsigned)(kq * 1024 + c * 32) * (N * 4u) + (unsigned)nb * 512u;
      L.own = (unsigned)mb2 * 256u;
      L.ldsW = lds0 + lane * 16;
      L.fl = fl;
      L.afl = afl;
      L.tmo = tmo;
      L.po = p.out[j & 7] + (size_t)rank * M * 512;
      L.pw = p.ws[j & 7];
      L.pe = p.ws[j & 7] + WS_E + (size_t)rank * 128 * 2048;
      L.rt = c == 3 && j < NR;
      run_op<0>(L);
      __builtin_amdgcn_s_barrier();
      CFENCE();
      run_rest(L, std::make_integer_sequence<int, NOPS - 1>{});
    }
    const unsigned call = lds_get(fl + F_CALL * 4);
    const unsigned par = call & 1u;
    const u64 t0 = now();
    if (c == 2 && remote) {
      while (lds_get(fl + F_XS * 4) < 4u) __builtin_amdgcn_s_sleep(2);
      GAS u64* bk = (GAS u64*)(ws + WS_BOOK) + dst * 16;
      u64 tb = 0;
      if (lane == 0) {
        __hip_atomic_fetch_max(bk, now() + XBOOK, RLX_AGENT);
        tb = __hip_atomic_fetch_add(bk, (u64)(4u * EGAP), RLX_AGENT);
      }
      lds_put(fl + F_TS * 4, UNI((unsigned)tb) | 1u);
    }
    if (!remote && idx == 0) {
      GAS u64* g = (GAS u64*)(ws + WS_LOC) + (c * 64 + lane) * 16;
      const int srcid = rank;
      GAS u64* mb = (GAS u64*)(p.ws[lane >> 3] + WS_MBOX) + ((par * 8 + (lane & 7)) * 8 + srcid) * 16;
      asm volatile("" : "+v"(mb));
      while (lds_get(fl + F_GS * 4) == 0u) __builtin_amdgcn_s_sleep(2);
      u64 v;
      while (true) {
        unsigned tv;
        poll64(g, tmo, v, tv);
        if (__all((unsigned)(v >> 32) == call) || expired(tmo, tv, call, t0)) break;
      }
      lds_put(fl + (F_LMAX + c) * 4, wave_max((unsigned)v));
      if (lds_count(fl + F_LCNT * 4) == 3u) {
        const u32x4 m4 = lds_get4(fl + F_LMAX * 4);
        const unsigned mx = max(max(m4.x, m4.y), max(m4.z, m4.w));
        __hip_atomic_store(mb, ((u64)call << 32) | (u64)mx, RLX_SYS);
      }
    }
    if (c == 1) {
      while (lds_get(fl + F_GS * 4) == 0u) __builtin_amdgcn_s_sleep(2);
      GAS u64* g = (GAS u64*)(ws + WS_MBOX) + ((par * 8 + xcd) * 8 + (lane & 7)) * 16;
      u64 v;
      bool bad;
      while (true) {
        unsigned tv;
        poll64(g, tmo, v, tv);
        bad = UNI(tv) == call;
        if (__all((unsigned)(v >> 32) == call)) break;
        if (expired(tmo, tv, call, t0)) { bad = true; break; }
      }
      const unsigned mx = wave_max((unsigned)v);
      const float amax = __builtin_bit_cast(float, mx);
      const float rs = amax > 0.f ? 127.0f / amax : 0.f;
      const u32x2 gw = {bad ? 2u : 1u, __builtin_bit_cast(unsigned, rs)};
      asm volatile("ds_write_b64 %0, %1" ::"v"(fl + F_GOK * 4), "v"(gw) : "memory");
    }
    if (remote) {
      GAS unsigned* ef = (GAS unsigned*)(ws + WS_EF) + ((par * 128 + slot) * 8 + dst) * 32;
      while (lds_get(fl + F_GS * 4) == 0u) __builtin_amdgcn_s_sleep(4);
      unsigned e;
      bool bad = false;
      while (true) {
        unsigned tv;
        poll32(ef, tmo, e, tv);
        e = UNI(e);
        if (e >= 2u * call) break;
        if (expired(tmo, tv, call, t0)) { bad = true; break; }
        if (lds_get(fl + F_GOK * 4) == 0u) __builtin_amdgcn_s_sleep(12);
      }
      if (bad || (e & 1u)) {
        __builtin_amdgcn_fence(__ATOMIC_ACQUIRE, "wavefront");
        const __amdgpu_buffer_rsrc_t rI = __builtin_amdgcn_make_buffer_rsrc(
            (void*)(ws + WS_E + (size_t)((par * 8 + dst) * 128 + slot) * 2048), 0, bad ? 0 : 2048, 0x00020000);
        const u32x4 h0 = __builtin_amdgcn_raw_buffer_load_b128(rI, lane * 16, 0, AUX_SYS);
        const u32x4 h1 = __builtin_amdgcn_raw_buffer_load_b128(rI, lane * 16, 1024, AUX_SYS);
        u32x2 gv;
        while (gv = lds_get2(fl + F_GOK * 4), UNI(gv.x) == 0u) __builtin_amdgcn_s_sleep(1);
        const unsigned rsb = gv.y;
        const float rs = __builtin_bit_cast(float, rsb);
        u32x4 o;
        o.x = q4(bf_lo(h0.x), bf_hi(h0.x), bf_lo(h0.y), bf_hi(h0.y), rs);
        o.y = q4(bf_lo(h0.z), bf_hi(h0.z), bf_lo(h0.w), bf_hi(h0.w), rs);
        o.z = q4(bf_lo(h1.x), bf_hi(h1.x), bf_lo(h1.y), bf_hi(h1.y), rs);
        o.w = q4(bf_lo(h1.z), bf_hi(h1.z), bf_lo(h1.w), bf_hi(h1.w), rs);
        if (bad || gv.x != 1u) o = u32x4{0x80808080u, 0x80808080u, 0x80808080u, 0x80808080u};
        store_sys(p.out[rank] + (size_t)(dst * M + rowin) * 512 + colb, o);
      }
    } else {
      while (lds_get(fl + F_GOK * 4) == 0u) __builtin_amdgcn_s_sleep(2);
      GAS unsigned* d = (GAS unsigned*)(ws + WS_DONE) + (slot * 8 + (lane & 7)) * 32;
      bool bad = false;
      while (true) {
        unsigned v, tv;
        poll32(d, tmo, v, tv);
        if (__all(v >= call || (lane & 7) == rank)) break;
        if (expired(tmo, tv, call, t0)) { bad = true; break; }
      }
      if (bad && lane == 0) p.out[rank][(size_t)(rank * M + rowin) * 512 + colb] = 0x80;
    }
  }
}

extern "C" void kernel_launch(int rank, int nranks, ncclComm_t comm, hipStream_t stream,
                              void* const* d_in, const int* in_sizes, int n_in,
                              void* d_out, int out_size, const int* in_dtypes, int out_dtype,
                              void* const* d_out_all, void* const* d_ws_all, size_t ws_bytes) {
  if (nranks != NR || rank < 0 || rank >= NR || n_in < 2 || ws_bytes < WS_NEED || in_sizes[0] != M * K ||
      in_sizes[1] != K * N || out_size != NR * M * (N / NR)) {
    fprintf(stderr, "unexpected problem shape\n");
    return;
  }
  static bool init = false;
  if (!init) {
    HIP_CHECK(hipFuncSetAttribute((const void*)fused_gemm_a2a, hipFuncAttributeMaxDynamicSharedMemorySize, LDS_BYTES));
    init = true;
  }
  Params p;
  p.x = (const float*)d_in[0];
  p.w = (const float*)d_in[1];
  for (int i = 0; i < NR; i++) { p.out[i] = (unsigned char*)d_out_all[i]; p.ws[i] = (unsigned char*)d_ws_all[i]; }
  p.self = (unsigned char*)d_ws_all[rank];
  p.rank = rank;
  p.pad = 0;
  hipLaunchKernelGGL(fused_gemm_a2a, dim3(NWG), dim3(512), LDS_BYTES, stream, p);
  HIP_CHECK(hipGetLastError());
}
```

```cpp
#include <hip/hip_runtime.h>
#include <rccl/rccl.h>
#include <cstdio>
#include <utility>

#define HIP_CHECK(expr) do { hipError_t _e = (expr); if (_e != hipSuccess) { \
    fprintf(stderr, "HIP error %s at %s:%d\n", hipGetErrorString(_e), __FILE__, __LINE__); \
    return; } } while (0)

typedef unsigned u32x4 __attribute__((ext_vector_type(4)));
typedef unsigned u32x2 __attribute__((ext_vector_type(2)));
typedef float f32x4 __attribute__((ext_vector_type(4)));
typedef float f32x2 __attribute__((ext_vector_type(2)));
typedef __bf16 bf16x2 __attribute__((ext_vector_type(2)));
typedef unsigned long long u64;
#define GAS __attribute__((address_space(1)))
#define LAS __attribute__((address_space(3)))
#define DI __device__ __forceinline__
#pragma clang diagnostic ignored "-Wint-to-pointer-cast"

constexpr int NR = 8;
constexpr int M = 512, N = 4096, K = 4096;
constexpr int NWG = 256;
constexpr int NSTEP = 32;
constexpr int WS_TMO = 0;
constexpr int WS_CNT = 4096;
constexpr int WS_LOC = 8192;
constexpr int WS_AFLAG = 64 * 1024;
constexpr int WS_MBOX = 96 * 1024;
constexpr int WS_BOOK = 120 * 1024;
constexpr int WS_DONE = 128 * 1024;
constexpr int WS_EF = 256 * 1024;
constexpr int WS_XFLAG = 576 * 1024;
constexpr int WS_AT = 1 << 20;
constexpr int WS_X = 5 << 20;
constexpr int X_BYTES = NWG * 3 * 4 * 4096;
constexpr int WS_E = 18 << 20;
constexpr size_t WS_NEED = (size_t)WS_E + 2 * 8 * 128 * 2048;
constexpr u64 DEADLINE = 100000000ull;
constexpr int NWS = 19;
constexpr int L_F = NWS * 8192;
constexpr int L_SC = L_F - 16384;
constexpr int LDS_BYTES = L_F + 256;
static_assert(LDS_BYTES <= 160 * 1024, "LDS");
enum : int {
  F_FULLW = 0,
  F_DONE = 4,
  F_RDY = 32,
  F_MAX = 12,
  F_LMAX = 16,
  F_GOK = 20,
  F_CALL = 22, F_ST, F_DR, F_FS, F_CNT, F_GS, F_LCNT, F_TS, F_XS
};
constexpr int AUX_SYS = 17;
constexpr unsigned XBOOK = 70;
constexpr unsigned EGAP = 4;

struct Params {
  const float* x;
  const float* w;
  unsigned char* self;
  int rank;
  int pad;
  unsigned char* out[NR];
  unsigned char* ws[NR];
};

#define RLX_AGENT __ATOMIC_RELAXED, __HIP_MEMORY_SCOPE_AGENT
#define RLX_SYS __ATOMIC_RELAXED, __HIP_MEMORY_SCOPE_SYSTEM
#define CFENCE() asm volatile("" ::: "memory")
#define SB() __builtin_amdgcn_sched_barrier(0)
#define UNI(x) __builtin_amdgcn_readfirstlane(x)

DI unsigned pk(float a, float b) {
  f32x2 v = {a, b};
  bf16x2 r = __builtin_convertvector(v, bf16x2);
  return __builtin_bit_cast(unsigned, r);
}
DI float bf_lo(unsigned v) { return __builtin_bit_cast(float, v << 16); }
DI float bf_hi(unsigned v) { return __builtin_bit_cast(float, v & 0xffff0000u); }
DI unsigned q1(float v, float rs) {
  constexpr float MAGIC = 12582912.0f;
  return __builtin_bit_cast(unsigned, __builtin_amdgcn_fmed3f(__builtin_fmaf(v, rs, MAGIC), MAGIC - 127.0f, MAGIC + 127.0f));
}
DI unsigned q4(float a, float b, float c, float d, float rs) {
  return __builtin_amdgcn_perm(q1(b, rs), q1(a, rs), 0x0c0c0400u) | __builtin_amdgcn_perm(q1(d, rs), q1(c, rs), 0x04000c0cu);
}
DI u32x4 q16(u32x4 h0, u32x4 h1, float rs) {
  u32x4 o;
  o.x = q4(bf_lo(h0.x), bf_hi(h0.x), bf_lo(h0.y), bf_hi(h0.y), rs);
  o.y = q4(bf_lo(h0.z), bf_hi(h0.z), bf_lo(h0.w), bf_hi(h0.w), rs);
  o.z = q4(bf_lo(h1.x), bf_hi(h1.x), bf_lo(h1.y), bf_hi(h1.y), rs);
  o.w = q4(bf_lo(h1.z), bf_hi(h1.z), bf_lo(h1.w), bf_hi(h1.w), rs);
  return o;
}
DI u64 now() { return __builtin_amdgcn_s_memrealtime(); }
DI unsigned wave_max(unsigned v) {
  v = max(v, (unsigned)__builtin_amdgcn_update_dpp(0, (int)v, 0x111, 0xf, 0xf, true));
  v = max(v, (unsigned)__builtin_amdgcn_update_dpp(0, (int)v, 0x112, 0xf, 0xf, true));
  v = max(v, (unsigned)__builtin_amdgcn_update_dpp(0, (int)v, 0x114, 0xf, 0xf, true));
  v = max(v, (unsigned)__builtin_amdgcn_update_dpp(0, (int)v, 0x118, 0xf, 0xf, true));
  const unsigned a = __builtin_amdgcn_readlane(v, 15), b = __builtin_amdgcn_readlane(v, 31);
  const unsigned c = __builtin_amdgcn_readlane(v, 47), d = __builtin_amdgcn_readlane(v, 63);
  return max(max(a, b), max(c, d));
}
DI bool expired(GAS unsigned* tmo, unsigned tv, unsigned call, u64 t0) {
  if (UNI(tv) == call) return true;
  if (now() - t0 > DEADLINE) { __hip_atomic_store(tmo, call, RLX_AGENT); return true; }
  return false;
}
DI void poll32(const GAS unsigned* f, const GAS unsigned* tmo, unsigned& v, unsigned& tv) {
  asm volatile("global_load_dword %0, %2, off sc0 sc1\n\tglobal_load_dword %1, %3, off sc1\n\ts_waitcnt vmcnt(0)"
               : "=&v"(v), "=&v"(tv) : "v"(f), "v"(tmo) : "memory");
}
DI void poll64(const GAS u64* f, const GAS unsigned* tmo, u64& v, unsigned& tv) {
  asm volatile("global_load_dwordx2 %0, %2, off sc0 sc1\n\tglobal_load_dword %1, %3, off sc1\n\ts_waitcnt vmcnt(0)"
               : "=&v"(v), "=&v"(tv) : "v"(f), "v"(tmo) : "memory");
}
DI void store_sys(const void* p, u32x4 v) {
  asm volatile("global_store_dwordx4 %0, %1, off sc0 sc1\n\ts_nop 1" ::"v"(p), "v"(v) : "memory");
}
DI void lds_put(unsigned addr, unsigned v) { asm volatile("ds_write_b32 %0, %1" ::"v"(addr), "v"(v) : "memory"); }
DI unsigned lds_get(unsigned addr) {
  unsigned v;
  asm volatile("ds_read_b32 %0, %1\n\ts_waitcnt lgkmcnt(0)" : "=v"(v) : "v"(addr) : "memory");
  return UNI(v);
}
DI u32x2 lds_get2(unsigned addr) {
  u32x2 v;
  asm volatile("ds_read_b64 %0, %1\n\ts_waitcnt lgkmcnt(0)" : "=v"(v) : "v"(addr) : "memory");
  return v;
}
DI u32x4 lds_get4(unsigned addr) {
  u32x4 v;
  asm volatile("ds_read_b128 %0, %1\n\ts_waitcnt lgkmcnt(0)" : "=v"(v) : "v"(addr) : "memory");
  return v;
}
DI unsigned lds_count(unsigned cnt) {
  unsigned old;
  asm volatile("s_mov_b64 exec, 1\n\tds_add_rtn_u32 %0, %1, %2\n\ts_mov_b64 exec, -1\n\ts_waitcnt lgkmcnt(0)"
               : "=&v"(old) : "v"(cnt), "v"(1u) : "memory");
  return UNI(old);
}
DI void wait8(const GAS unsigned* f8, unsigned call, GAS unsigned* tmo) {
  const u64 t0 = now();
  for (unsigned it = 1;; it++) {
    unsigned a0, a1, a2, a3, a4, a5, a6, a7;
    asm volatile(
        "s_load_dword %0, %8, 0x0 glc\n\ts_load_dword %1, %8, 0x80 glc\n\ts_load_dword %2, %8, 0x100 glc\n\t"
        "s_load_dword %3, %8, 0x180 glc\n\ts_load_dword %4, %8, 0x200 glc\n\ts_load_dword %5, %8, 0x280 glc\n\t"
        "s_load_dword %6, %8, 0x300 glc\n\ts_load_dword %7, %8, 0x380 glc\n\ts_waitcnt lgkmcnt(0)"
        : "=&s"(a0), "=&s"(a1), "=&s"(a2), "=&s"(a3), "=&s"(a4), "=&s"(a5), "=&s"(a6), "=&s"(a7)
        : "s"(f8) : "memory");
    if (min(min(min(a0, a1), min(a2, a3)), min(min(a4, a5), min(a6, a7))) >= call) break;
    if ((it & 255u) == 0u) {
      unsigned tv;
      asm volatile("global_load_dword %0, %1, off sc1\n\ts_waitcnt vmcnt(0)" : "=&v"(tv) : "v"(tmo) : "memory");
      if (UNI(tv) == call) break;
      if (now() - t0 > DEADLINE) {
        asm volatile("global_store_dword %0, %1, off sc1\n\ts_waitcnt vmcnt(0)" ::"v"(tmo), "v"(call) : "memory");
        break;
      }
    }
    __builtin_amdgcn_s_sleep(2);
  }
}

enum : int { O_, P_, CW_, G_, WS_, WF_, Q_ };
struct Op { int k, a; };
#define RND(r) {G_, 4 * (r) - 6}, {P_, r}, {O_, (r) + 1}, {CW_, r}
constexpr Op SCHED[] = {
  {O_, 0}, {WS_, 0}, {P_, 0}, {WF_, 0}, {O_, 1}, {CW_, 0}, {P_, 1}, {O_, 2}, {CW_, 1}, {Q_, 0},
  RND(2), RND(3), RND(4), RND(5), RND(6), {G_, 22}, {P_, 7}, {CW_, 7},
};
constexpr int NOPS = sizeof(SCHED) / sizeof(Op);
constexpr int NSET = 5;
constexpr bool is_ld(int k) { return k == O_ || k == P_; }
constexpr int find_op(int k, int a) { for (int i = 0; i < NOPS; i++) if (SCHED[i].k == k && SCHED[i].a == a) return i; return -1; }
constexpr int younger(int t, int i) { int n = 0; for (int j = t + 1; j < i; j++) n += is_ld(SCHED[j].k) ? 8 : 0; return n; }
constexpr int set_of(int op) {
  int owner[NSET] = {-1, -1, -1, -1, -1};
  for (int i = 0; i < NOPS; i++) {
    if (is_ld(SCHED[i].k)) {
      int s = -1;
      for (int q = 0; q < NSET; q++) if (owner[q] < 0) { s = q; break; }
      if (s < 0) return -1;
      owner[s] = SCHED[i].a;
      if (i == op) return s;
    } else if (SCHED[i].k == CW_) {
      for (int q = 0; q < NSET; q++) if (owner[q] == SCHED[i].a) owner[q] = -1;
    }
  }
  return -1;
}
constexpr bool can_reach(int n, int i) { return n <= 0 || find_op(CW_, n / 4) < i; }
constexpr bool sched_ok() {
  for (int r = 0; r < 8; r++) {
    const int o = find_op(O_, r), p = find_op(P_, r), c = find_op(CW_, r);
    if (o < 0 || p < o || c < p || set_of(o) < 0 || set_of(p) < 0 || younger(o, c) > 63) return false;
    if (r > 0 && c < find_op(CW_, r - 1)) return false;
    if (!can_reach(4 * r + 3 - NWS + 1, c)) return false;
    if (4 * r + 3 >= NWS - 2 && c < find_op(WS_, 0)) return false;
  }
  for (int i = 0; i < NOPS; i++) if (SCHED[i].k == G_ && (!can_reach(SCHED[i].a, i) || i < find_op(Q_, 0))) return false;
  return true;
}
static_assert(sched_ok(), "loader schedule");

struct Loader {
  f32x4 wb[NSET][8];
  __amdgpu_buffer_rsrc_t rW;
  int vW[8];
  unsigned sW, own, ldsW, l, fl;
  const GAS unsigned* afl;
  GAS unsigned* tmo;
  const unsigned char *po, *pw, *pe;
  bool rt;
};
#define WAIT_SET(S, N)                                                                               \
  asm volatile("s_waitcnt vmcnt(%8)"                                                                 \
               : "+v"(L.wb[S][0]), "+v"(L.wb[S][1]), "+v"(L.wb[S][2]), "+v"(L.wb[S][3]),             \
                 "+v"(L.wb[S][4]), "+v"(L.wb[S][5]), "+v"(L.wb[S][6]), "+v"(L.wb[S][7])              \
               : "i"(N) : "memory")
template <int I> DI void run_op(Loader& L) {
  constexpr int k = SCHED[I].k, a = SCHED[I].a;
  if constexpr (is_ld(k)) {
    constexpr int s = set_of(I);
    const unsigned so = (unsigned)a * (128u * N * 4u) + L.sW + ((k == O_) ? L.own : (256u - L.own));
    asm volatile(
        "buffer_load_dwordx4 %0, %8, %16, %17 offen\n\t"
        "buffer_load_dwordx4 %1, %9, %16, %17 offen\n\t"
        "buffer_load_dwordx4 %2, %10, %16, %17 offen\n\t"
        "buffer_load_dwordx4 %3, %11, %16, %17 offen\n\t"
        "buffer_load_dwordx4 %4, %12, %16, %17 offen\n\t"
        "buffer_load_dwordx4 %5, %13, %16, %17 offen\n\t"
        "buffer_load_dwordx4 %6, %14, %16, %17 offen\n\t"
        "buffer_load_dwordx4 %7, %15, %16, %17 offen"
        : "=&v"(L.wb[s][0]), "=&v"(L.wb[s][1]), "=&v"(L.wb[s][2]), "=&v"(L.wb[s][3]),
          "=&v"(L.wb[s][4]), "=&v"(L.wb[s][5]), "=&v"(L.wb[s][6]), "=&v"(L.wb[s][7])
        : "v"(L.vW[0]), "v"(L.vW[1]), "v"(L.vW[2]), "v"(L.vW[3]), "v"(L.vW[4]), "v"(L.vW[5]), "v"(L.vW[6]), "v"(L.vW[7]),
          "s"(L.rW), "s"(so)
        : "memory");
  } else if constexpr (k == CW_) {
    constexpr int io = find_op(O_, a), ip = find_op(P_, a), so = set_of(io), sp = set_of(ip);
    const unsigned t = 4u * a + L.l;
    if constexpr (4 * a + 3 >= NWS) {
      const int need = (int)t - NWS + 1;
      while (true) {
        const u32x4 d = lds_get4(L.fl + F_DONE * 4);
        if ((int)UNI(min(min(d.x, d.y), min(d.z, d.w))) >= need) break;
        __builtin_amdgcn_s_sleep(1);
      }
    }
    unsigned slot = (4u * a) % NWS + L.l;
    if (slot >= (unsigned)NWS) slot -= NWS;
    const unsigned base = L.ldsW + slot * 8192u;
    auto conv = [&](f32x4(&w)[8], unsigned dst) {
#pragma unroll
      for (int c = 0; c < 4; c++) {
        u32x4 bf;
        bf.x = pk(w[0][c], w[1][c]); bf.y = pk(w[2][c], w[3][c]);
        bf.z = pk(w[4][c], w[5][c]); bf.w = pk(w[6][c], w[7][c]);
        *(LAS u32x4*)(dst + c * 1024) = bf;
      }
    };
    WAIT_SET(so, younger(io, I));
    conv(L.wb[so], base + L.own * 16u);
    WAIT_SET(sp, younger(ip, I));
    conv(L.wb[sp], base + (4096u - L.own * 16u));
    lds_put(L.fl + (F_FULLW + L.l) * 4, t + 1u);
  } else if constexpr (k == G_) {
    while (lds_get(L.fl + (F_DONE + L.l) * 4) < (unsigned)a) __builtin_amdgcn_s_sleep(1);
  } else if constexpr (k == WS_) {
    if (L.rt) {
      unsigned r0, r1, r2, r3;
      asm volatile("s_load_dword %0, %4, 0x0\n\ts_load_dword %1, %5, 0x20000\n\ts_load_dword %2, %6, 0x0\n\t"
                   "s_load_dword %3, %7, 0x0\n\ts_waitcnt lgkmcnt(0)"
                   : "=&s"(r0), "=&s"(r1), "=&s"(r2), "=&s"(r3) : "s"(L.po), "s"(L.pw), "s"(L.pe), "s"(L.pe + (2 << 20)) : "memory");
    }
    while (lds_get(L.fl + F_ST * 4) < 4u) __builtin_amdgcn_s_sleep(1);
  } else if constexpr (k == WF_) {
    while (lds_get(L.fl + F_FS * 4) == 0u) __builtin_amdgcn_s_sleep(1);
  } else if (L.l != 0u) {
    wait8(L.afl + L.l * 8u * 32u, lds_get(L.fl + F_CALL * 4), L.tmo);
    lds_put(L.fl + (F_RDY + L.l) * 4, 1u);
  }
}
template <int... I> DI void run_rest(Loader& L, std::integer_sequence<int, I...>) { (run_op<I + 1>(L), ...); }


#define ME (unsigned)rank
#define COMMON_DEFS \
  const int rank = p.rank; \
  unsigned char* ws = p.self; \
  GAS unsigned* tmo = (GAS unsigned*)(ws + WS_TMO); \
  GAS unsigned* afl = (GAS unsigned*)(ws + WS_AFLAG) + grp * NSTEP * 32; \
  const int c = w & 3; \
  const unsigned dst = (unsigned)(nb >> 2) & 7u; \
  const int idx = ((nb & 3) * 2 + mb2) * 4 + kq; \
  const int slot = idx * 4 + c; \
  const bool remote = dst != (unsigned)rank; \
  const unsigned me = ME; \
  const int rowin = mb2 * 256 + c * 64 + kq * 16 + (lane & 15), colb = (nb & 3) * 128 + 16 * (lane >> 4);

__global__ void __launch_bounds__(512, 2) fused_gemm_a2a(Params p) {
  extern __shared__ __attribute__((aligned(16))) unsigned char smem[];
  const int tid = threadIdx.x, lane = tid & 63;
  const int w = UNI(tid >> 6);
  const int b = blockIdx.x;
  const int xcd = b & 7, j = b >> 3;
  const int kq = xcd >> 1, mb2 = j & 1;
  const int nb = (xcd & 1) * 16 + (j >> 1);
  const int grp = mb2 * 4 + kq;
  const unsigned lds0 = (unsigned)(size_t)(LAS unsigned char*)smem;
  const unsigned fl = lds0 + L_F;
  if (__builtin_expect(w < 4, 1)) {
    f32x4 xa[8];
    const __amdgpu_buffer_rsrc_t rX = __builtin_amdgcn_make_buffer_rsrc((void*)p.x, 0, M * K * 4, 0x00020000);
    const int vX = (lane >> 3) * (K * 4) + (lane & 7) * 16;
    const int so = (mb2 * 256 + w * 64) * (K * 4) + (kq * 1024 + nb * 32) * 4;
#pragma unroll
    for (int i = 0; i < 8; i++)
      xa[i] = __builtin_bit_cast(f32x4, __builtin_amdgcn_raw_buffer_load_b128(rX, vX, so + i * 8 * (K * 4), 0));
    SB();
    const unsigned cv = __hip_atomic_load((GAS unsigned*)(p.self + WS_CNT) + b * 4 + w, RLX_AGENT);
    unsigned tw;
    {
      const int t = lane & 15;
      const unsigned char* a = p.self + WS_AT + ((grp * NSTEP + nb) * 4 + w) * 4096;
      if (t == 1) a = p.self + WS_AFLAG + (grp * NSTEP + nb) * 128;
      if (t == 2) a = p.self + WS_XFLAG + b * 12 * 128;
      if (t == 3) a = p.self + WS_AT + grp * NSTEP * 16384;
      if (t >= 4 && t < 8) a = p.self + WS_X + (size_t)((b ^ ((t & 3) << 1)) * 12) * 4096;
      if (t >= 8) a = (const unsigned char*)p.w + (size_t)(kq * 1024 + (t - 8) * 128) * (N * 4) + nb * 512 + mb2 * 256;
      tw = __hip_atomic_load((GAS unsigned*)a, RLX_AGENT);
    }
    if (tid < 64) *(LAS unsigned*)(fl + tid * 4) = 0u;
    asm volatile("s_waitcnt lgkmcnt(0)" ::: "memory");
    SB();
    __builtin_amdgcn_s_barrier();
    CFENCE();
    SB();
    COMMON_DEFS
    unsigned call;
    {
      const unsigned sc = lds0 + L_SC + c * 4096;
      const unsigned tb = sc + ((lane & 7) >> 1) * 256 + (lane >> 3) * 16 + (lane & 1) * 8;
#pragma unroll
      for (int i = 0; i < 8; i++) {
        u32x2 o = {pk(xa[i][0], xa[i][1]), pk(xa[i][2], xa[i][3])};
        *(LAS u32x2*)(tb + (((unsigned)(i >> 1)) ^ (unsigned)kq) * 1024u + (i & 1) * 128) = o;
      }
      call = UNI(cv) + 1u;
      asm volatile("" ::"v"(tw));
      u32x4 fr[4];
#pragma unroll
      for (int f = 0; f < 4; f++) fr[f] = *(LAS u32x4*)(sc + f * 1024 + lane * 16);
      unsigned char* pub = ws + WS_AT + ((grp * NSTEP + nb) * 4 + c) * 4096 + lane * 16;
#pragma unroll
      for (int f = 0; f < 4; f++) store_sys(pub + f * 1024, fr[f]);
      lds_put(fl + F_CALL * 4, call);
      lds_count(fl + F_ST * 4);
      asm volatile("s_waitcnt vmcnt(0)" ::: "memory");
      if (lds_count(fl + F_DR * 4) == 3u) {
        if (lane == 0) __hip_atomic_store(afl + nb * 32, call, RLX_SYS);
        lds_put(fl + F_FS * 4, 1u);
      }
      if (lane == 0) __hip_atomic_store((GAS unsigned*)(ws + WS_CNT) + b * 4 + c, call, RLX_AGENT);
      if (c == 0) {
        while (lds_get(fl + F_FS * 4) == 0u) {}
        wait8(afl, call, tmo);
        lds_put(fl + (F_RDY + 4) * 4, 1u);
        lds_put(fl + F_RDY * 4, 1u);
      }
    }
    f32x4 acc[4][8];
#pragma unroll
    for (int m = 0; m < 4; m++)
#pragma unroll
      for (int n = 0; n < 8; n++) { acc[m][n] = f32x4{0, 0, 0, 0}; asm volatile("" : "+v"(acc[m][n])); }
    const unsigned wBase = lds0 + lane * 16;
    unsigned pfw = fl + F_FULLW * 4, pdone = fl + (F_DONE + c) * 4, vdone = 0;
    asm volatile("" : "+v"(pfw), "+v"(pdone), "+v"(vdone));
    const __amdgpu_buffer_rsrc_t rA = __builtin_amdgcn_make_buffer_rsrc(
        (void*)(ws + WS_AT + (grp * NSTEP * 4 + c) * 4096), 0, (NSTEP - 1) * 16384 + 4096, 0x00020000);
    int vA[4];
#pragma unroll
    for (int m = 0; m < 4; m++) { vA[m] = lane * 16 + m * 1024; asm volatile("" : "+v"(vA[m])); }
    auto ldA = [&](int m, unsigned t) { return __builtin_amdgcn_raw_buffer_load_b128(rA, vA[m], t * 16384u, AUX_SYS); };
    u32x4 A[5][4], B[4];
#define SETOF(t) ((((t) & 7) == 5) ? 4 : ((t) & 3))
    unsigned fw;
    while (lds_get(fl + F_RDY * 4) == 0u) __builtin_amdgcn_s_sleep(1);
#pragma unroll
    for (int d = 0; d < 2; d++)
#pragma unroll
      for (int m = 0; m < 4; m++) { A[d][m] = ldA(m, d); SB(); }
    while (lds_get(pfw) < 1u) {}
    __builtin_amdgcn_s_setprio(3);
    B[0] = *(LAS u32x4*)(wBase);
    B[1] = *(LAS u32x4*)(wBase + 1024);
    B[2] = *(LAS u32x4*)(wBase + 2048);
    CFENCE(); fw = *(LAS unsigned*)(pfw + 4); CFENCE();
    unsigned woff = 0, wc = wBase, rdy = 0, prdy = fl + (F_RDY + 1) * 4;
    asm volatile("" : "+v"(prdy));
    auto mf = [&](int m, int n, int q) {
      asm volatile("v_mfma_f32_16x16x32_bf16 %0, %1, %2, %0" : "+v"(acc[m][n]) : "v"(B[n & 3]), "v"(A[q][m]));
    };
    auto step = [&](unsigned s, int q8) {
      const int q = q8 & 3;
      unsigned wn = 0, sf = 0, nd = 0;
#pragma unroll
      for (int n = 0; n < 8; n++) {
        mf(0, n, SETOF(q8)); SB();
        if (n == 0 && q8 == 5) {
          if (__builtin_expect(UNI(rdy) == 0u, 0)) while (lds_get(prdy) == 0u) {}
          prdy += 4u;
          CFENCE();
        }
        if (q == 1 && n < 4) { A[SETOF(q8 + 3)][n] = ldA(n, s + 3u); SB(); }
        if (n < 5) B[(n + 3) & 3] = *(LAS u32x4*)(wc + (n + 3) * 1024);
        else B[(n + 3) & 3] = *(LAS u32x4*)(wn + (n - 5) * 1024);
        SB(); mf(1, n, SETOF(q8)); SB();
        if (n == 1) { woff = (woff == (NWS - 1) * 8192u) ? 0u : woff + 8192u; wn = wBase + woff; asm volatile("" : "+v"(wn)); }
        if (n == 2) sf = UNI(fw);
        if (n == 4) lds_put(pdone, vdone);
        if (n == 7) { CFENCE(); fw = *(LAS unsigned*)(pfw + ((q + 2) & 3) * 4); CFENCE(); }
        SB(); mf(2, n, SETOF(q8)); SB();
        if (q == 0) A[SETOF(q8 + 2 + (n >> 2))][n & 3] = ldA(n & 3, s + 2u + (n >> 2));
        if (q == 1 && n < 4) A[SETOF(q8 + 4)][n] = ldA(n, s + 4u);
        if (n == 1) { vdone += 1; asm volatile("" : "+v"(vdone)); }
        if (n == 3) { nd = (q8 == 7 && s == NSTEP - 1u) ? 0u : s + 2u; asm volatile("" : "+s"(nd)); }
        if (n == 5 && q8 == 4) { CFENCE(); rdy = *(LAS unsigned*)(prdy); CFENCE(); }
        SB(); mf(3, n, SETOF(q8)); SB();
        if (n == 3) {
          if (__builtin_expect(sf < nd, 0))
            while (lds_get(pfw + ((q + 1) & 3) * 4) < nd) {}
          CFENCE();
          SB();
        }
      }
      wc = wn;
    };
    asm volatile("s_nop 7");
    for (unsigned s = 0; s < (unsigned)NSTEP; s += 8) {
      step(s, 0); step(s + 1, 1); step(s + 2, 2); step(s + 3, 3);
      step(s + 4, 4); step(s + 5, 5); step(s + 6, 6); step(s + 7, 7);
    }
    asm volatile("s_nop 15\n\ts_nop 15");
    SB();
    __builtin_amdgcn_s_setprio(0);
    const __amdgpu_buffer_rsrc_t rE = __builtin_amdgcn_make_buffer_rsrc((void*)(ws + WS_X), 0, X_BYTES, 0x00020000);
#pragma unroll
    for (int m = 1; m < 4; m++) {
      const int so = (((b ^ (m << 1)) * 3 + (m - 1)) * 4 + c) * 4096;
#pragma unroll
      for (int t = 0; t < 4; t++) {
        u32x4 o;
        o.x = pk(acc[m][2 * t][0], acc[m][2 * t][1]); o.y = pk(acc[m][2 * t][2], acc[m][2 * t][3]);
        o.z = pk(acc[m][2 * t + 1][0], acc[m][2 * t + 1][1]); o.w = pk(acc[m][2 * t + 1][2], acc[m][2 * t + 1][3]);
        __builtin_amdgcn_raw_buffer_store_b128(o, rE, lane * 16, so + t * 1024, AUX_SYS);
      }
    }
    GAS unsigned* xfl = (GAS unsigned*)(ws + WS_XFLAG);
    asm volatile("s_waitcnt vmcnt(0)" ::: "memory");
    if (lane < 3) __hip_atomic_store(xfl + (((b ^ ((lane + 1) << 1)) * 4 + c) * 3 + lane) * 32, call, RLX_SYS);
    {
      const u64 t0 = now();
      while (true) {
        unsigned v, tv;
        poll32(xfl + ((b * 4 + c) * 3 + min(lane, 2)) * 32, tmo, v, tv);
        if (__all(v >= call) || expired(tmo, tv, call, t0)) break;
      }
      __builtin_amdgcn_fence(__ATOMIC_ACQUIRE, "wavefront");
    }
    lds_count(fl + F_XS * 4);
    u32x4 in[3][4];
#pragma unroll
    for (int m = 0; m < 3; m++)
#pragma unroll
      for (int t = 0; t < 4; t++)
        in[m][t] = __builtin_amdgcn_raw_buffer_load_b128(rE, lane * 16, ((b * 3 + m) * 4 + c) * 4096 + t * 1024, AUX_SYS);
    float am = 0.f;
#pragma unroll
    for (int t = 0; t < 4; t++) {
#pragma unroll
      for (int m = 0; m < 3; m++)
#pragma unroll
        for (int e = 0; e < 4; e++) {
          const unsigned v = in[m][t][e];
          acc[0][2 * t + (e >> 1)][(e & 1) * 2] += bf_lo(v);
          acc[0][2 * t + (e >> 1)][(e & 1) * 2 + 1] += bf_hi(v);
        }
#pragma unroll
      for (int e = 0; e < 8; e++) am = fmaxf(am, fabsf(acc[0][2 * t + (e >> 2)][e & 3]));
    }
    lds_put(fl + (F_MAX + c) * 4, wave_max(__builtin_bit_cast(unsigned, am)));
    if (lds_count(fl + F_CNT * 4) == 3u) {
      const u32x4 m4 = lds_get4(fl + F_MAX * 4);
      const unsigned mx = max(max(m4.x, m4.y), max(m4.z, m4.w));
      if (lane == 0) __hip_atomic_store((GAS u64*)(ws + WS_LOC) + b * 16, ((u64)call << 32) | (u64)mx, RLX_SYS);
      lds_put(fl + F_GS * 4, 1u);
    }
    const unsigned par = call & 1u;
    u32x4 h0, h1;
    h0.x = pk(acc[0][0][0], acc[0][1][0]); h0.y = pk(acc[0][2][0], acc[0][3][0]);
    h0.z = pk(acc[0][0][1], acc[0][1][1]); h0.w = pk(acc[0][2][1], acc[0][3][1]);
    h1.x = pk(acc[0][0][2], acc[0][1][2]); h1.y = pk(acc[0][2][2], acc[0][3][2]);
    h1.z = pk(acc[0][0][3], acc[0][1][3]); h1.w = pk(acc[0][2][3], acc[0][3][3]);
    unsigned char* optr = p.out[dst] + (size_t)(rank * M + rowin) * 512 + colb;
    GAS unsigned* dnp = (GAS unsigned*)(p.ws[dst] + WS_DONE) + (slot * 8 + me) * 32;
    GAS unsigned* efp = (GAS unsigned*)(p.ws[dst] + WS_EF) + ((par * 128 + slot) * 8 + me) * 32;
    asm volatile("" : "+v"(optr), "+v"(dnp), "+v"(efp));
    bool sent = false;
    u32x2 gv;
    {
      unsigned ts = 0;
      while (gv = lds_get2(fl + F_GOK * 4), UNI(gv.x) == 0u) {
        if (remote && ts == 0u && lds_get(fl + F_GS * 4) != 0u) ts = lds_get(fl + F_TS * 4);
        if (ts != 0u && !sent && (int)((unsigned)now() - (ts + (unsigned)c * EGAP)) >= 0) {
          unsigned char* ep = p.ws[dst] + WS_E + (size_t)((par * 8 + me) * 128 + slot) * 2048 + lane * 16;
          store_sys(ep, h0);
          store_sys(ep + 1024, h1);
          asm volatile("s_waitcnt vmcnt(0)" ::: "memory");
          if (lane == 0) __hip_atomic_store(efp, 2u * call + 1u, RLX_SYS);
          sent = true;
        }
      }
    }
    const unsigned rsb = gv.y;
    const float rs = __builtin_bit_cast(float, rsb);
    const bool failed = gv.x != 1u;
    const u32x4 poison = {0x80808080u, 0x80808080u, 0x80808080u, 0x80808080u};
    {
      u32x4 o;
#pragma unroll
      for (int rg = 0; rg < 4; rg++) o[rg] = q4(acc[0][4][rg], acc[0][5][rg], acc[0][6][rg], acc[0][7][rg], rs);
      store_sys(optr + 64, failed ? poison : o);
    }
    if (!sent) store_sys(optr, failed ? poison : q16(h0, h1, rs));
    if (remote) {
      if (!sent && lane == 0) __hip_atomic_store(efp, 2u * call, RLX_SYS);
      asm volatile("s_waitcnt vmcnt(0)" ::: "memory");
      if (lane == 0) __hip_atomic_store(dnp, call, RLX_SYS);
    }
  } else {
    COMMON_DEFS
    {
      Loader L;
      L.l = c;
      L.rW = __builtin_amdgcn_make_buffer_rsrc((void*)p.w, 0, K * N * 4, 0x00020000);
#pragma unroll
      for (int i = 0; i < 8; i++) L.vW[i] = ((lane >> 4) * 8 + i) * (N * 4) + (lane & 15) * 16;
      L.sW = (unsigned)(kq * 1024 + c * 32) * (N * 4u) + (unsigned)nb * 512u;
      L.own = (unsigned)mb2 * 256u;
      L.ldsW = lds0 + lane * 16;
      L.fl = fl;
      L.afl = afl;
      L.tmo = tmo;
      L.po = p.out[j & 7] + (size_t)rank * M * 512;
      L.pw = p.ws[j & 7];
      L.pe = p.ws[j & 7] + WS_E + (size_t)rank * 128 * 2048;
      L.rt = c == 3 && j < NR;
      run_op<0>(L);
      __builtin_amdgcn_s_barrier();
      CFENCE();
      run_rest(L, std::make_integer_sequence<int, NOPS - 1>{});
    }
    const unsigned call = lds_get(fl + F_CALL * 4);
    const unsigned par = call & 1u;
    const u64 t0 = now();
    if (c == 2 && remote) {
      while (lds_get(fl + F_XS * 4) < 4u) __builtin_amdgcn_s_sleep(2);
      GAS u64* bk = (GAS u64*)(ws + WS_BOOK) + dst * 16;
      u64 tb = 0;
      if (lane == 0) {
        __hip_atomic_fetch_max(bk, now() + XBOOK, RLX_AGENT);
        tb = __hip_atomic_fetch_add(bk, (u64)(4u * EGAP), RLX_AGENT);
      }
      lds_put(fl + F_TS * 4, UNI((unsigned)tb) | 1u);
    }
    if (!remote && idx == 0) {
      GAS u64* g = (GAS u64*)(ws + WS_LOC) + (c * 64 + lane) * 16;
      const int srcid = rank;
      GAS u64* mb = (GAS u64*)(p.ws[lane >> 3] + WS_MBOX) + ((par * 8 + (lane & 7)) * 8 + srcid) * 16;
      asm volatile("" : "+v"(mb));
      while (lds_get(fl + F_GS * 4) == 0u) __builtin_amdgcn_s_sleep(2);
      u64 v;
      while (true) {
        unsigned tv;
        poll64(g, tmo, v, tv);
        if (__all((unsigned)(v >> 32) == call) || expired(tmo, tv, call, t0)) break;
      }
      lds_put(fl + (F_LMAX + c) * 4, wave_max((unsigned)v));
      if (lds_count(fl + F_LCNT * 4) == 3u) {
        const u32x4 m4 = lds_get4(fl + F_LMAX * 4);
        const unsigned mx = max(max(m4.x, m4.y), max(m4.z, m4.w));
        __hip_atomic_store(mb, ((u64)call << 32) | (u64)mx, RLX_SYS);
      }
    }
    if (c == 1) {
      while (lds_get(fl + F_GS * 4) == 0u) __builtin_amdgcn_s_sleep(2);
      GAS u64* g = (GAS u64*)(ws + WS_MBOX) + ((par * 8 + xcd) * 8 + (lane & 7)) * 16;
      u64 v;
      bool bad;
      while (true) {
        unsigned tv;
        poll64(g, tmo, v, tv);
        bad = UNI(tv) == call;
        if (__all((unsigned)(v >> 32) == call)) break;
        if (expired(tmo, tv, call, t0)) { bad = true; break; }
      }
      const unsigned mx = wave_max((unsigned)v);
      const float amax = __builtin_bit_cast(float, mx);
      const float rs = amax > 0.f ? 127.0f / amax : 0.f;
      const u32x2 gw = {bad ? 2u : 1u, __builtin_bit_cast(unsigned, rs)};
      asm volatile("ds_write_b64 %0, %1" ::"v"(fl + F_GOK * 4), "v"(gw) : "memory");
    }
    if (remote) {
      GAS unsigned* ef = (GAS unsigned*)(ws + WS_EF) + ((par * 128 + slot) * 8 + dst) * 32;
      while (lds_get(fl + F_GS * 4) == 0u) __builtin_amdgcn_s_sleep(4);
      unsigned e;
      bool bad = false;
      while (true) {
        unsigned tv;
        poll32(ef, tmo, e, tv);
        e = UNI(e);
        if (e >= 2u * call) break;
        if (expired(tmo, tv, call, t0)) { bad = true; break; }
        if (lds_get(fl + F_GOK * 4) == 0u) __builtin_amdgcn_s_sleep(12);
      }
      if (bad || (e & 1u)) {
        __builtin_amdgcn_fence(__ATOMIC_ACQUIRE, "wavefront");
        const __amdgpu_buffer_rsrc_t rI = __builtin_amdgcn_make_buffer_rsrc(
            (void*)(ws + WS_E + (size_t)((par * 8 + dst) * 128 + slot) * 2048), 0, bad ? 0 : 2048, 0x00020000);
        const u32x4 h0 = __builtin_amdgcn_raw_buffer_load_b128(rI, lane * 16, 0, AUX_SYS);
        const u32x4 h1 = __builtin_amdgcn_raw_buffer_load_b128(rI, lane * 16, 1024, AUX_SYS);
        u32x2 gv;
        while (gv = lds_get2(fl + F_GOK * 4), UNI(gv.x) == 0u) __builtin_amdgcn_s_sleep(1);
        const unsigned rsb = gv.y;
        const float rs = __builtin_bit_cast(float, rsb);
        u32x4 o = q16(h0, h1, rs);
        if (bad || gv.x != 1u) o = u32x4{0x80808080u, 0x80808080u, 0x80808080u, 0x80808080u};
        store_sys(p.out[rank] + (size_t)(dst * M + rowin) * 512 + colb, o);
      }
    } else {
      while (lds_get(fl + F_GOK * 4) == 0u) __builtin_amdgcn_s_sleep(2);
      GAS unsigned* d = (GAS unsigned*)(ws + WS_DONE) + (slot * 8 + (lane & 7)) * 32;
      bool bad = false;
      while (true) {
        unsigned v, tv;
        poll32(d, tmo, v, tv);
        if (__all(v >= call || (lane & 7) == rank)) break;
        if (expired(tmo, tv, call, t0)) { bad = true; break; }
      }
      if (bad && lane == 0) p.out[rank][(size_t)(rank * M + rowin) * 512 + colb] = 0x80;
    }
  }
}

extern "C" void kernel_launch(int rank, int nranks, ncclComm_t comm, hipStream_t stream,
                              void* const* d_in, const int* in_sizes, int n_in,
                              void* d_out, int out_size, const int* in_dtypes, int out_dtype,
                              void* const* d_out_all, void* const* d_ws_all, size_t ws_bytes) {
  if (nranks != NR || rank < 0 || rank >= NR || n_in < 2 || ws_bytes < WS_NEED || in_sizes[0] != M * K ||
      in_sizes[1] != K * N || out_size != NR * M * (N / NR)) {
    fprintf(stderr, "unexpected problem shape\n");
    return;
  }
  static bool init = false;
  if (!init) {
    HIP_CHECK(hipFuncSetAttribute((const void*)fused_gemm_a2a, hipFuncAttributeMaxDynamicSharedMemorySize, LDS_BYTES));
    init = true;
  }
  Params p;
  p.x = (const float*)d_in[0];
  p.w = (const float*)d_in[1];
  for (int i = 0; i < NR; i++) { p.out[i] = (unsigned char*)d_out_all[i]; p.ws[i] = (unsigned char*)d_ws_all[i]; }
  p.self = (unsigned char*)d_ws_all[rank];
  p.rank = rank;
  p.pad = 0;
  hipLaunchKernelGGL(fused_gemm_a2a, dim3(NWG), dim3(512), LDS_BYTES, stream, p);
  HIP_CHECK(hipGetLastError());
}
```
